```python
import numpy as np

kernel_name = "gpu_gemm_ar_m4096_k4096_n16384_f32_none"


ABSMAX_RTOL = 0.001

COMPUTE_DTYPE = "float32"
EPILOGUE = "none"
IN_QUANT = None
OUT_QUANT = None
IN_DTYPES = {"x": "float32", "w_mat": "float32"}
WORLD_SIZE = 8

def setup_inputs():

    x = np.random.randn(4096, 4096).astype(np.float32)
    w_mat = (np.random.randn(4096, 16384) / np.sqrt(4096)).astype(np.float32)
    return {"x": x, "w_mat": w_mat}

def _epilogue(y):
    return y

def reference(x, w_mat):
    y = (x @ w_mat).astype(np.float32)
    return _epilogue(y)
```

```cpp
#include <hip/hip_runtime.h>
#include <rccl/rccl.h>
#include <cstdio>

#define HIP_CHECK(expr) do { hipError_t _e = (expr); if (_e != hipSuccess) { \
    fprintf(stderr, "HIP error %s at %s:%d\n", hipGetErrorString(_e), __FILE__, __LINE__); \
    return; } } while (0)

namespace {
#define LAS __attribute__((address_space(3)))
typedef _Float16 f16x8 __attribute__((ext_vector_type(8)));
typedef float f32x4 __attribute__((ext_vector_type(4)));
typedef unsigned u32x4 __attribute__((ext_vector_type(4)));
typedef unsigned u32x2 __attribute__((ext_vector_type(2)));

constexpr int NR = 8, GM = 4096, GN = 16384, KSH = 512;
constexpr int PIECE = 32768, HTB = 16384, NKS = 64;
constexpr unsigned TILE_B = (unsigned)NKS * PIECE;
constexpr int RCOLS = 14, ROUNDS = 4, NRC = RCOLS * ROUNDS;
constexpr unsigned A_BYTES = 16u * TILE_B, B_BYTES = (unsigned)(NRC + 1) * TILE_B, STG_BYTES = A_BYTES + B_BYTES;
constexpr size_t WS_CTR = 0, WS_FLAG = 16384, WS_DONE = 18432, WS_TMO = 20480, WS_DBG = 32768, WS_STG = 1u << 20;
constexpr size_t WS_NEED = WS_STG + 2 * (size_t)STG_BYTES;
constexpr int NCOMM = 16, JOBS1 = 16, JOBS2 = 21, SEQ = 1024;
constexpr int LDS_FLAG = 131072, LDS_BAD = LDS_FLAG + 2048, LDS_BYTES = LDS_BAD + 64;
constexpr unsigned long long DEADLINE = 50000000ull;
constexpr int AUX_WT = 17, AUX_NT = 2;

struct Params { const float* x; const float* w; float* out[NR]; char* ws[NR]; int rank; };

#define WAIT_V(n) asm volatile("s_waitcnt vmcnt(" #n ")" ::: "memory")
#define WAIT_L(n) asm volatile("s_waitcnt lgkmcnt(" #n ")" ::: "memory")
#define BAR __builtin_amdgcn_s_barrier()
#define SCHED __builtin_amdgcn_sched_barrier(0)
#define STAMPC(i) do {} while (0)
#define STAMP(i) do {} while (0)

__device__ __forceinline__ bool wave_wait(const char* words, bool skip, unsigned need, unsigned* tmo, unsigned call) {
  const unsigned long long t0 = __builtin_amdgcn_s_memrealtime();
  bool ok = true;
  for (;;) {
    const unsigned long long v = __hip_atomic_load((const unsigned long long*)words, __ATOMIC_RELAXED, __HIP_MEMORY_SCOPE_SYSTEM);
    const unsigned lo = (unsigned)v, hi = (unsigned)(v >> 32);
    if (__builtin_amdgcn_ballot_w64(skip || (lo < hi ? lo : hi) > need) == ~0ull) break;
    if ((unsigned)__builtin_amdgcn_readfirstlane(__hip_atomic_load(tmo, __ATOMIC_RELAXED, __HIP_MEMORY_SCOPE_AGENT)) == call ||
        __builtin_amdgcn_s_memrealtime() - t0 > DEADLINE) {
      __hip_atomic_store(tmo, call, __ATOMIC_RELAXED, __HIP_MEMORY_SCOPE_AGENT);
      ok = false; break;
    }
    __builtin_amdgcn_s_sleep(8);
  }
  __builtin_amdgcn_fence(__ATOMIC_ACQUIRE, "");
  WAIT_V(0);
  return ok;
}

__device__ __forceinline__ u32x4 pack8(float a0, float a1, float a2, float a3, float a4, float a5, float a6, float a7) {
  f16x8 h = {(_Float16)a0, (_Float16)a1, (_Float16)a2, (_Float16)a3, (_Float16)a4, (_Float16)a5, (_Float16)a6, (_Float16)a7};
  return __builtin_bit_cast(u32x4, h);
}

__global__ void __launch_bounds__(512, 2) fused_kernel(Params p) {
  extern __shared__ __attribute__((aligned(16))) char smem[];
  LAS char* lds = (LAS char*)smem;
  const int tid = threadIdx.x, wid = __builtin_amdgcn_readfirstlane(tid >> 6), lane = tid & 63;
  const int b = blockIdx.x, xcd = b & 7, li = b >> 3, rank = p.rank;
  char* myws = p.ws[rank];
  unsigned* tmo = (unsigned*)(myws + WS_TMO);
  STAMP(0);
  if (tid == 0) {
    const unsigned c = __hip_atomic_fetch_add((unsigned*)(myws + WS_CTR + b * 64), 1u, __ATOMIC_RELAXED, __HIP_MEMORY_SCOPE_AGENT) + 1u;
    *(LAS unsigned*)(lds + LDS_BAD + 4) = c; *(LAS unsigned*)(lds + LDS_BAD) = 0u;
  }
  __syncthreads();
  const unsigned call = __builtin_amdgcn_readfirstlane(*(LAS unsigned*)(lds + LDS_BAD + 4));
  const unsigned seq0 = call * SEQ;
  const size_t stg = WS_STG + (size_t)(call & 1u) * STG_BYTES;
  const char* flagw = myws + WS_FLAG + (lane >> 3) * 128 + (lane & 7) * 8;

  if (li < 2) {
    const int c = li * 8 + xcd, g = lane >> 4, r16 = lane & 15;
    const unsigned kdst = (unsigned)(rank >> 1) * PIECE + (unsigned)(rank & 1) * 1024u;
    __amdgpu_buffer_rsrc_t rd[NR];
#pragma unroll
    for (int d = 0; d < NR; ++d) rd[d] = __builtin_amdgcn_make_buffer_rsrc(p.ws[(rank + 1 + d + wid) & 7] + stg, 0, STG_BYTES, 0x00020000);
    const __amdgpu_buffer_rsrc_t rx = __builtin_amdgcn_make_buffer_rsrc((void*)p.x, 0, GM * KSH * 4, 0x00020000);
    const __amdgpu_buffer_rsrc_t rw = __builtin_amdgcn_make_buffer_rsrc((void*)p.w, 0, KSH * GN * 4, 0x00020000);
    const unsigned vx = (unsigned)(r16 * KSH + 4 * g) * 4u, vw = (unsigned)(4 * g * GN + 4 * r16) * 4u, vd = (unsigned)lane * 16u;
    unsigned* myflag = (unsigned*)(p.ws[lane & 7] + WS_FLAG + rank * 128 + c * 4);
    int done = 0;
#define PUBLISH() do { __builtin_amdgcn_s_waitcnt(0x0F70); WAIT_V(0); BAR; asm volatile("" ::: "memory"); ++done; if (tid < NR) __hip_atomic_store(myflag, seq0 + (unsigned)(done * 16 + c), __ATOMIC_RELAXED, __HIP_MEMORY_SCOPE_SYSTEM); } while (0)
    if (c == NCOMM - 1) {
      const __amdgpu_buffer_rsrc_t rp = __builtin_amdgcn_make_buffer_rsrc(p.ws[wid] + stg, 0, STG_BYTES, 0x00020000);
      for (int hs = 0; hs < JOBS1; ++hs) {
        f32x4 v[4][8];
#pragma unroll
        for (int q = 0; q < 4; ++q)
#pragma unroll
          for (int j = 0; j < 8; ++j)
            v[q][j] = __builtin_bit_cast(f32x4, __builtin_amdgcn_raw_buffer_load_b128(rw, vw, (unsigned)((hs * 32 + (j >> 2) * 16 + (j & 3)) * GN + (NRC + wid) * 256 + q * 64) * 4u, 0));
#pragma unroll
        for (int q = 0; q < 4; ++q)
#pragma unroll
          for (int i = 0; i < 4; ++i)
            __builtin_amdgcn_raw_buffer_store_b128(pack8(v[q][0][i], v[q][1][i], v[q][2][i], v[q][3][i], v[q][4][i], v[q][5][i], v[q][6][i], v[q][7][i]), rp,
                vd + A_BYTES + (unsigned)NRC * TILE_B + (unsigned)hs * 4u * PIECE + kdst + (unsigned)((i >> 1) * HTB + (q * 2 + (i & 1)) * 2048), 0, AUX_WT);
        PUBLISH();
      }
      if (wid == 0) wave_wait(myws + WS_FLAG + rank * 128 + (lane & 7) * 8, false, seq0 + (JOBS1 - 1) * 16 - 1, tmo, call);
      BAR;
    }
    const int q4 = wid & 3, hi = wid >> 2;
    auto job = [&](int s, bool& isx, unsigned& src, unsigned& dst) {
      if (s < JOBS1) {
        isx = c < 8;
        if (isx) { const int mt = 2 * c + hi; src = (unsigned)((mt * 256 + q4 * 64) * KSH + s * 32) * 4u; dst = (unsigned)mt * TILE_B + (unsigned)s * 4u * PIECE + kdst + (unsigned)q4 * 8192u; }
        else { const int col = 2 * (c - 8) + hi; src = (unsigned)(s * 32 * GN + col * 256 + q4 * 64) * 4u; dst = A_BYTES + (unsigned)col * TILE_B + (unsigned)s * 4u * PIECE + kdst + (unsigned)q4 * 4096u; }
      } else {
        const int q = (s - JOBS1) * 16 + c, r = 1 + q / 112, hs = (q % 112) / 7, col = r * RCOLS + 2 * (q % 7) + hi;
        isx = false; src = (unsigned)(hs * 32 * GN + col * 256 + q4 * 64) * 4u; dst = A_BYTES + (unsigned)col * TILE_B + (unsigned)hs * 4u * PIECE + kdst + (unsigned)q4 * 4096u;
      }
    };
    f32x4 v[8];
    auto load = [&](bool isx, unsigned src) {
      if (isx) {
#pragma unroll
        for (int j = 0; j < 8; ++j) v[j] = __builtin_bit_cast(f32x4, __builtin_amdgcn_raw_buffer_load_b128(rx, vx, src + (unsigned)((j >> 1) * 16 * KSH + (j & 1) * 16) * 4u, 0));
      } else {
#pragma unroll
        for (int j = 0; j < 8; ++j) v[j] = __builtin_bit_cast(f32x4, __builtin_amdgcn_raw_buffer_load_b128(rw, vw, src + (unsigned)(((j >> 2) * 16 + (j & 3)) * GN) * 4u, 0));
      }
    };
    bool isx; unsigned src, dst;
    job(done, isx, src, dst); load(isx, src);
    for (int s = done; s < JOBS1 + JOBS2; ++s) {
      u32x4 o[4]; unsigned oo[4];
      if (isx) {
#pragma unroll
        for (int i = 0; i < 4; ++i) { o[i] = pack8(v[2 * i][0], v[2 * i][1], v[2 * i][2], v[2 * i][3], v[2 * i + 1][0], v[2 * i + 1][1], v[2 * i + 1][2], v[2 * i + 1][3]); oo[i] = dst + i * 2048; }
      } else {
#pragma unroll
        for (int i = 0; i < 4; ++i) { o[i] = pack8(v[0][i], v[1][i], v[2][i], v[3][i], v[4][i], v[5][i], v[6][i], v[7][i]); oo[i] = dst + (i >> 1) * HTB + (i & 1) * 2048; }
      }
      if (s + 1 < JOBS1 + JOBS2) { job(s + 1, isx, src, dst); load(isx, src); }
      SCHED;
#pragma unroll
      for (int i = 0; i < 4; ++i)
#pragma unroll
        for (int d = 0; d < NR; ++d) __builtin_amdgcn_raw_buffer_store_b128(o[i], rd[d], vd + oo[i], 0, AUX_WT);
      PUBLISH();
      if (s == JOBS1 - 1) STAMP(1);
    }
    STAMP(2);
    if (b == 0 && wid == 0) {
      const bool ok = wave_wait(myws + WS_DONE + (lane >> 3) * 128 + (lane & 7) * 8, (lane >> 3) == rank, call - 1u, tmo, call);
      if (!ok && lane == 0) __hip_atomic_store((unsigned*)p.out[rank] + (NRC + ((rank + 1) & 7)) * 256, 0x7fc00000u, __ATOMIC_RELAXED, __HIP_MEMORY_SCOPE_SYSTEM);
    }
    STAMP(3);
    return;
  }

  const bool isP = li < 4;
  const int wr = wid >> 2, wc = wid & 3;
  const int lr = li - 4;
  const int mt = isP ? (xcd & 3) * 4 + (xcd >> 2) * 2 + (li - 2) : (xcd & 3) * 4 + (lr & 3);
  const int cr = isP ? 0 : (xcd >> 2) * 7 + (lr >> 2);
  const int rounds = isP ? 1 : ROUNDS;
  const unsigned voff = (unsigned)tid * 16u, ldsw = (unsigned)wid * 1024u;
  const int aoff = wr * 8192 + lane * 16, boff = wc * 4096 + lane * 16;
  const char* fsrc = myws + WS_FLAG + wid * 128 + (lane & 15) * 4;
  const int foff = LDS_FLAG + (lane >> 3) * 256 + (lane & 7) * 8;
#define SA(b_, h) (((b_) * 2 + (h)) * HTB)
#define SB(b_, h) ((4 + (b_) * 2 + (h)) * HTB)
#define STAGE(bufoff, gbase) do { _Pragma("unroll") for (int _i = 0; _i < 2; ++_i) \
    __builtin_amdgcn_global_load_lds((const unsigned*)((const char*)(gbase) + _i * 8192 + voff), (LAS unsigned*)(lds + (bufoff) + ldsw + _i * 8192), 16, 0, 0); } while (0)
#define FLAGDMA() __builtin_amdgcn_global_load_lds((const unsigned*)fsrc, (LAS unsigned*)(lds + LDS_FLAG + wid * 256), 4, 0, AUX_WT)
#define LDA(dst, b_, h) do { _Pragma("unroll") for (int m = 0; m < 4; ++m) _Pragma("unroll") for (int k = 0; k < 2; ++k) dst[m][k] = *(const LAS f16x8*)(lds + SA(b_, h) + aoff + m * 2048 + k * 1024); } while (0)
#define LDB(dst, b_, h) do { _Pragma("unroll") for (int n = 0; n < 2; ++n) _Pragma("unroll") for (int k = 0; k < 2; ++k) dst[n][k] = *(const LAS f16x8*)(lds + SB(b_, h) + boff + n * 2048 + k * 1024); } while (0)
#define MMA(ai, bj, At, Bt) do { __builtin_amdgcn_s_setprio(1); _Pragma("unroll") for (int m = 0; m < 4; ++m) _Pragma("unroll") for (int n = 0; n < 2; ++n) _Pragma("unroll") for (int k = 0; k < 2; ++k) \
    acc[ai][bj][m][n] = __builtin_amdgcn_mfma_f32_16x16x32_f16(At[m][k], Bt[n][k], acc[ai][bj][m][n], 0, 0, 0); __builtin_amdgcn_s_setprio(0); } while (0)
#define ZERO_ACC() do { _Pragma("unroll") for (int a = 0; a < 2; ++a) _Pragma("unroll") for (int c2 = 0; c2 < 2; ++c2) _Pragma("unroll") for (int m = 0; m < 4; ++m) _Pragma("unroll") for (int n = 0; n < 2; ++n) \
    acc[a][c2][m][n] = (f32x4){0.f, 0.f, 0.f, 0.f}; } while (0)
#define SLOW_WAIT(need) do { if (wid == 0) { if (!wave_wait(flagw, false, (need), tmo, call)) *(LAS unsigned*)(lds + LDS_BAD) = 1u; WAIT_L(0); } \
    asm volatile("" ::: "memory"); BAR; asm volatile("" ::: "memory"); } while (0)
  f32x4 acc[2][2][4][2];
  ZERO_ACC();
  f16x8 At[4][2], B0[2][2], B1[2][2];
  const char* cA = myws + stg + (size_t)mt * TILE_B;
  const char* cB = myws + stg + A_BYTES + (size_t)(isP ? NRC : cr) * TILE_B;
  SLOW_WAIT(seq0 + 15u);
  STAMP(1);
  FLAGDMA();
  STAGE(SB(0, 0), cB); STAGE(SA(0, 0), cA); STAGE(SB(0, 1), cB + HTB); STAGE(SA(0, 1), cA + HTB);
  if (wr == 1) BAR;
  WAIT_V(4); BAR;
  STAGE(SB(1, 0), cB + PIECE); STAGE(SA(1, 0), cA + PIECE); STAGE(SB(1, 1), cB + PIECE + HTB);
  WAIT_V(6); BAR;
  for (int r = 0; r < rounds; ++r) {
    const bool has_next = r + 1 < rounds;
    const char* nB = has_next ? cB + (size_t)RCOLS * TILE_B : cB;
    for (int t = 0; t < NKS; t += 2) {
      const bool last = (t == NKS - 2);
      const char* a1 = cA + (size_t)(t + 1) * PIECE;
      const char* a2 = last ? cA : cA + (size_t)(t + 2) * PIECE; const char* b2 = last ? nB : cB + (size_t)(t + 2) * PIECE;
      const char* a3 = a2 + PIECE; const char* b3 = b2 + PIECE;
      unsigned need = 0u;
      if ((t & 2) != 0) {
        const int rn = last ? r + 1 : r, hs = last ? 0 : (t + 2) >> 2;
        need = rn == 0 ? seq0 + (unsigned)(16 * hs + 15) : seq0 + (unsigned)(256 + 112 * (rn - 1) + 7 * hs + (cr >> 1));
        if (last && !has_next) need = 0u;
      }
      const u32x2 fl = *(const LAS u32x2*)(lds + foff);
      LDB(B0, 0, 0); SCHED; LDA(At, 0, 0); STAGE(SA(1, 1), a1 + HTB);
      WAIT_L(8); BAR; WAIT_L(0); MMA(0, 0, At, B0); BAR; SCHED;
      if (need != 0u && __builtin_amdgcn_ballot_w64((fl.x < fl.y ? fl.x : fl.y) > need) != ~0ull) SLOW_WAIT(need);
      LDB(B1, 0, 1); STAGE(SB(0, 0), b2); FLAGDMA();
      BAR; WAIT_L(0); MMA(0, 1, At, B1); BAR;
      LDA(At, 0, 1); STAGE(SA(0, 0), a2);
      BAR; WAIT_L(0); MMA(1, 0, At, B0); BAR; SCHED;
      STAGE(SB(0, 1), b2 + HTB);
      WAIT_V(7); BAR; MMA(1, 1, At, B1); BAR;
      LDB(B0, 1, 0); SCHED; LDA(At, 1, 0); STAGE(SA(0, 1), a2 + HTB);
      WAIT_L(8); BAR; WAIT_L(0); MMA(0, 0, At, B0); BAR; SCHED;
      LDB(B1, 1, 1); STAGE(SB(1, 0), b3);
      BAR; WAIT_L(0); MMA(0, 1, At, B1); BAR;
      LDA(At, 1, 1); STAGE(SA(1, 0), a3);
      BAR; WAIT_L(0); MMA(1, 0, At, B0); BAR; SCHED;
      STAGE(SB(1, 1), b3 + HTB);
      WAIT_V(6); BAR; MMA(1, 1, At, B1); BAR;
    }
    STAMP(2 + 2 * r);
    {
      const float sc = *(const LAS unsigned*)(lds + LDS_BAD) != 0u ? __builtin_nanf("") : 1.0f;
      const int nt = isP ? NRC + rank : r * RCOLS + cr;
      const __amdgpu_buffer_rsrc_t rc = __builtin_amdgcn_make_buffer_rsrc(p.out[rank] + (size_t)(mt * 256 + wr * 64) * GN + nt * 256 + wc * 64, 0, 192u * GN * 4u, 0x00020000);
      const unsigned vo = (unsigned)((lane >> 4) * 4 * GN + (lane & 15) * 4) * 4u;
#pragma unroll
      for (int ai = 0; ai < 2; ++ai)
#pragma unroll
        for (int m = 0; m < 4; ++m)
#pragma unroll
          for (int j = 0; j < 4; ++j) {
            const f32x4 v = {acc[ai][0][m][0][j] * sc, acc[ai][0][m][1][j] * sc, acc[ai][1][m][0][j] * sc, acc[ai][1][m][1][j] * sc};
            __builtin_amdgcn_raw_buffer_store_b128(__builtin_bit_cast(u32x4, v), rc, vo + (unsigned)((ai * 128 + m * 16 + j) * GN * 4), 0, AUX_NT);
          }
    }
    STAMP(3 + 2 * r);
    if (r == 0) STAMPC(20);
    if (r == 3) STAMPC(21);
    if (!has_next) break;
    ZERO_ACC();
    cB = nB;
  }
  WAIT_V(0);
  if (wr == 0) BAR;
  if (isP) {
    if (wid == 0) { wave_wait(myws + WS_FLAG + rank * 128 + (lane & 7) * 8, false, seq0 + (JOBS1 + JOBS2 - 1) * 16 - 1, tmo, call); }
    BAR;
    STAMP(10);
    const float sc = *(const LAS unsigned*)(lds + LDS_BAD) != 0u ? __builtin_nanf("") : 1.0f;
    __amdgpu_buffer_rsrc_t rc[NR - 1];
#pragma unroll
    for (int d = 0; d < NR - 1; ++d) {
      const int dd = (d + wid) % (NR - 1);
      rc[d] = __builtin_amdgcn_make_buffer_rsrc(p.out[(rank + 1 + dd) & 7] + (size_t)(mt * 256 + wr * 64) * GN + (NRC + rank) * 256 + wc * 64, 0, 192u * GN * 4u, 0x00020000);
    }
    const unsigned vo = (unsigned)((lane >> 4) * 4 * GN + (lane & 15) * 4) * 4u;
#pragma unroll
    for (int ai = 0; ai < 2; ++ai)
#pragma unroll
      for (int m = 0; m < 4; ++m)
#pragma unroll
        for (int j = 0; j < 4; ++j) {
          const f32x4 v = {acc[ai][0][m][0][j] * sc, acc[ai][0][m][1][j] * sc, acc[ai][1][m][0][j] * sc, acc[ai][1][m][1][j] * sc};
#pragma unroll
          for (int d = 0; d < NR - 1; ++d) __builtin_amdgcn_raw_buffer_store_b128(__builtin_bit_cast(u32x4, v), rc[d], vo + (unsigned)((ai * 128 + m * 16 + j) * GN * 4), 0, AUX_WT);
        }
    WAIT_V(0); BAR;
    if (tid < NR) __hip_atomic_store((unsigned*)(p.ws[tid] + WS_DONE + rank * 128 + mt * 4), call, __ATOMIC_RELAXED, __HIP_MEMORY_SCOPE_SYSTEM);
    STAMP(11);
  }
}
}

extern "C" void kernel_launch(int rank, int nranks, ncclComm_t comm, hipStream_t stream,
                              void* const* d_in, const int* in_sizes, int n_in,
                              void* d_out, int out_size, const int* in_dtypes, int out_dtype,
                              void* const* d_out_all, void* const* d_ws_all, size_t ws_bytes) {
  (void)comm; (void)in_dtypes; (void)out_dtype; (void)d_out;
  if (nranks != NR || (unsigned)rank >= (unsigned)NR || n_in < 2 || ws_bytes < WS_NEED || out_size != GM * GN ||
      in_sizes[0] != GM * KSH || in_sizes[1] != KSH * GN) { fprintf(stderr, "unexpected problem shape\n"); return; }
  static bool attr_set = false;
  if (!attr_set) { HIP_CHECK(hipFuncSetAttribute((const void*)fused_kernel, hipFuncAttributeMaxDynamicSharedMemorySize, LDS_BYTES)); attr_set = true; }
  Params p;
  p.x = (const float*)d_in[0]; p.w = (const float*)d_in[1]; p.rank = rank;
  for (int r = 0; r < NR; ++r) { p.out[r] = (float*)d_out_all[r]; p.ws[r] = (char*)d_ws_all[r]; }
  fused_kernel<<<256, 512, LDS_BYTES, stream>>>(p);
  HIP_CHECK(hipGetLastError());
}
```

```cpp
#include <hip/hip_runtime.h>
#include <rccl/rccl.h>
#include <cstdio>

#define HIP_CHECK(expr) do { hipError_t _e = (expr); if (_e != hipSuccess) { \
    fprintf(stderr, "HIP error %s at %s:%d\n", hipGetErrorString(_e), __FILE__, __LINE__); \
    return; } } while (0)

namespace {
#define LAS __attribute__((address_space(3)))
typedef _Float16 f16x8 __attribute__((ext_vector_type(8)));
typedef float f32x4 __attribute__((ext_vector_type(4)));
typedef unsigned u32x4 __attribute__((ext_vector_type(4)));
typedef unsigned u32x2 __attribute__((ext_vector_type(2)));

constexpr int NR = 8, GM = 4096, GN = 16384, KSH = 512;
constexpr int PIECE = 32768, HTB = 16384, NKS = 64;
constexpr int RCOLS = 14, ROUNDS = 4, NRC = RCOLS * ROUNDS;
constexpr unsigned A_STEP = 16u * PIECE, B_STEP = (unsigned)RCOLS * PIECE;
constexpr unsigned A_BYTES = NKS * A_STEP, B_BYTES = (unsigned)(ROUNDS * NKS) * B_STEP, P_BYTES = (unsigned)NKS * PIECE;
constexpr unsigned STG_BYTES = A_BYTES + B_BYTES + P_BYTES;
constexpr size_t WS_CTR = 0, WS_FLAG = 16384, WS_DONE = 18432, WS_TMO = 20480, WS_TICKET = 21504, WS_PROG = 22528, WS_PIN = 24576, WS_LCNT = 26624, WS_SENT = 26752, WS_DBG = 32768, WS_STG = 1u << 20;
constexpr size_t WS_NEED = WS_STG + 2 * (size_t)STG_BYTES;
#define AW 1
#define PAW 1
#define PLEAVE 3100
#define PSTART 64
#define PREMODE 1
#define PREDELAY 60
#define FAUX 17
#define WPOS 2
constexpr int U1 = 16 * 120, NUNITS = U1 + 3 * 16 * 56, SEQ = 8192, NAGENT = 16 * (AW + PAW), NRW = 224;
static_assert(NAGENT <= 128, "publisher reads two words per lane at most");
constexpr int LDS_FLAG = 131072, LDS_BAD = LDS_FLAG + 2048, LDS_BYTES = LDS_BAD + 64;
constexpr unsigned long long DEADLINE = 50000000ull;
constexpr int AUX_WT = 17, AUX_NT = 2;

struct Params { const float* x; const float* w; float* out[NR]; char* ws[NR]; int rank; };

#define WAIT_V(n) asm volatile("s_waitcnt vmcnt(" #n ")" ::: "memory")
#define WAIT_L(n) asm volatile("s_waitcnt lgkmcnt(" #n ")" ::: "memory")
#define BAR __builtin_amdgcn_s_barrier()
#define SCHED __builtin_amdgcn_sched_barrier(0)
#define STAMPC(i) do {} while (0)
#define STAMP(i) do {} while (0)

template <bool TWO>
__device__ __forceinline__ bool wave_wait(const char* words, bool skip, unsigned need, unsigned* tmo, unsigned call) {
  const unsigned long long t0 = __builtin_amdgcn_s_memrealtime();
  bool ok = true;
  for (;;) {
    unsigned v;
    if (TWO) {
      const unsigned long long q = __hip_atomic_load((const unsigned long long*)words, __ATOMIC_RELAXED, __HIP_MEMORY_SCOPE_SYSTEM);
      const unsigned lo = (unsigned)q, hi = (unsigned)(q >> 32); v = lo < hi ? lo : hi;
    } else v = __hip_atomic_load((const unsigned*)words, __ATOMIC_RELAXED, __HIP_MEMORY_SCOPE_SYSTEM);
    if (__builtin_amdgcn_ballot_w64(skip || v > need) == ~0ull) break;
    if ((unsigned)__builtin_amdgcn_readfirstlane(__hip_atomic_load(tmo, __ATOMIC_RELAXED, __HIP_MEMORY_SCOPE_AGENT)) == call ||
        __builtin_amdgcn_s_memrealtime() - t0 > DEADLINE) {
      __hip_atomic_store(tmo, call, __ATOMIC_RELAXED, __HIP_MEMORY_SCOPE_AGENT);
      ok = false; break;
    }
    __builtin_amdgcn_s_sleep(8);
  }
  __builtin_amdgcn_fence(__ATOMIC_ACQUIRE, "");
  WAIT_V(0);
  return ok;
}

__device__ __forceinline__ u32x4 pack8(float a0, float a1, float a2, float a3, float a4, float a5, float a6, float a7) {
  f16x8 h = {(_Float16)a0, (_Float16)a1, (_Float16)a2, (_Float16)a3, (_Float16)a4, (_Float16)a5, (_Float16)a6, (_Float16)a7};
  return __builtin_bit_cast(u32x4, h);
}

__global__ void __launch_bounds__(512, 2) fused_kernel(Params p) {
  extern __shared__ __attribute__((aligned(16))) char smem[];
  LAS char* lds = (LAS char*)smem;
  const int tid = threadIdx.x, wid = __builtin_amdgcn_readfirstlane(tid >> 6), lane = tid & 63;
  const int b = blockIdx.x, xcd = b & 7, li = b >> 3, rank = p.rank;
  char* myws = p.ws[rank];
  unsigned* tmo = (unsigned*)(myws + WS_TMO);
  STAMP(0);
  if (tid == 0) {
    const unsigned c = __hip_atomic_fetch_add((unsigned*)(myws + WS_CTR + b * 64), 1u, __ATOMIC_RELAXED, __HIP_MEMORY_SCOPE_AGENT) + 1u;
    *(LAS unsigned*)(lds + LDS_BAD + 4) = c; *(LAS unsigned*)(lds + LDS_BAD) = 0u;
  }
  __syncthreads();
  const unsigned call = __builtin_amdgcn_readfirstlane(*(LAS unsigned*)(lds + LDS_BAD + 4));
  const unsigned seq0 = call * SEQ;
  const size_t stg = WS_STG + (size_t)(call & 1u) * STG_BYTES;
  const char* flagw = myws + WS_FLAG + (lane & 7) * 128;

  const int g4 = lane >> 4, r16 = lane & 15;
  const unsigned vx = (unsigned)(r16 * KSH + 4 * g4) * 4u, vw = (unsigned)(4 * g4 * GN + 4 * r16) * 4u, vd = (unsigned)lane * 16u;
  auto place = [&](bool isx, int col, int q4, int hs, unsigned& src, unsigned& dst) {
    const unsigned ks = (unsigned)(hs * 4 + (rank >> 1)), kk = (unsigned)(rank & 1) * 1024u;
    if (isx) { src = (unsigned)((col * 256 + q4 * 64) * KSH + hs * 32) * 4u; dst = ks * A_STEP + (unsigned)col * PIECE + kk + (unsigned)q4 * 8192u; }
    else {
      src = (unsigned)(hs * 32 * GN + col * 256 + q4 * 64) * 4u;
      const unsigned rr = (unsigned)col / (unsigned)RCOLS;
      dst = A_BYTES + (col < NRC ? (rr * NKS + ks) * B_STEP + ((unsigned)col - rr * RCOLS) * PIECE : B_BYTES + ks * PIECE) + kk + (unsigned)q4 * 4096u;
    }
  };
  auto load_unit = [&](f32x4 (&v)[8], bool valid, bool isx, unsigned src) {
    const __amdgpu_buffer_rsrc_t rs = __builtin_amdgcn_make_buffer_rsrc((void*)(isx ? p.x : p.w), 0, valid ? (isx ? GM * KSH * 4 : KSH * GN * 4) : 0, 0x00020000);
    const unsigned vv = isx ? vx : vw;
#pragma unroll
    for (int j = 0; j < 8; ++j)
      v[j] = __builtin_bit_cast(f32x4, __builtin_amdgcn_raw_buffer_load_b128(rs, vv, src + (isx ? (unsigned)((j >> 1) * 16 * KSH + (j & 1) * 16) * 4u : (unsigned)(((j >> 2) * 16 + (j & 3)) * GN) * 4u), 0));
  };
  auto pack_unit = [&](const f32x4 (&v)[8], bool isx, unsigned dst, u32x4 (&o)[4], unsigned (&oo)[4]) {
#pragma unroll
    for (int i = 0; i < 4; ++i) {
      o[i] = isx ? pack8(v[2 * i][0], v[2 * i][1], v[2 * i][2], v[2 * i][3], v[2 * i + 1][0], v[2 * i + 1][1], v[2 * i + 1][2], v[2 * i + 1][3])
                 : pack8(v[0][i], v[1][i], v[2][i], v[3][i], v[4][i], v[5][i], v[6][i], v[7][i]);
      oo[i] = vd + dst + (isx ? (unsigned)i * 2048u : (unsigned)((i >> 1) * HTB + (i & 1) * 2048));
    }
  };

  if (li < 4) {
    unsigned* prog = (unsigned*)(myws + WS_PROG);
    if (wid < (li < 2 ? AW : PAW)) {
      const unsigned limit = li < 2 ? (unsigned)NUNITS : (unsigned)PLEAVE;
      unsigned* ticket = (unsigned*)(myws + WS_TICKET + (call & 3u) * 128u);
      unsigned* myprog = prog + (li < 2 ? b * AW + wid : 16 * AW + (b - 16) * PAW + wid);
      const __amdgpu_buffer_rsrc_t rprog = __builtin_amdgcn_make_buffer_rsrc(myprog, 0, 4, 0x00020000);
      auto draw = [&](bool real) -> unsigned {
        return (unsigned)__builtin_amdgcn_raw_ptr_buffer_atomic_add_i32(1, __builtin_amdgcn_make_buffer_rsrc(ticket, 0, real ? 4 : 0, 0x00020000), (int)vd, 0, 0); };
      auto progress = [&](unsigned val) { __builtin_amdgcn_raw_buffer_store_b32(val, rprog, vd, 0, 16); };
      auto decode = [&](unsigned u, bool& isx, unsigned& src, unsigned& dst) {
        int hs, col, q4; isx = false;
        if (u < (unsigned)U1) { hs = u / 120u; const int i = u % 120u; q4 = i & 3; isx = i < 64; col = isx ? i >> 2 : (i - 64) >> 2; }
        else { const unsigned v = u - U1; const int rr = v / 896u; hs = (v % 896u) / 56u; const int i = v % 56u; q4 = i & 3; col = (1 + rr) * RCOLS + (i >> 2); }
        place(isx, col, q4, hs, src, dst);
      };
      __amdgpu_buffer_rsrc_t rd[NR - 1];
#pragma unroll
      for (int d = 0; d < NR - 1; ++d) rd[d] = __builtin_amdgcn_make_buffer_rsrc(p.ws[(rank + 1 + (d + b + wid) % (NR - 1)) & 7] + stg, 0, STG_BYTES, 0x00020000);
      f32x4 v[8];
      unsigned cur = __builtin_amdgcn_readfirstlane(draw(true));
      bool isx; unsigned src, dst;
      decode(cur < (unsigned)NUNITS ? cur : 0u, isx, src, dst); load_unit(v, cur < (unsigned)NUNITS, isx, src);
      bool real = true; unsigned tnext = draw(true);
      progress(seq0 + cur);
#pragma unroll
      for (int j = 0; j < 8; ++j) asm volatile("" : "+v"(v[j]));
      asm volatile("" : "+v"(tnext));
      while (cur < (unsigned)NUNITS) {
        u32x4 o[4]; unsigned oo[4];
        pack_unit(v, isx, dst, o, oo);
        const unsigned nxt = real ? __builtin_amdgcn_readfirstlane(tnext) : 0xffffffffu;
        SCHED;
        decode(nxt < (unsigned)NUNITS ? nxt : 0u, isx, src, dst); load_unit(v, nxt < (unsigned)NUNITS, isx, src);
        real = nxt < limit; tnext = draw(real);
        SCHED;
#pragma unroll
        for (int i = 0; i < WPOS; ++i)
#pragma unroll
          for (int d = 0; d < NR - 1; ++d) __builtin_amdgcn_raw_buffer_store_b128(o[i], rd[d], oo[i], 0, AUX_WT);
        asm volatile("s_waitcnt vmcnt(%0)" :: "i"(9 + (NR - 1) * WPOS) : "memory");
        progress(seq0 + cur);
        SCHED;
#pragma unroll
        for (int i = WPOS; i < 4; ++i)
#pragma unroll
          for (int d = 0; d < NR - 1; ++d) __builtin_amdgcn_raw_buffer_store_b128(o[i], rd[d], oo[i], 0, AUX_WT);
        SCHED;
        cur = nxt;
      }
      WAIT_V(0);
      progress(seq0 + (unsigned)(SEQ - 1));
      STAMP(12);
    } else if (li < 2 && wid == 6) {
      wave_wait<false>(myws + WS_SENT, false, seq0 + (unsigned)(U1 - 64), tmo, call);
      for (int d = 1; d < NR; ++d) {
        const int pr = (rank + 1 + (d + b) % (NR - 1)) & 7;
        const __amdgpu_buffer_rsrc_t rp = __builtin_amdgcn_make_buffer_rsrc(p.ws[pr] + stg, 0, STG_BYTES, 0x00020000);
        for (int q2 = 0; q2 < 4; q2 += 2) {
          f32x4 v[2][8]; unsigned src, dst[2];
#pragma unroll
          for (int e = 0; e < 2; ++e) { place(false, NRC + pr, q2 + e, b, src, dst[e]); load_unit(v[e], true, false, src); }
#pragma unroll
          for (int e = 0; e < 2; ++e) { u32x4 o[4]; unsigned oo[4]; pack_unit(v[e], false, dst[e], o, oo);
#pragma unroll
            for (int i = 0; i < 4; ++i) __builtin_amdgcn_raw_buffer_store_b128(o[i], rp, oo[i], 0, AUX_WT); }
        }
      }
      WAIT_V(0);
      if (lane < NR) __hip_atomic_store((unsigned*)(p.ws[lane] + WS_PIN + rank * 128 + b * 4), call, __ATOMIC_RELAXED, __HIP_MEMORY_SCOPE_SYSTEM);
    } else if (b == 0 && wid == 7) {
      if (lane == 0) __hip_atomic_store((unsigned*)(myws + WS_TICKET + ((call + 2u) & 3u) * 128u), 0u, __ATOMIC_RELAXED, __HIP_MEMORY_SCOPE_AGENT);
      unsigned* myflag = (lane & 7) == rank ? (unsigned*)(myws + WS_SENT) : (unsigned*)(p.ws[lane & 7] + WS_FLAG + rank * 128);
      const unsigned long long t0 = __builtin_amdgcn_s_memrealtime();
      unsigned last = seq0; bool ok = true, staged = false;
      while (last < seq0 + (unsigned)NUNITS || !staged) {
        unsigned m = 0xffffffffu;
#pragma unroll
        for (int j = 0; j < (NAGENT + 63) / 64; ++j) { const int a = j * 64 + lane; if (a < NAGENT) { const unsigned x = __hip_atomic_load(prog + a, __ATOMIC_RELAXED, __HIP_MEMORY_SCOPE_AGENT); m = x < m ? x : m; } }
        if (!staged && (unsigned)__builtin_amdgcn_readfirstlane(__hip_atomic_load((unsigned*)(myws + WS_LCNT), __ATOMIC_RELAXED, __HIP_MEMORY_SCOPE_AGENT)) >= call * (unsigned)NRW) {
          staged = true;
          if (lane == 0) __hip_atomic_store((unsigned*)(myws + WS_FLAG + rank * 128), seq0 + (unsigned)(SEQ - 1), __ATOMIC_RELAXED, __HIP_MEMORY_SCOPE_SYSTEM);
        }
#pragma unroll
        for (int sft = 32; sft >= 1; sft >>= 1) { const unsigned x = __shfl_xor(m, sft); m = x < m ? x : m; }
        m = __builtin_amdgcn_readfirstlane(m);
        if (m > last) { last = m; if (lane < NR) __hip_atomic_store(myflag, m, __ATOMIC_RELAXED, __HIP_MEMORY_SCOPE_SYSTEM); }
        if (__builtin_amdgcn_s_memrealtime() - t0 > DEADLINE) { __hip_atomic_store(tmo, call, __ATOMIC_RELAXED, __HIP_MEMORY_SCOPE_AGENT); ok = false; break; }
      }
      ok = wave_wait<true>(myws + WS_DONE + (lane >> 3) * 128 + (lane & 7) * 8, (lane >> 3) == rank, call - 1u, tmo, call) && ok;
      if (!ok && lane == 0) __hip_atomic_store((unsigned*)p.out[rank] + (NRC + ((rank + 1) & 7)) * 256, 0x7fc00000u, __ATOMIC_RELAXED, __HIP_MEMORY_SCOPE_SYSTEM);
    }
    if (li < 2) return;
  } else {
    const __amdgpu_buffer_rsrc_t rl = __builtin_amdgcn_make_buffer_rsrc(myws + stg, 0, STG_BYTES, 0x00020000);
    f32x4 v[3][8]; unsigned dst[3]; bool isx[3], val[3];
    { const unsigned long long t0 = __builtin_amdgcn_s_memrealtime(); while (__builtin_amdgcn_s_memrealtime() - t0 < (unsigned long long)(PREDELAY * (li - 3))) __builtin_amdgcn_s_sleep(4); }
#pragma unroll
    for (int e = 0; e < 3; ++e) {
      const int j = (b - 32) + e * NRW; unsigned src;
      isx[e] = j < 128; val[e] = j < 584;
      if (j < 128) place(true, 2 * (j & 7) + (wid >> 2), wid & 3, j >> 3, src, dst[e]);
      else if (j < 576) { const int c2 = 2 * ((j - 128) % 28) + (wid >> 2); place(false, c2, wid & 3, (j - 128) / 28, src, dst[e]); }
      else place(false, NRC + rank, wid & 3, (2 * (j - 576) + (wid >> 2)) & 15, src, dst[e]);
      load_unit(v[e], val[e], isx[e], src);
      if (!val[e]) continue;
      u32x4 o[4]; unsigned oo[4]; pack_unit(v[e], isx[e], dst[e], o, oo);
#pragma unroll
      for (int i = 0; i < 4; ++i) __builtin_amdgcn_raw_buffer_store_b128(o[i], rl, oo[i], 0, AUX_WT);
      WAIT_V(0);
    }
    WAIT_V(0); BAR; asm volatile("" ::: "memory");
    if (tid == 0) __hip_atomic_fetch_add((unsigned*)(myws + WS_LCNT), 1u, __ATOMIC_RELAXED, __HIP_MEMORY_SCOPE_AGENT);
    STAMP(13);
  }

  const bool isP = li < 4;
  const int wr = wid >> 2, wc = wid & 3;
  const int lr = li - 4;
  const int mt = isP ? (xcd & 3) * 4 + (xcd >> 2) * 2 + (li - 2) : (xcd & 3) * 4 + (lr & 3);
  const int cr = isP ? 0 : (xcd >> 2) * 7 + (lr >> 2);
  const int rounds = isP ? 1 : ROUNDS;
  const unsigned voff = (unsigned)(wid * 2048 + lane * 16), ldsw = (unsigned)wid * 2048u;
  const int aoff = wr * 8192 + lane * 16, boff = wc * 4096 + lane * 16;
  const char* fsrc = myws + WS_FLAG + wid * 128;
  const int foff = LDS_FLAG + (lane & 7) * 256;
  const __amdgpu_buffer_rsrc_t ra = __builtin_amdgcn_make_buffer_rsrc(myws + stg, 0, A_BYTES, 0x00020000);
  const __amdgpu_buffer_rsrc_t rb = __builtin_amdgcn_make_buffer_rsrc(myws + stg + A_BYTES + (isP ? B_BYTES : 0u), 0, isP ? P_BYTES : B_BYTES, 0x00020000);
  const unsigned bstep = isP ? (unsigned)PIECE : B_STEP;
#define SA(b_, h) (((b_) * 2 + (h)) * HTB)
#define SB(b_, h) ((4 + (b_) * 2 + (h)) * HTB)
#define STAGE(bufoff, rs, so) do { \
    __builtin_amdgcn_raw_ptr_buffer_load_lds(rs, (LAS void*)(lds + (bufoff) + ldsw), 16, voff, (so), 0, 0); \
    __builtin_amdgcn_raw_ptr_buffer_load_lds(rs, (LAS void*)(lds + (bufoff) + ldsw), 16, voff, (so), 1024, 0); } while (0)
#define FLAGDMA() __builtin_amdgcn_global_load_lds((const unsigned*)fsrc, (LAS unsigned*)(lds + LDS_FLAG + wid * 256), 4, 0, FAUX)
#define LDA(dst, b_, h) do { _Pragma("unroll") for (int m = 0; m < 4; ++m) _Pragma("unroll") for (int k = 0; k < 2; ++k) dst[m][k] = *(const LAS f16x8*)(lds + SA(b_, h) + aoff + m * 2048 + k * 1024); } while (0)
#define LDB(dst, b_, h) do { _Pragma("unroll") for (int n = 0; n < 2; ++n) _Pragma("unroll") for (int k = 0; k < 2; ++k) dst[n][k] = *(const LAS f16x8*)(lds + SB(b_, h) + boff + n * 2048 + k * 1024); } while (0)
#define MMA2(ai) do { __builtin_amdgcn_s_waitcnt(0xC07F); SCHED; BAR; SCHED; __builtin_amdgcn_s_setprio(1); SCHED; \
    _Pragma("unroll") for (int m = 0; m < 4; ++m) _Pragma("unroll") for (int q = 0; q < 4; ++q) { const int nn = (m & 1) ? 3 - q : q; \
      asm volatile("v_mfma_f32_16x16x32_f16 %0, %1, %2, %0\n\tv_mfma_f32_16x16x32_f16 %0, %3, %4, %0" : "+v"(acc[ai][nn >> 1][m][nn & 1]) \
          : "v"(At[m][0]), "v"((nn >> 1) ? B1[nn & 1][0] : B0[nn & 1][0]), "v"(At[m][1]), "v"((nn >> 1) ? B1[nn & 1][1] : B0[nn & 1][1])); } \
    SCHED; __builtin_amdgcn_s_setprio(0); BAR; SCHED; } while (0)
#define ZERO_ACC() do { _Pragma("unroll") for (int a = 0; a < 2; ++a) _Pragma("unroll") for (int c2 = 0; c2 < 2; ++c2) _Pragma("unroll") for (int m = 0; m < 4; ++m) _Pragma("unroll") for (int n = 0; n < 2; ++n) \
    acc[a][c2][m][n] = (f32x4){0.f, 0.f, 0.f, 0.f}; } while (0)
#define SLOW_WAIT(need) do { if (wid == 0) { if (!wave_wait<false>(flagw, false, (need), tmo, call)) *(LAS unsigned*)(lds + LDS_BAD) = 1u; WAIT_L(0); } \
    asm volatile("" ::: "memory"); BAR; asm volatile("" ::: "memory"); } while (0)
#define SEG1(bf, an) do { LDB(B0, bf, 0); LDB(B1, bf, 1); LDA(At, bf, 0); STAGE(SA(1 - (bf), 1), ra, (an) + HTB); } while (0)
#define SEG2(bf, a2, b2) do { LDA(At, bf, 1); STAGE(SB(bf, 0), rb, (b2)); STAGE(SA(bf, 0), ra, (a2)); STAGE(SB(bf, 1), rb, (b2) + HTB); } while (0)
  f32x4 acc[2][2][4][2];
  ZERO_ACC();
  f16x8 At[4][2], B0[2][2], B1[2][2];
  const unsigned cA = (unsigned)mt * PIECE;
  unsigned bo = (unsigned)cr * PIECE;
  unsigned need = seq0 + (unsigned)(67 + 4 * cr);
  if (isP && wid == 0) { if (!wave_wait<true>(myws + WS_PIN + (lane >> 3) * 128 + (lane & 7) * 8, (lane >> 3) == rank, call - 1u, tmo, call)) *(LAS unsigned*)(lds + LDS_BAD) = 1u; }
  SLOW_WAIT(isP ? seq0 + (unsigned)(U1 - 1) : need);
  STAMP(1);
  STAGE(SB(0, 0), rb, bo); STAGE(SA(0, 0), ra, cA); STAGE(SB(0, 1), rb, bo + HTB); STAGE(SA(0, 1), ra, cA + HTB);
  if (wr == 1) BAR;
  WAIT_V(2); BAR;
  STAGE(SB(1, 0), rb, bo + bstep); STAGE(SA(1, 0), ra, cA + A_STEP); STAGE(SB(1, 1), rb, bo + bstep + HTB);
  BAR;
  for (int r = 0; r < rounds; ++r) {
    const bool has_next = r + 1 < rounds;
    for (int hs = 0; hs < 16; ++hs) {
      const unsigned ao = cA + (unsigned)hs * 4u * A_STEP, an = cA + (unsigned)((hs + 1) & 15) * 4u * A_STEP;
      need += (r == 0 && hs < 15) ? 120u : 56u;
      const bool check = !isP && (hs < 15 || has_next);
      SEG1(0, ao + A_STEP); WAIT_V(8); FLAGDMA(); MMA2(0);
      SEG2(0, ao + 2 * A_STEP, bo + 2 * bstep); WAIT_V(9); MMA2(1);
      SEG1(1, ao + 2 * A_STEP); WAIT_V(9); MMA2(0);
      SEG2(1, ao + 3 * A_STEP, bo + 3 * bstep); WAIT_V(8); MMA2(1);
      const unsigned fl = *(const LAS unsigned*)(lds + foff);
      SEG1(0, ao + 3 * A_STEP); WAIT_V(8); MMA2(0);
      if (check && __builtin_amdgcn_ballot_w64(fl > need) != ~0ull) SLOW_WAIT(need);
      SEG2(0, an, bo + 4 * bstep); WAIT_V(8); MMA2(1);
      SEG1(1, an); WAIT_V(8); MMA2(0);
      SEG2(1, an + A_STEP, bo + 5 * bstep); WAIT_V(8); MMA2(1);
      bo += 4 * bstep;
    }
    asm volatile("s_nop 15\n\ts_nop 15" ::: "memory");
    SCHED;
    STAMP(2 + 2 * r);
    {
      const float sc = *(const LAS unsigned*)(lds + LDS_BAD) != 0u ? __builtin_nanf("") : 1.0f;
      const int nt = isP ? NRC + rank : r * RCOLS + cr;
      const __amdgpu_buffer_rsrc_t rc = __builtin_amdgcn_make_buffer_rsrc(p.out[rank] + (size_t)(mt * 256 + wr * 64) * GN + nt * 256 + wc * 64, 0, 192u * GN * 4u, 0x00020000);
      unsigned vo = (unsigned)((lane >> 4) * 4 * GN + (lane & 15) * 4) * 4u; asm volatile("" : "+v"(vo));
#pragma unroll
      for (int ai = 0; ai < 2; ++ai)
#pragma unroll
        for (int m = 0; m < 4; ++m)
#pragma unroll
          for (int j = 0; j < 4; ++j) {
            const f32x4 v = {acc[ai][0][m][0][j] * sc, acc[ai][0][m][1][j] * sc, acc[ai][1][m][0][j] * sc, acc[ai][1][m][1][j] * sc};
            __builtin_amdgcn_raw_buffer_store_b128(__builtin_bit_cast(u32x4, v), rc, vo + (unsigned)((ai * 128 + m * 16 + j) * GN * 4), 0, AUX_NT);
          }
    }
    STAMP(3 + 2 * r);
    STAMPC(20 + r);
    if (!has_next) break;
    ZERO_ACC();
  }
  WAIT_V(0);
  if (wr == 0) BAR;
  if (isP) {
    if (wid == 0) { wave_wait<false>(myws + WS_SENT, false, seq0 + (unsigned)(NUNITS - PSTART), tmo, call); }
    BAR;
    STAMP(10);
    const float sc = *(const LAS unsigned*)(lds + LDS_BAD) != 0u ? __builtin_nanf("") : 1.0f;
    __amdgpu_buffer_rsrc_t rc[NR - 1];
#pragma unroll
    for (int d = 0; d < NR - 1; ++d) {
      const int dd = (d + wid) % (NR - 1);
      rc[d] = __builtin_amdgcn_make_buffer_rsrc(p.out[(rank + 1 + dd) & 7] + (size_t)(mt * 256 + wr * 64) * GN + (NRC + rank) * 256 + wc * 64, 0, 192u * GN * 4u, 0x00020000);
    }
    const unsigned vo = (unsigned)((lane >> 4) * 4 * GN + (lane & 15) * 4) * 4u;
#pragma unroll
    for (int ai = 0; ai < 2; ++ai)
#pragma unroll
      for (int m = 0; m < 4; ++m)
#pragma unroll
        for (int j = 0; j < 4; ++j) {
          const f32x4 v = {acc[ai][0][m][0][j] * sc, acc[ai][0][m][1][j] * sc, acc[ai][1][m][0][j] * sc, acc[ai][1][m][1][j] * sc};
#pragma unroll
          for (int d = 0; d < NR - 1; ++d) __builtin_amdgcn_raw_buffer_store_b128(__builtin_bit_cast(u32x4, v), rc[d], vo + (unsigned)((ai * 128 + m * 16 + j) * GN * 4), 0, AUX_WT);
        }
    WAIT_V(0); BAR;
    if (tid < NR) __hip_atomic_store((unsigned*)(p.ws[tid] + WS_DONE + rank * 128 + mt * 4), call, __ATOMIC_RELAXED, __HIP_MEMORY_SCOPE_SYSTEM);
    STAMP(11);
  }
}
}

extern "C" void kernel_launch(int rank, int nranks, ncclComm_t comm, hipStream_t stream,
                              void* const* d_in, const int* in_sizes, int n_in,
                              void* d_out, int out_size, const int* in_dtypes, int out_dtype,
                              void* const* d_out_all, void* const* d_ws_all, size_t ws_bytes) {
  (void)comm; (void)in_dtypes; (void)out_dtype; (void)d_out;
  if (nranks != NR || (unsigned)rank >= (unsigned)NR || n_in < 2 || ws_bytes < WS_NEED || out_size != GM * GN ||
      in_sizes[0] != GM * KSH || in_sizes[1] != KSH * GN) { fprintf(stderr, "unexpected problem shape\n"); return; }
  static bool attr_set = false;
  if (!attr_set) { HIP_CHECK(hipFuncSetAttribute((const void*)fused_kernel, hipFuncAttributeMaxDynamicSharedMemorySize, LDS_BYTES)); attr_set = true; }
  Params p;
  p.x = (const float*)d_in[0]; p.w = (const float*)d_in[1]; p.rank = rank;
  for (int r = 0; r < NR; ++r) { p.out[r] = (float*)d_out_all[r]; p.ws[r] = (char*)d_ws_all[r]; }
  fused_kernel<<<256, 512, LDS_BYTES, stream>>>(p);
  HIP_CHECK(hipGetLastError());
}
```

```cpp
#include <hip/hip_runtime.h>
#include <rccl/rccl.h>
#include <cstdio>

#define HIP_CHECK(expr) do { hipError_t _e = (expr); if (_e != hipSuccess) { \
    fprintf(stderr, "HIP error %s at %s:%d\n", hipGetErrorString(_e), __FILE__, __LINE__); \
    return; } } while (0)

namespace {
#define LAS __attribute__((address_space(3)))
typedef _Float16 f16x8 __attribute__((ext_vector_type(8)));
typedef float f32x4 __attribute__((ext_vector_type(4)));
typedef unsigned u32x4 __attribute__((ext_vector_type(4)));
typedef unsigned u32x2 __attribute__((ext_vector_type(2)));

#define KKL_ 0
constexpr int NR = 8, GM = 4096, GN = 16384, KSH = 512;
constexpr int PIECE = 32768, HTB = 16384, NKS = 64;
constexpr int FRS = KKL_ ? 1024 : 2048, KHS = KKL_ ? 8192 : 1024;
constexpr int RCOLS = 14, ROUNDS = 4, NRC = RCOLS * ROUNDS;
constexpr unsigned A_STEP = 16u * PIECE, B_STEP = (unsigned)RCOLS * PIECE;
constexpr unsigned A_BYTES = NKS * A_STEP, B_BYTES = (unsigned)(ROUNDS * NKS) * B_STEP, P_BYTES = (unsigned)NKS * PIECE;
constexpr unsigned STG_BYTES = A_BYTES + B_BYTES + P_BYTES;
constexpr size_t WS_CTR = 0, WS_FLAG = 16384, WS_TMO = 20480, WS_TICKET = 21504, WS_PROG = 22528, WS_PIN = 24576, WS_LCNT = 26624, WS_SENT = 26752, WS_PCNT = 26880, WS_PINC = 27008, WS_LCNT2 = 27136, WS_DCNT = 27264, WS_LMAX = 27392, WS_GO = 27520, WS_AMX = 28672, WS_DBG = 32768, WS_STG = 1u << 20;
constexpr size_t WS_PARK = WS_STG + 2 * (size_t)STG_BYTES;
constexpr size_t WS_NEED = WS_PARK + 3 * 256 * (size_t)262144;
#define JOIN2 2950
#define PLEAVE 3050
#define PREDELAY 40
#define FAUX 16
#define PSTART 64
#define OPERM 1
#define PRENT 0
#define POLLS 16
#define POLLA 1
#define NLMODE 0
#define PINAT 2500
#define ADDLATE 1
#define PHS 1
#define WIN 40
constexpr int U1 = 8 * 240, NUNITS = U1 + 24 * 112, NOUT = GM / 4, SEQ = 8192, NAGENT = 32, NPROG = 48, NRW = 224;
constexpr int SHIFT0 = 8;
constexpr int LDS_FLAG = 131072, LDS_BAD = LDS_FLAG + 2048, LDS_RING = LDS_BAD + 128, LDS_BYTES = LDS_RING + 3 * 8192;
constexpr unsigned long long DEADLINE = 50000000ull;
constexpr int AUX_WT = 17, AUX_NT = 2;

template <int V> struct IC { static constexpr int value = V; };
struct Params { const float* x; const float* w; float* out[NR]; char* ws[NR]; int rank; };

#define WAIT_V(n) asm volatile("s_waitcnt vmcnt(" #n ")" ::: "memory")
#define WAIT_L(n) asm volatile("s_waitcnt lgkmcnt(" #n ")" ::: "memory")
#define BAR __builtin_amdgcn_s_barrier()
#define SCHED __builtin_amdgcn_sched_barrier(0)
#define STAMPC(i) do {} while (0)
#define STAMPW(i) do {} while (0)
#define STAMP(i) do {} while (0)

template <int ACQ = 2>
__device__ __forceinline__ bool wave_wait(const char* words, bool skip, unsigned need, unsigned* tmo, unsigned call, unsigned* seen = nullptr) {
  const unsigned long long t0 = __builtin_amdgcn_s_memrealtime();
  bool ok = true;
  for (unsigned n = 1;; ++n) {
    const unsigned v = (POLLA && (n & 7u)) ? __hip_atomic_load((const unsigned*)words, __ATOMIC_RELAXED, __HIP_MEMORY_SCOPE_AGENT)
                                           : __hip_atomic_load((const unsigned*)words, __ATOMIC_RELAXED, __HIP_MEMORY_SCOPE_SYSTEM);
    if (seen) *seen = v;
    if (__builtin_amdgcn_ballot_w64(skip || v > need) == ~0ull) break;
    if ((n & 7u) == 0u &&
        ((unsigned)__builtin_amdgcn_readfirstlane(__hip_atomic_load(tmo, __ATOMIC_RELAXED, __HIP_MEMORY_SCOPE_AGENT)) == call ||
         __builtin_amdgcn_s_memrealtime() - t0 > DEADLINE)) {
      __hip_atomic_store(tmo, call, __ATOMIC_RELAXED, __HIP_MEMORY_SCOPE_AGENT);
      ok = false; break;
    }
    __builtin_amdgcn_s_sleep(ACQ ? POLLS : 1);
  }
  if (ACQ) __builtin_amdgcn_fence(__ATOMIC_ACQUIRE, "");
  if (ACQ == 2) WAIT_V(0);
  return ok;
}

__device__ __forceinline__ float pow2f(int e) { return __builtin_bit_cast(float, (unsigned)(e + 127) << 23); }
__device__ __forceinline__ u32x4 pack8(float m, float a0, float a1, float a2, float a3, float a4, float a5, float a6, float a7) {
  f16x8 h = {(_Float16)(a0 * m), (_Float16)(a1 * m), (_Float16)(a2 * m), (_Float16)(a3 * m), (_Float16)(a4 * m), (_Float16)(a5 * m), (_Float16)(a6 * m), (_Float16)(a7 * m)};
  return __builtin_bit_cast(u32x4, h);
}

#define SEND_UNIT(NPRE, REPORT) do { \
    _Pragma("unroll") for (int i = 0; i < 4; ++i) { \
      if (i == 0) asm volatile("s_waitcnt vmcnt(%0)" :: "i"((WIN) + (NPRE)) : "memory"); \
      else if (i == 1) asm volatile("s_waitcnt vmcnt(%0)" :: "i"((WIN) > 7 ? (WIN) + (NPRE) : (WIN)) : "memory"); \
      else if (i == 2) { asm volatile("s_waitcnt vmcnt(%0)" :: "i"((WIN) > 14 ? 14 + (NPRE) : (WIN)) : "memory");     \
        if (REPORT) progress(seq0 + cur); } \
      else asm volatile("s_waitcnt vmcnt(%0)" :: "i"((WIN) + 1) : "memory"); \
      SCHED; \
      _Pragma("unroll") for (int d = 0; d < NR - 1; ++d) __builtin_amdgcn_raw_buffer_store_b128(o[i], rd[d], oo[i], 0, AUX_WT); \
      SCHED; } } while (0)

__global__ void __launch_bounds__(512, 2) fused_kernel(Params p) {
  extern __shared__ __attribute__((aligned(16))) char smem[];
  LAS char* lds = (LAS char*)smem;
  const int tid = threadIdx.x, wid = __builtin_amdgcn_readfirstlane(tid >> 6), lane = tid & 63;
  const int b = blockIdx.x, xcd = b & 7, li = b >> 3, rank = p.rank;
  char* myws = p.ws[rank];
  unsigned* tmo = (unsigned*)(myws + WS_TMO);
  STAMP(0);
  const int g4 = lane >> 4, r16 = lane & 15;
  const unsigned vx = (unsigned)(r16 * KSH + 4 * g4) * 4u, vw = (unsigned)(4 * g4 * GN + 4 * r16) * 4u, vd = (unsigned)lane * 16u;
  auto phase_of = [&](int cr) -> int { return PHS ? (cr >> 2) + (((cr >> 1) & 1) && rank < 4 ? 1 : 0) : 0; };
  auto place = [&](bool isx, int col, int q4, int hs, unsigned& src, unsigned& dst) {
    const unsigned ks = (unsigned)((hs >> 1) * 8 + rank), kk = (unsigned)((hs & 1) * KHS);
    if (isx) { src = (unsigned)((col * 256 + q4 * 64) * KSH + hs * 32) * 4u; dst = ks * A_STEP + (unsigned)col * PIECE + kk + (unsigned)((q4 >> 1) * HTB + (q4 & 1) * 4 * FRS); }
    else {
      const int rr = col / RCOLS, cc = col - rr * RCOLS;
      const int wcol = col < NRC ? ((rr + ((hs >> 1) >= phase_of(cc) ? 1 : 0)) & 3) * RCOLS + cc : col;
      src = (unsigned)(hs * 32 * GN + wcol * 256 + q4 * 64) * 4u;
      dst = A_BYTES + (col < NRC ? ((unsigned)rr * NKS + ks) * B_STEP + (unsigned)cc * PIECE : B_BYTES + ks * PIECE) + kk + (unsigned)(q4 * 2 * FRS);
    }
  };
  auto load_unit = [&](f32x4 (&v)[8], bool valid, bool isx, unsigned src, auto aux) {
    const __amdgpu_buffer_rsrc_t rs = __builtin_amdgcn_make_buffer_rsrc((void*)(isx ? p.x : p.w), 0, valid ? (isx ? GM * KSH * 4 : KSH * GN * 4) : 0, 0x00020000);
    const unsigned vv = isx ? vx : vw;
#pragma unroll
    for (int j = 0; j < 8; ++j)
      v[j] = __builtin_bit_cast(f32x4, __builtin_amdgcn_raw_buffer_load_b128(rs, vv, src + (isx ? (unsigned)((j >> 1) * 16 * KSH + (j & 1) * 16) * 4u : (unsigned)(((j >> 2) * 16 + (j & 3)) * GN) * 4u), decltype(aux)::value));
  };
  auto pack_unit = [&](const f32x4 (&v)[8], float m, bool isx, unsigned dst, u32x4 (&o)[4], unsigned (&oo)[4]) {
#pragma unroll
    for (int i = 0; i < 4; ++i) {
      o[i] = isx ? pack8(m, v[2 * i][0], v[2 * i][1], v[2 * i][2], v[2 * i][3], v[2 * i + 1][0], v[2 * i + 1][1], v[2 * i + 1][2], v[2 * i + 1][3])
                 : pack8(m, v[0][i], v[1][i], v[2][i], v[3][i], v[4][i], v[5][i], v[6][i], v[7][i]);
      oo[i] = vd + dst + (unsigned)(isx ? i * FRS : (i >> 1) * HTB + (i & 1) * FRS);
    }
  };

  auto decode = [&](unsigned u, bool& isx, unsigned& src, unsigned& dst) {
    int sg, i; isx = false;
    if (u < (unsigned)U1) { sg = u / 240u; i = u % 240u; isx = i < 128; if (!isx) i -= 128; }
    else { const unsigned v = u - U1; sg = 8 + v / 112u; i = v % 112u; }
    place(isx, (sg >> 3) * RCOLS + (i >> 3), i & 3, (sg & 7) * 2 + ((i >> 2) & 1), src, dst);
  };
  const int aid = wid == 0 ? b : NAGENT + b;
  f32x4 v[8]; bool isx = false; unsigned src = 0, dst = 0;
  if (li < 4 && wid == 0) { decode((unsigned)b, isx, src, dst); load_unit(v, true, isx, src, IC<0>{}); }
  if (tid == 0) {
    const unsigned c = __hip_atomic_fetch_add((unsigned*)(myws + WS_CTR + b * 64), 1u, __ATOMIC_RELAXED, __HIP_MEMORY_SCOPE_AGENT) + 1u;
    *(LAS unsigned*)(lds + LDS_BAD + 4) = c; *(LAS unsigned*)(lds + LDS_BAD) = 0u;
  }
  *(LAS unsigned*)(lds + LDS_FLAG + tid * 4) = 0u;
  WAIT_L(0); BAR; asm volatile("" ::: "memory");
  const unsigned call = __builtin_amdgcn_readfirstlane(*(LAS unsigned*)(lds + LDS_BAD + 4));
  const unsigned seq0 = call * SEQ;
  const size_t stg = WS_STG + (size_t)(call & 1u) * STG_BYTES;
  const char* flagw = myws + WS_FLAG + (lane & 7) * 128;

  unsigned* prog = (unsigned*)(myws + WS_PROG);
  unsigned* ticket = (unsigned*)(myws + WS_TICKET + (call & 3u) * 128u);
  auto draw = [&](unsigned* ctr, bool real) -> unsigned {
    return (unsigned)__builtin_amdgcn_raw_ptr_buffer_atomic_add_i32(1, __builtin_amdgcn_make_buffer_rsrc(ctr, 0, real ? 4 : 0, 0x00020000), (int)vd, 0, 0); };
  auto progress = [&](unsigned val) { __builtin_amdgcn_raw_buffer_store_b32(val, __builtin_amdgcn_make_buffer_rsrc(prog + aid, 0, 4, 0x00020000), vd, 0, 16); };
  auto read_go = [&]() -> unsigned { return __builtin_amdgcn_readfirstlane(__hip_atomic_load((unsigned*)(myws + WS_GO + 4), __ATOMIC_RELAXED, __HIP_MEMORY_SCOPE_AGENT)); };
  auto send_ops = [&](unsigned go, unsigned cur, unsigned tnext, unsigned limit) {
    unsigned* ctr = ticket + (go >> 31) * 8; const unsigned base = (go >> 31) ? 0u : 2u * NAGENT;
    const float mx = pow2f((int)((go >> 12) & 0xfffu) - 256), mw = pow2f((int)(go & 0xfffu) - 256);
    __amdgpu_buffer_rsrc_t rd[NR - 1];
#pragma unroll
    for (int d = 0; d < NR - 1; ++d) rd[d] = __builtin_amdgcn_make_buffer_rsrc(p.ws[(rank + 1 + (d + b + wid) % (NR - 1)) & 7] + stg, 0, STG_BYTES, 0x00020000);
    bool real = true;
#pragma unroll
    for (int j = 0; j < 8; ++j) asm volatile("" : "+v"(v[j]));
    asm volatile("" : "+v"(tnext));
    while (cur < (unsigned)NUNITS) {
      u32x4 o[4]; unsigned oo[4];
      pack_unit(v, isx ? mx : mw, isx, dst, o, oo);
      const unsigned nxt = real ? __builtin_amdgcn_readfirstlane(tnext) + base : 0xffffffffu;
      SCHED;
      decode(nxt < (unsigned)NUNITS ? nxt : 0u, isx, src, dst); load_unit(v, nxt < (unsigned)NUNITS, isx, src, IC<0>{});
      real = nxt < limit; tnext = draw(ctr, real);
      SCHED;
      SEND_UNIT(9, true);
      cur = nxt;
    }
    WAIT_V(0);
  };
  auto join_ops = [&](unsigned go, unsigned limit) {
    unsigned* ctr = ticket + (go >> 31) * 8;
    const unsigned cur = __builtin_amdgcn_readfirstlane(draw(ctr, true)) + ((go >> 31) ? 0u : 2u * NAGENT);
    decode(cur < (unsigned)NUNITS ? cur : 0u, isx, src, dst); load_unit(v, cur < (unsigned)NUNITS, isx, src, IC<0>{});
    send_ops(go, cur, draw(ctr, true), limit);
  };
  auto send_out = [&]() {
    unsigned cur = draw(ticket + 16, true), tnext = draw(ticket + 16, true);
    const bool ok = wave_wait<1>(myws + WS_PCNT, false, call * 16u - 1u, tmo, call);
    cur = __builtin_amdgcn_readfirstlane(cur);
    __amdgpu_buffer_rsrc_t rd[NR - 1];
#pragma unroll
    for (int d = 0; d < NR - 1; ++d) rd[d] = __builtin_amdgcn_make_buffer_rsrc(p.out[(rank + 1 + (d + b + wid) % (NR - 1)) & 7], 0, GM * GN * 4, 0x00020000);
    const unsigned vb = vd + (unsigned)((NRC + rank) * 256 * 4);
    auto orow = [](unsigned u, unsigned i) -> unsigned {
      return OPERM == 0 ? (u * 4u + i) % (unsigned)GM : OPERM == 1 ? (((u * 37u) & 1023u) * 4u + i) : ((u * 37u) & 1023u) + i * 1024u; };
    auto load_out = [&](unsigned u) {
      const __amdgpu_buffer_rsrc_t rs = __builtin_amdgcn_make_buffer_rsrc(p.out[rank], 0, u < (unsigned)NOUT ? GM * GN * 4 : 0, 0x00020000);
#pragma unroll
      for (int i = 0; i < 4; ++i) v[i] = __builtin_bit_cast(f32x4, __builtin_amdgcn_raw_buffer_load_b128(rs, vb, orow(u, i) * (unsigned)(GN * 4), 16));
    };
    load_out(cur);
    if (li >= 2) wave_wait<0>(myws + WS_SENT, false, seq0 + (unsigned)(NUNITS - PSTART), tmo, call);
#pragma unroll
    for (int j = 0; j < 4; ++j) asm volatile("" : "+v"(v[j]));
    asm volatile("" : "+v"(tnext));
    bool real = true;
    while (cur < (unsigned)NOUT) {
      u32x4 o[4]; unsigned oo[4];
#pragma unroll
      for (int i = 0; i < 4; ++i) { o[i] = __builtin_bit_cast(u32x4, v[i]); if (!ok) o[i] = (u32x4){0x7fc00000u, 0x7fc00000u, 0x7fc00000u, 0x7fc00000u}; oo[i] = vb + orow(cur, i) * (unsigned)(GN * 4); asm volatile("" : "+v"(o[i])); }
      const unsigned nxt = real ? __builtin_amdgcn_readfirstlane(tnext) : 0xffffffffu;
      SCHED;
      load_out(nxt);
      real = nxt < (unsigned)NOUT; tnext = draw(ticket + 16, real);
      SCHED;
      SEND_UNIT(5, false);
      cur = nxt;
    }
    WAIT_V(0);
    progress(seq0 + (unsigned)(SEQ - 1));
    STAMPW(11);
  };

  bool ready = false;
  if (li < 4) {
    if (wid == 0) {
      const unsigned limit = li < 2 ? (unsigned)NUNITS : (unsigned)PLEAVE;
      send_ops((unsigned)(SHIFT0 + 256) << 12 | (unsigned)(SHIFT0 + 256), (unsigned)b, (unsigned)(b - NAGENT), limit);
      wave_wait<0>(myws + WS_GO, false, call - 1u, tmo, call);
      const unsigned go = read_go();
      if (go >> 31) { progress(seq0); WAIT_V(0); }
      if (lane == 0) __hip_atomic_fetch_add((unsigned*)(myws + WS_DCNT), 1u, __ATOMIC_RELAXED, __HIP_MEMORY_SCOPE_AGENT);
      if (go >> 31) { wave_wait<0>(myws + WS_DCNT, false, call * (unsigned)NAGENT - 1u, tmo, call); join_ops(go, limit); }
      progress(seq0 + (unsigned)(NUNITS + 8));
      STAMP(12);
      if (li < 2) send_out();
    } else if (wid == 2 && li < 2) {
      progress(seq0 + (unsigned)(SEQ - 1));
      wave_wait<1>(myws + WS_SENT, false, seq0 + (unsigned)JOIN2, tmo, call);
      progress(seq0 + (unsigned)JOIN2);
      WAIT_V(0);
      join_ops(read_go(), (unsigned)NUNITS);
      progress(seq0 + (unsigned)(SEQ - 1));
    } else if (wid == 6) {
      wave_wait<1>(myws + WS_SENT, false, seq0 + (unsigned)(PINAT), tmo, call);
      const float mw = pow2f((int)(read_go() & 0xfffu) - 256);
#pragma unroll 1
      for (int d = 1; d < NR; ++d) {
        const int pr = (rank + 1 + (d + b) % (NR - 1)) & 7;
        const __amdgpu_buffer_rsrc_t rp = __builtin_amdgcn_make_buffer_rsrc(p.ws[pr] + stg, 0, STG_BYTES, 0x00020000);
        f32x4 pv[2][8]; unsigned psrc, pdst[2];
#pragma unroll
        for (int e = 0; e < 2; ++e) { place(false, NRC + pr, (b >> 4) * 2 + e, b & 15, psrc, pdst[e]); load_unit(pv[e], true, false, psrc, IC<PRENT>{}); }
#pragma unroll
        for (int e = 0; e < 2; ++e) { u32x4 o[4]; unsigned oo[4]; pack_unit(pv[e], mw, false, pdst[e], o, oo);
#pragma unroll
          for (int i = 0; i < 4; ++i) __builtin_amdgcn_raw_buffer_store_b128(o[i], rp, oo[i], 0, AUX_WT); }
      }
      WAIT_V(0);
      if (lane == 0) __hip_atomic_fetch_add((unsigned*)(myws + WS_PINC), 1u, __ATOMIC_RELAXED, __HIP_MEMORY_SCOPE_AGENT);
    } else if (b == 0 && wid == 7) {
      if (lane < 3) __hip_atomic_store((unsigned*)(myws + WS_TICKET + ((call + 2u) & 3u) * 128u) + lane * 8, 0u, __ATOMIC_RELAXED, __HIP_MEMORY_SCOPE_AGENT);
      unsigned* myflag = (lane & 7) == rank ? (unsigned*)(myws + WS_SENT) : (unsigned*)(p.ws[lane & 7] + WS_FLAG + rank * 128);
      const unsigned long long t0 = __builtin_amdgcn_s_memrealtime();
      bool ok = wave_wait<0>(myws + WS_GO, false, call - 1u, tmo, call);
      if (read_go() >> 31) {
        ok = wave_wait<0>(myws + WS_LCNT2, false, call * (unsigned)NRW - 1u, tmo, call) && ok;
        ok = wave_wait<0>(myws + WS_DCNT, false, call * (unsigned)NAGENT - 1u, tmo, call) && ok;
      }
      if (lane == 0) __hip_atomic_store((unsigned*)(myws + WS_FLAG + rank * 128), seq0 + (unsigned)(SEQ - 1), __ATOMIC_RELAXED, __HIP_MEMORY_SCOPE_SYSTEM);
      unsigned last = seq0; bool pin = false;
      while (last <= seq0 + (unsigned)(NUNITS + 8) || !pin) {
        unsigned m = 0xffffffffu;
#pragma unroll
        for (int j = 0; j < (NPROG + 63) / 64; ++j) { const int a = j * 64 + lane; if (a < NPROG) { const unsigned x = __hip_atomic_load(prog + a, __ATOMIC_RELAXED, __HIP_MEMORY_SCOPE_AGENT); m = x < m ? x : m; } }
        if (!pin && (unsigned)__builtin_amdgcn_readfirstlane(__hip_atomic_load((unsigned*)(myws + WS_PINC), __ATOMIC_RELAXED, __HIP_MEMORY_SCOPE_AGENT)) >= call * 32u) {
          pin = true;
          if (lane < NR) __hip_atomic_store((unsigned*)(p.ws[lane] + WS_PIN + rank * 128), call, __ATOMIC_RELAXED, __HIP_MEMORY_SCOPE_SYSTEM);
        }
#pragma unroll
        for (int sft = 32; sft >= 1; sft >>= 1) { const unsigned x = __shfl_xor(m, sft); m = x < m ? x : m; }
        m = __builtin_amdgcn_readfirstlane(m);
        if (m > last) { last = m; if (lane < NR) __hip_atomic_store(myflag, m, __ATOMIC_RELAXED, __HIP_MEMORY_SCOPE_SYSTEM); }
        if (__builtin_amdgcn_s_memrealtime() - t0 > DEADLINE) { __hip_atomic_store(tmo, call, __ATOMIC_RELAXED, __HIP_MEMORY_SCOPE_AGENT); ok = false; break; }
      }
      ok = wave_wait<0>(flagw, false, seq0 + (unsigned)(NUNITS + 8), tmo, call) && ok;
      ok = ok && __hip_atomic_load(tmo, __ATOMIC_RELAXED, __HIP_MEMORY_SCOPE_AGENT) != call;
      if (!ok && lane == 0) __hip_atomic_store((unsigned*)p.out[rank] + (NRC + rank) * 256, 0x7fc00000u, __ATOMIC_RELAXED, __HIP_MEMORY_SCOPE_SYSTEM);
    }
    if (li < 2) return;
  } else {
    const __amdgpu_buffer_rsrc_t rl = __builtin_amdgcn_make_buffer_rsrc(myws + stg, 0, STG_BYTES, 0x00020000);
    { const unsigned long long t0 = __builtin_amdgcn_s_memrealtime(); while (__builtin_amdgcn_s_memrealtime() - t0 < (unsigned long long)(PREDELAY * (li - 3))) __builtin_amdgcn_s_sleep(4); }
    float mx = pow2f(SHIFT0), mw = mx;
    ready = true;
    for (int pass = 0; pass < 2; ++pass) {
      unsigned ax = 0, aw = 0;
      for (int j = b - 32; j < 640; j += NRW) {
        isx = j < 128;
        if (j < 128) place(true, 2 * (j & 7) + (wid >> 2), wid & 3, j >> 3, src, dst);
        else if (j < 576) place(false, 2 * ((j - 128) % 28) + (wid >> 2), wid & 3, (j - 128) / 28, src, dst);
        else place(false, NRC + ((j - 576) >> 3), wid & 3, 2 * (j & 7) + (wid >> 2), src, dst);
        load_unit(v, true, isx, src, IC<PRENT>{});
        unsigned am = 0;
#pragma unroll
        for (int q = 0; q < 32; ++q) { unsigned a = __builtin_bit_cast(unsigned, v[q >> 2][q & 3]) & 0x7fffffffu; a = a < 0x7f800000u ? a : 0u; am = a > am ? a : am; }
        if (isx) ax = am > ax ? am : ax; else aw = am > aw ? am : aw;
        if (j < 576 || ((j - 576) >> 3) == rank) {
          u32x4 o[4]; unsigned oo[4]; pack_unit(v, isx ? mx : mw, isx, dst, o, oo);
#pragma unroll
          for (int i = 0; i < 4; ++i) __builtin_amdgcn_raw_buffer_store_b128(o[i], rl, oo[i], 0, AUX_WT);
        }
      }
      if (pass == 0) {
#pragma unroll
        for (int sft = 32; sft >= 1; sft >>= 1) { const unsigned a = __shfl_xor(ax, sft), c = __shfl_xor(aw, sft); ax = a > ax ? a : ax; aw = c > aw ? c : aw; }
        if (lane == 0) { *(LAS unsigned*)(lds + LDS_BAD + 32 + wid * 8) = ax; *(LAS unsigned*)(lds + LDS_BAD + 36 + wid * 8) = aw; }
      }
      WAIT_V(0); WAIT_L(0); BAR; asm volatile("" ::: "memory");
      if (pass == 1) break;
      if (wid == 0) {
        if (lane < 2) {
          unsigned a = 0;
#pragma unroll
          for (int q = 0; q < 8; ++q) { const unsigned c = *(LAS unsigned*)(lds + LDS_BAD + 32 + q * 8 + lane * 4); a = c > a ? c : a; }
          __hip_atomic_fetch_max((unsigned long long*)(myws + WS_LMAX) + lane, (unsigned long long)call << 32 | a, __ATOMIC_RELAXED, __HIP_MEMORY_SCOPE_AGENT);
        }
        WAIT_V(0);
        if ((unsigned)__builtin_amdgcn_readfirstlane(draw((unsigned*)(myws + WS_LCNT), true)) == call * (unsigned)NRW - 1u) {
          const unsigned long long mm = __hip_atomic_load((unsigned long long*)(myws + WS_LMAX) + (lane & 1), __ATOMIC_RELAXED, __HIP_MEMORY_SCOPE_AGENT);
          const unsigned e1 = (unsigned)mm ? 1u + ((unsigned)mm >> 23) : 0u;
          const unsigned e12 = (unsigned)__builtin_amdgcn_readlane(e1, 0) << 16 | (unsigned)__builtin_amdgcn_readlane(e1, 1);
          if (lane < NR) __hip_atomic_store((unsigned long long*)(p.ws[lane] + WS_AMX + rank * 128), (unsigned long long)e12 << 32 | call, __ATOMIC_RELAXED, __HIP_MEMORY_SCOPE_SYSTEM);
          wave_wait<0>(myws + WS_AMX + (lane & 7) * 128, false, call - 1u, tmo, call);
          const unsigned e2 = __hip_atomic_load((unsigned*)(myws + WS_AMX + (lane & 7) * 128 + 4), __ATOMIC_RELAXED, __HIP_MEMORY_SCOPE_SYSTEM);
          unsigned gx = e2 >> 16, gw = e2 & 0xffffu;
#pragma unroll
          for (int sft = 4; sft >= 1; sft >>= 1) { const unsigned a = __shfl_xor(gx, sft), c = __shfl_xor(gw, sft); gx = a > gx ? a : gx; gw = c > gw ? c : gw; }
          auto shift_of = [](unsigned g) -> int { const int e = (int)g - 128; return (g == 0u || (e >= -SHIFT0 && e < 15 - SHIFT0)) ? SHIFT0 : (14 - e > 126 ? 126 : 14 - e); };
          const int sx = shift_of(__builtin_amdgcn_readfirstlane(gx)), sw = shift_of(__builtin_amdgcn_readfirstlane(gw));
          const bool redo = sx != SHIFT0 || sw != SHIFT0;
          if (lane == 0) {
            __hip_atomic_store((unsigned long long*)(myws + WS_GO), (unsigned long long)((redo ? 0x80000000u : 0u) | (unsigned)(sx + 256) << 12 | (unsigned)(sw + 256)) << 32 | call, __ATOMIC_RELAXED, __HIP_MEMORY_SCOPE_AGENT);
            if (redo) __hip_atomic_fetch_add(ticket, 1u << 20, __ATOMIC_RELAXED, __HIP_MEMORY_SCOPE_AGENT);
          }
        }
        if (!wave_wait(myws + WS_GO, false, call - 1u, tmo, call)) *(LAS unsigned*)(lds + LDS_BAD) = 1u;
        *(LAS unsigned*)(lds + LDS_BAD + 8) = read_go();
        WAIT_L(0);
      }
      asm volatile("" ::: "memory"); BAR; asm volatile("" ::: "memory");
      const unsigned go = __builtin_amdgcn_readfirstlane(*(LAS unsigned*)(lds + LDS_BAD + 8));
      if (!(go >> 31)) break;
      mx = pow2f((int)((go >> 12) & 0xfffu) - 256); mw = pow2f((int)(go & 0xfffu) - 256); ready = false;
    }
    if (tid == 0) __hip_atomic_fetch_add((unsigned*)(myws + WS_LCNT2), 1u, __ATOMIC_RELAXED, __HIP_MEMORY_SCOPE_AGENT);
    STAMP(13);
  }

  const bool isP = li < 4;
  const int wr = wid >> 2, wc = wid & 3;
  const int lr = li - 4;
  const int mt = isP ? (xcd & 3) * 4 + (xcd >> 2) * 2 + (li - 2) : (xcd & 3) * 4 + (lr & 3);
  const int cr = isP ? 0 : (xcd >> 2) + 2 * (lr >> 2);
  const unsigned voff = (unsigned)(wid * 2048 + lane * 16), ldsw = (unsigned)wid * 2048u;
  const char* fsrc = myws + WS_FLAG + wid * 128;
  const __amdgpu_buffer_rsrc_t ra = __builtin_amdgcn_make_buffer_rsrc(myws + stg, 0, A_BYTES, 0x00020000);
  const __amdgpu_buffer_rsrc_t rb = __builtin_amdgcn_make_buffer_rsrc(myws + stg + A_BYTES + (isP ? B_BYTES : 0u), 0, isP ? P_BYTES : B_BYTES, 0x00020000);
  const unsigned bstep = isP ? (unsigned)PIECE : B_STEP;
#define STAGE(ldsoff, rs, so) do { \
    __builtin_amdgcn_raw_ptr_buffer_load_lds(rs, (LAS void*)(lds + (ldsoff) + ldsw), 16, voff, (so), 0, 0); \
    __builtin_amdgcn_raw_ptr_buffer_load_lds(rs, (LAS void*)(lds + (ldsoff) + ldsw), 16, voff, (so), 1024, 0); } while (0)
#define ST1(ldsoff, rs, so, imm) do { __builtin_amdgcn_raw_ptr_buffer_load_lds(rs, (LAS void*)(lds + (ldsoff) + ldsw), 16, voff, (so), imm, 0); SCHED; } while (0)
#define RA(h, m) do { _Pragma("unroll") for (int k = 0; k < 2; ++k) At[m][k] = *(const LAS f16x8*)((LAS char*)(size_t)va + (h) * HTB + (m) * FRS + k * KHS); SCHED; } while (0)
#define FLAGDMA() __builtin_amdgcn_global_load_lds((const unsigned*)fsrc, (LAS unsigned*)(lds + LDS_FLAG + wid * 256), 4, 0, FAUX)
#define LDA(dst, h) do { _Pragma("unroll") for (int m = 0; m < 4; ++m) _Pragma("unroll") for (int k = 0; k < 2; ++k) dst[m][k] = *(const LAS f16x8*)((LAS char*)(size_t)va + (h) * HTB + m * FRS + k * KHS); } while (0)
#define LDB(dst, h) do { _Pragma("unroll") for (int n = 0; n < 2; ++n) _Pragma("unroll") for (int k = 0; k < 2; ++k) dst[n][k] = *(const LAS f16x8*)((LAS char*)(size_t)vb + (h) * HTB + n * FRS + k * KHS); } while (0)
#define ACC(ai, nn, m) (128 + (((ai) * 4 + (nn)) * 4 + (m)) * 4)
#define MMA2(ai) do { MMA2_(ai); BAR; SCHED; } while (0)
#define MMA2_(ai) do { __builtin_amdgcn_s_waitcnt(0xC07F); SCHED; BAR; SCHED; __builtin_amdgcn_s_setprio(1); SCHED; \
    _Pragma("unroll") for (int m = 0; m < 4; ++m) _Pragma("unroll") for (int q = 0; q < 4; ++q) { const int nn = (m & 1) ? 3 - q : q; \
      asm volatile("v_mfma_f32_16x16x32_f16 v[%c0:%c1], %2, %3, v[%c0:%c1]\n\tv_mfma_f32_16x16x32_f16 v[%c0:%c1], %4, %5, v[%c0:%c1]" \
          :: "i"(ACC(ai, nn, m)), "i"(ACC(ai, nn, m) + 3), "v"(At[m][0]), "v"((nn >> 1) ? B1[nn & 1][0] : B0[nn & 1][0]), "v"(At[m][1]), "v"((nn >> 1) ? B1[nn & 1][1] : B0[nn & 1][1])); } \
    SCHED; __builtin_amdgcn_s_setprio(0); SCHED; } while (0)
#define ZERO_ACC() do { _Pragma("unroll") for (int i_ = 128; i_ < 256; i_ += 2) asm volatile("v_mov_b64 v[%c0:%c1], 0" :: "i"(i_), "i"(i_ + 1)); } while (0)
#define ACC_ROW(dst, ai, m, j) do { float t0_, t1_, t2_, t3_; \
    asm volatile("v_ldexp_f32 %0, v[%c4], %8\n\tv_ldexp_f32 %1, v[%c5], %8\n\tv_ldexp_f32 %2, v[%c6], %8\n\tv_ldexp_f32 %3, v[%c7], %8" \
        : "=&v"(t0_), "=&v"(t1_), "=&v"(t2_), "=&v"(t3_) : "i"(ACC(ai, 0, m) + (j)), "i"(ACC(ai, 1, m) + (j)), "i"(ACC(ai, 2, m) + (j)), "i"(ACC(ai, 3, m) + (j)), "s"(oexp)); \
    dst = (f32x4){t0_, t1_, t2_, t3_}; } while (0)
  asm volatile("" ::: "v128","v129","v130","v131","v132","v133","v134","v135","v136","v137","v138","v139","v140","v141","v142","v143","v144","v145","v146","v147","v148","v149","v150","v151","v152","v153","v154","v155","v156","v157","v158","v159","v160","v161","v162","v163","v164","v165","v166","v167","v168","v169","v170","v171","v172","v173","v174","v175","v176","v177","v178","v179","v180","v181","v182","v183","v184","v185","v186","v187","v188","v189","v190","v191","v192","v193","v194","v195","v196","v197","v198","v199","v200","v201","v202","v203","v204","v205","v206","v207","v208","v209","v210","v211","v212","v213","v214","v215","v216","v217","v218","v219","v220","v221","v222","v223","v224","v225","v226","v227","v228","v229","v230","v231","v232","v233","v234","v235","v236","v237","v238","v239","v240","v241","v242","v243","v244","v245","v246","v247","v248","v249","v250","v251","v252","v253","v254","v255");
  ZERO_ACC();
  f16x8 At[4][2], B0[2][2], B1[2][2];
  const int ph = isP ? 0 : phase_of(cr);
  const int Tend = isP ? 64 : 256;
  unsigned xcc_; asm volatile("s_getreg_b32 %0, hwreg(HW_REG_XCC_ID)" : "=s"(xcc_));
  const int NL = NLMODE == 0 ? 0 : (int)(xcc_ & 1u) * NLMODE;
  const int E0 = 8 + (PHS ? 7 * (cr >> 2) + (((cr >> 1) & 1) ? (rank < 4 ? 3 : 4) : 0) : 0);
  unsigned tA, tB, tN;
  auto steps = [&](int t0) {
    const int t = t0 + lane, u = t - E0, w = u - 8 * NL - 8, z = w - 56 * (1 + NL);
    const bool inl = z >= 0 && z < 64 * (2 - NL);
    const bool local = t < 8 || (u >= 0 && w < 0) || (inl && (z & 7) == 0);
    const int rho = u < 0 ? t - 8 : z < 0 ? E0 - 8 + w : inl ? E0 - 8 + w - (z >> 3) - 1 : t - 32, pos = rho % 7;
    const int sgm = !local ? rho / 7 : t < 8 ? (t < ph ? t : t + 24) : inl ? ph + 8 * (1 + NL) + (z >> 3) : u < 8 * NL ? ph + 8 + u : ph + (u & 7);
    const unsigned sr = (unsigned)(local ? rank : pos + (pos >= rank ? 1 : 0)), ix = isP ? (unsigned)t : (unsigned)sgm * 8u + sr;
    tA = t < Tend ? (unsigned)mt * PIECE + (ix & 63u) * A_STEP : 0x80000000u;
    tB = t < Tend ? (unsigned)cr * PIECE + ix * bstep : 0x80000000u;
    tN = (isP || t >= Tend || local) ? 0u : seq0 + (unsigned)((sgm < 8 ? 240 * sgm + 128 : U1 - 8 * 112 + 112 * sgm) + 8 * cr + 8) + sr;
  };
  steps(0);
  if (!ready) {
    if (wid == 0) {
      if (isP && !wave_wait<1>(myws + WS_PIN + (lane & 7) * 128, false, call - 1u, tmo, call)) *(LAS unsigned*)(lds + LDS_BAD) = 1u;
      if (!wave_wait(isP ? flagw : myws + WS_FLAG + rank * 128, false, seq0 + (unsigned)(U1 - 1), tmo, call)) *(LAS unsigned*)(lds + LDS_BAD) = 1u;
      *(LAS unsigned*)(lds + LDS_BAD + 8) = read_go(); WAIT_L(0);
    }
    asm volatile("" ::: "memory"); BAR; asm volatile("" ::: "memory");
  }
  int oexp;
  { const unsigned go = __builtin_amdgcn_readfirstlane(*(LAS unsigned*)(lds + LDS_BAD + 8)); oexp = 512 - (int)((go >> 12) & 0xfffu) - (int)(go & 0xfffu); }
  STAMP(1);
  __builtin_amdgcn_s_waitcnt(0x0F70); SCHED;
  unsigned par = 0, l1 = PIECE + HTB, a1, fl = 0;
  unsigned a2 = __builtin_amdgcn_readlane(tA, 2), b2 = __builtin_amdgcn_readlane(tB, 2), need = 0;
  unsigned va = (unsigned)(size_t)lds + wr * 4 * FRS + lane * 16, vb = (unsigned)(size_t)lds + 2 * PIECE + wc * 2 * FRS + lane * 16;
  LAS unsigned* fimg = (LAS unsigned*)(lds + LDS_FLAG + (lane & 7) * 256); asm volatile("" : "+v"(fimg));
  { const unsigned a0 = __builtin_amdgcn_readlane(tA, 0), b0 = __builtin_amdgcn_readlane(tB, 0), b1 = __builtin_amdgcn_readlane(tB, 1); a1 = __builtin_amdgcn_readlane(tA, 1);
    STAGE(2 * PIECE, rb, b0); STAGE(0, ra, a0); STAGE(2 * PIECE + HTB, rb, b0 + HTB); STAGE(HTB, ra, a0 + HTB);
    if (wr == 1) BAR;
    WAIT_V(2); BAR;
    FLAGDMA();
    STAGE(3 * PIECE, rb, b1); STAGE(PIECE, ra, a1); STAGE(3 * PIECE + HTB, rb, b1 + HTB);
    a1 += HTB;
    BAR; }
  const unsigned vpk = (unsigned)tid * 16u;
  const unsigned vring = (unsigned)(size_t)lds + LDS_RING + wid * 1024 + lane * 16;
#define SLOW_T0
#define SLOW_T1
#define STEP_STAMP
#define KLOOP(ADD, LIM) for (;;) { \
      f32x4 pt; \
      ST1(l1, ra, a1, 0); LDB(B0, 0); SCHED; ST1(l1, ra, a1, 1024); LDB(B1, 1); SCHED; \
      if (ADD) { __builtin_amdgcn_raw_ptr_buffer_load_lds(rpk, (LAS void*)(lds + LDS_RING + rs_ + wid * 1024), 16, vpk, (unsigned)(li3 - a0_) * 8192u, 0, 0); SCHED; \
        rs_ = rs_ == 16384u ? 0u : rs_ + 8192u; pt = *(const LAS f32x4*)(size_t)(vring + rs_); } \
      else FLAGDMA(); \
      SCHED; LDA(At, 0); WAIT_V(10); \
      MMA2(0); \
      if ((unsigned)__builtin_amdgcn_readlane(fl, need & 63u) < need) { SLOW_T0; \
        const unsigned f2 = *(const volatile LAS unsigned*)fimg; fl = f2 > fl ? f2 : fl;     \
        if ((unsigned)__builtin_amdgcn_readlane(fl, need & 63u) < need) { \
          if (!wave_wait<1>(flagw, (unsigned)(lane & 7) != (need & 7u), need - 1u, tmo, call, &fl)) *(LAS unsigned*)(lds + LDS_BAD) = 1u; \
          *(volatile LAS unsigned*)fimg = fl; } \
        SLOW_T1; } \
      ST1(2 * PIECE + par, rb, b2, 0); RA(1, 0); ST1(2 * PIECE + par, rb, b2, 1024); RA(1, 1); \
      ST1(par, ra, a2, 0); RA(1, 2); ST1(par, ra, a2, 1024); RA(1, 3); \
      const unsigned fn = ADD ? 0u : *fimg; \
      if (ADD && li3 - a0_ >= 2) { unsigned keep_; \
        asm volatile("s_mov_b32 %0, m0\n\ts_set_gpr_idx_on %1, gpr_idx(SRC0,DST)\n\tv_add_f32 v128, v128, %2\n\tv_add_f32 v129, v129, %3\n\tv_add_f32 v130, v130, %4\n\tv_add_f32 v131, v131, %5\n\ts_set_gpr_idx_off\n\ts_mov_b32 m0, %0" \
            : "=&s"(keep_) : "s"(((li3 - a0_ - 2) & 31) * 4), "v"(pt[0]), "v"(pt[1]), "v"(pt[2]), "v"(pt[3])); } \
      SCHED; \
      ST1(2 * PIECE + par + HTB, rb, b2 + HTB, 0); \
      l1 = par + HTB; a1 = a2 + HTB; va ^= PIECE; vb ^= PIECE; \
      a2 = __builtin_amdgcn_readlane(tA, li3); need = __builtin_amdgcn_readlane(tN, li3); \
      asm volatile("" : "+v"(va), "+v"(vb), "+s"(a2), "+s"(need), "+s"(l1), "+s"(a1)); SCHED; \
      ST1(2 * PIECE + par + HTB, rb, b2 + HTB, 1024); \
      par ^= PIECE; b2 = __builtin_amdgcn_readlane(tB, li3); ++li3; \
      STEP_STAMP \
      fl = fn > fl ? fn : fl; \
      asm volatile("" : "+s"(par), "+s"(li3), "+s"(b2), "+v"(fl)); \
      WAIT_V(9); \
      MMA2_(1); \
      if (li3 >= (LIM)) break; \
      BAR; SCHED; }
  int t = 0;
  for (int sg = isP ? 4 : -NL; sg < 5; ++sg) {
    const int Ts = sg == 4 ? Tend : sg == -NL ? E0 : t + (sg <= 0 ? 8 : (sg >= 2 && sg < 2 + NL) ? 56 : 64);
    const int slot = sg <= 0 ? sg + NL : (sg == 4 ? 0 : sg - 1);
    char* pk = myws + WS_PARK + (size_t)(b * 3 + slot) * 262144;
    const __amdgpu_buffer_rsrc_t rpk = __builtin_amdgcn_make_buffer_rsrc(pk, 0, 262144, 0x00020000);
    steps(t + 3);
    const int left = Ts - t; int li3 = 0;
    unsigned rs_ = 0;
    const bool add = !isP && (sg == 4 || (sg >= 2 && sg < 2 + NL));
    const int a0_ = !add ? 0 : (ADDLATE && sg < 4) ? left - 35 : 0;
    for (int part = a0_ ? 0 : 1; part < 2; ++part) {
      if (part == 1 && add) { KLOOP(true, a0_ + 34); BAR; SCHED; }
      const int lim_ = part ? left : a0_;
      KLOOP(false, lim_);
      if (!part) { BAR; SCHED; }
    }
    t = Ts;
    if (wr == 0) BAR;
    asm volatile("s_nop 15\n\ts_nop 15" ::: "memory");
    SCHED;
    STAMP(2 + 2 * r);
    if (sg <= 0) {
#pragma unroll
      for (int i = 0; i < 32; ++i)
        asm volatile("global_store_dwordx4 %0, v[%c1:%c2], %3" :: "v"(vpk), "i"(128 + 4 * i), "i"(128 + 4 * i + 3), "s"(pk + i * 8192) : "memory");
      asm volatile("s_nop 1");
    } else {
      if (__builtin_amdgcn_readfirstlane(*(const LAS unsigned*)(lds + LDS_BAD)) != 0u) {
#pragma unroll
        for (int i = 128; i < 256; ++i) asm volatile("v_mov_b32 v[%c0], 0x7fc00000" :: "i"(i));
      }
      const int nt = isP ? NRC + rank : (sg & 3) * RCOLS + cr;
      const __amdgpu_buffer_rsrc_t rc = __builtin_amdgcn_make_buffer_rsrc(p.out[rank] + (size_t)(mt * 256 + wr * 64) * GN + nt * 256 + wc * 64, 0, 192u * GN * 4u, 0x00020000);
      unsigned vo = (unsigned)((lane >> 4) * 4 * GN + (lane & 15) * 4) * 4u; asm volatile("" : "+v"(vo));
#pragma unroll
      for (int ai = 0; ai < 2; ++ai)
#pragma unroll
        for (int m = 0; m < 4; ++m)
#pragma unroll
          for (int j = 0; j < 4; ++j) {
            f32x4 v4; ACC_ROW(v4, ai, m, j);
            __builtin_amdgcn_raw_buffer_store_b128(__builtin_bit_cast(u32x4, v4), rc, vo + (unsigned)((ai * 128 + m * 16 + j) * GN * 4), 0, AUX_NT);
          }
    }
    ZERO_ACC();
    if (wr == 1) BAR;
    STAMP(3 + 2 * r);
    STAMPC(20 + r);
  }
  WAIT_V(0);
  STAMP(15);
  if (wr == 0) BAR;
  if (isP) {
    BAR; asm volatile("" ::: "memory");
    if (tid == 0) {
      __builtin_amdgcn_fence(__ATOMIC_RELEASE, "agent");
      WAIT_V(0);
      __hip_atomic_fetch_add((unsigned*)(myws + WS_PCNT), 1u, __ATOMIC_RELAXED, __HIP_MEMORY_SCOPE_AGENT);
    }
    STAMP(10);
    if (wid == 0) send_out();
  }
}
}

extern "C" void kernel_launch(int rank, int nranks, ncclComm_t comm, hipStream_t stream,
                              void* const* d_in, const int* in_sizes, int n_in,
                              void* d_out, int out_size, const int* in_dtypes, int out_dtype,
                              void* const* d_out_all, void* const* d_ws_all, size_t ws_bytes) {
  (void)comm; (void)in_dtypes; (void)out_dtype; (void)d_out;
  if (nranks != NR || (unsigned)rank >= (unsigned)NR || n_in < 2 || ws_bytes < WS_NEED || out_size != GM * GN ||
      in_sizes[0] != GM * KSH || in_sizes[1] != KSH * GN) { fprintf(stderr, "unexpected problem shape\n"); return; }
  static bool attr_set = false;
  if (!attr_set) { HIP_CHECK(hipFuncSetAttribute((const void*)fused_kernel, hipFuncAttributeMaxDynamicSharedMemorySize, LDS_BYTES)); attr_set = true; }
  Params p;
  p.x = (const float*)d_in[0]; p.w = (const float*)d_in[1]; p.rank = rank;
  for (int r = 0; r < NR; ++r) { p.out[r] = (float*)d_out_all[r]; p.ws[r] = (char*)d_ws_all[r]; }
  fused_kernel<<<256, 512, LDS_BYTES, stream>>>(p);
  HIP_CHECK(hipGetLastError());
}
```

```cpp
#include <hip/hip_runtime.h>
#include <rccl/rccl.h>
#include <cstdio>

#define HIP_CHECK(expr) do { hipError_t _e = (expr); if (_e != hipSuccess) { \
    fprintf(stderr, "HIP error %s at %s:%d\n", hipGetErrorString(_e), __FILE__, __LINE__); \
    return; } } while (0)

namespace {
#define LAS __attribute__((address_space(3)))
typedef _Float16 f16x8 __attribute__((ext_vector_type(8)));
typedef float f32x4 __attribute__((ext_vector_type(4)));
typedef unsigned u32x4 __attribute__((ext_vector_type(4)));
typedef unsigned u32x2 __attribute__((ext_vector_type(2)));

#define KKL_ 0
constexpr int NR = 8, GM = 4096, GN = 16384, KSH = 512;
constexpr int PIECE = 32768, HTB = 16384, NKS = 64;
constexpr int FRS = KKL_ ? 1024 : 2048, KHS = KKL_ ? 8192 : 1024;
constexpr int RCOLS = 14, ROUNDS = 4, NRC = RCOLS * ROUNDS;
constexpr unsigned A_STEP = 16u * PIECE, B_STEP = (unsigned)RCOLS * PIECE;
constexpr unsigned A_BYTES = NKS * A_STEP, B_BYTES = (unsigned)(ROUNDS * NKS) * B_STEP, P_BYTES = (unsigned)NKS * PIECE;
constexpr unsigned STG_BYTES = A_BYTES + B_BYTES + P_BYTES;
constexpr size_t WS_CTR = 0, WS_FLAG = 16384, WS_TMO = 20480, WS_TICKET = 21504, WS_PROG = 22528, WS_PIN = 24576, WS_LCNT = 26624, WS_SENT = 26752, WS_PCNT = 26880, WS_PINC = 27008, WS_LCNT2 = 27136, WS_DCNT = 27264, WS_LMAX = 27392, WS_GO = 27520, WS_AMX = 28672, WS_DONE = 29696, WS_PKF = 30720, WS_DBG = 65536, WS_STG = 1u << 20;
constexpr size_t WS_PARK = WS_STG + 2 * (size_t)STG_BYTES;
constexpr size_t WS_NEED = WS_PARK + 3 * 256 * (size_t)262144;
#define JOIN2 2950
#define PLEAVE 3050
#define PREDELAY 40
#define FAUX 16
#define PSTART 120
#define OPERM 1
#define PRENT 0
#define POLLS 32
#define PKWT 1
#define PKAUX " sc1"
#define LDAUX 16
#define ADDOFF 8
#define ADDOFF2 0
#define PDAUX 18
#define NLV 1
#define NLRANK -1
#define PK2AUX " nt"
#define ROT 1
#define XSH 1
#define PINAT 2500
#define WIN 40
constexpr int U1 = 8 * 240, NUNITS = U1 + 24 * 112, NOUT = GM / 4, SEQ = 8192, NAGENT = 32, NPROG = 48, NRW = 224;
constexpr int SHIFT0 = 8;
constexpr int LDS_FLAG = 131072, LDS_BAD = LDS_FLAG + 2048, LDS_RING = LDS_BAD + 128, LDS_BYTES = LDS_RING + 3 * 8192;
constexpr unsigned long long DEADLINE = 50000000ull;
constexpr int AUX_WT = 17, AUX_NT = 2;

template <int V> struct IC { static constexpr int value = V; };
struct Params { const float* x; const float* w; float* out[NR]; char* ws[NR]; int rank; };

#define WAIT_V(n) asm volatile("s_waitcnt vmcnt(" #n ")" ::: "memory")
#define WAIT_L(n) asm volatile("s_waitcnt lgkmcnt(" #n ")" ::: "memory")
#define BAR __builtin_amdgcn_s_barrier()
#define SCHED __builtin_amdgcn_sched_barrier(0)
#define STAMPC(i) do {} while (0)
#define STAMPW(i) do {} while (0)
#define STAMP(i) do {} while (0)

template <bool FAST = false>
__device__ __forceinline__ bool wave_wait(const char* words, bool skip, unsigned need, unsigned* tmo, unsigned call, unsigned* seen = nullptr) {
  const unsigned long long t0 = __builtin_amdgcn_s_memrealtime();
  bool ok = true;
  for (unsigned n = 1;; ++n) {
    const unsigned v = (n & 7u) ? __hip_atomic_load((const unsigned*)words, __ATOMIC_RELAXED, __HIP_MEMORY_SCOPE_AGENT)
                                : __hip_atomic_load((const unsigned*)words, __ATOMIC_RELAXED, __HIP_MEMORY_SCOPE_SYSTEM);
    if (seen) *seen = v;
    if (__builtin_amdgcn_ballot_w64(skip || v > need) == ~0ull) break;
    if ((n & 7u) == 0u &&
        ((unsigned)__builtin_amdgcn_readfirstlane(__hip_atomic_load(tmo, __ATOMIC_RELAXED, __HIP_MEMORY_SCOPE_AGENT)) == call ||
         __builtin_amdgcn_s_memrealtime() - t0 > DEADLINE)) {
      __hip_atomic_store(tmo, call, __ATOMIC_RELAXED, __HIP_MEMORY_SCOPE_AGENT);
      ok = false; break;
    }
    __builtin_amdgcn_s_sleep(FAST ? 1 : POLLS);
  }
  asm volatile("" ::: "memory");
  return ok;
}

__device__ __forceinline__ float pow2f(int e) { return __builtin_bit_cast(float, (unsigned)(e + 127) << 23); }
__device__ __forceinline__ u32x4 pack8(float m, float a0, float a1, float a2, float a3, float a4, float a5, float a6, float a7) {
  f16x8 h = {(_Float16)(a0 * m), (_Float16)(a1 * m), (_Float16)(a2 * m), (_Float16)(a3 * m), (_Float16)(a4 * m), (_Float16)(a5 * m), (_Float16)(a6 * m), (_Float16)(a7 * m)};
  return __builtin_bit_cast(u32x4, h);
}

#define SEND_UNIT(NPRE, REPORT) do { \
    _Pragma("unroll") for (int i = 0; i < 4; ++i) { \
      if (i == 0) asm volatile("s_waitcnt vmcnt(%0)" :: "i"((WIN) + (NPRE)) : "memory"); \
      else if (i == 1) asm volatile("s_waitcnt vmcnt(%0)" :: "i"((WIN) > 7 ? (WIN) + (NPRE) : (WIN)) : "memory"); \
      else if (i == 2) { asm volatile("s_waitcnt vmcnt(%0)" :: "i"((WIN) > 14 ? 14 + (NPRE) : (WIN)) : "memory");     \
        if (REPORT) progress(seq0 + cur); } \
      else asm volatile("s_waitcnt vmcnt(%0)" :: "i"((WIN) + 1) : "memory"); \
      SCHED; \
      _Pragma("unroll") for (int d = 0; d < NR - 1; ++d) __builtin_amdgcn_raw_buffer_store_b128(o[i], rd[d], oo[i], 0, AUX_WT); \
      SCHED; } } while (0)

__global__ void __launch_bounds__(512, 2) fused_kernel(Params p) {
  extern __shared__ __attribute__((aligned(16))) char smem[];
  LAS char* lds = (LAS char*)smem;
  const int tid = threadIdx.x, wid = __builtin_amdgcn_readfirstlane(tid >> 6), lane = tid & 63;
  const int b = blockIdx.x, xcd = b & 7, li = b >> 3, rank = p.rank;
  char* myws = p.ws[rank];
  unsigned* tmo = (unsigned*)(myws + WS_TMO);
  STAMP(0);
  const int g4 = lane >> 4, r16 = lane & 15;
  const unsigned vx = (unsigned)(r16 * KSH + 4 * g4) * 4u, vw = (unsigned)(4 * g4 * GN + 4 * r16) * 4u, vd = (unsigned)lane * 16u;
  auto rot = [](int r) -> int { return ROT ? (r * 7) >> 2 : 0; };
  auto first_src = [](int cr) -> int { return ROT && cr > 1 ? (4 * (14 - cr) + 6) / 7 : 0; };
  auto half_of = [](int cr) -> int { return XSH ? ((cr >> 1) < 5 ? cr >> 1 : 5) + 1 - (cr & 1) : cr >> 1; };
  auto early = [&](int cr) -> bool { return ((rank - first_src(cr)) & 7) < 4; };
  auto phase_of = [&](int cr) -> int { return (half_of(cr) >> 1) + ((half_of(cr) & 1) && early(cr) ? 1 : 0); };
  auto rho0_of = [&](int cr) -> int { return 7 * (half_of(cr) >> 1) + ((half_of(cr) & 1) ? (early(cr) ? 3 : 4) : 0); };
  auto place = [&](bool isx, int col, int q4, int hs, unsigned& src, unsigned& dst) {
    const unsigned ks = (unsigned)((hs >> 1) * 8 + rank), kk = (unsigned)((hs & 1) * KHS);
    if (isx) { src = (unsigned)((col * 256 + q4 * 64) * KSH + hs * 32) * 4u; dst = ks * A_STEP + (unsigned)col * PIECE + kk + (unsigned)((q4 >> 1) * HTB + (q4 & 1) * 4 * FRS); }
    else {
      const int rr = col / RCOLS, cc = col - rr * RCOLS;
      const int wcol = col < NRC ? ((rr + ((hs >> 1) >= phase_of(cc) ? 1 : 0)) & 3) * RCOLS + cc : col;
      src = (unsigned)(hs * 32 * GN + wcol * 256 + q4 * 64) * 4u;
      dst = A_BYTES + (col < NRC ? ((unsigned)rr * NKS + ks) * B_STEP + (unsigned)cc * PIECE : B_BYTES + ks * PIECE) + kk + (unsigned)(q4 * 2 * FRS);
    }
  };
  auto load_unit = [&](f32x4 (&v)[8], bool valid, bool isx, unsigned src, auto aux) {
    const __amdgpu_buffer_rsrc_t rs = __builtin_amdgcn_make_buffer_rsrc((void*)(isx ? p.x : p.w), 0, valid ? (isx ? GM * KSH * 4 : KSH * GN * 4) : 0, 0x00020000);
    const unsigned vv = isx ? vx : vw;
#pragma unroll
    for (int j = 0; j < 8; ++j)
      v[j] = __builtin_bit_cast(f32x4, __builtin_amdgcn_raw_buffer_load_b128(rs, vv, src + (isx ? (unsigned)((j >> 1) * 16 * KSH + (j & 1) * 16) * 4u : (unsigned)(((j >> 2) * 16 + (j & 3)) * GN) * 4u), decltype(aux)::value));
  };
  auto pack_unit = [&](const f32x4 (&v)[8], float m, bool isx, unsigned dst, u32x4 (&o)[4], unsigned (&oo)[4]) {
#pragma unroll
    for (int i = 0; i < 4; ++i) {
      o[i] = isx ? pack8(m, v[2 * i][0], v[2 * i][1], v[2 * i][2], v[2 * i][3], v[2 * i + 1][0], v[2 * i + 1][1], v[2 * i + 1][2], v[2 * i + 1][3])
                 : pack8(m, v[0][i], v[1][i], v[2][i], v[3][i], v[4][i], v[5][i], v[6][i], v[7][i]);
      oo[i] = vd + dst + (unsigned)(isx ? i * FRS : (i >> 1) * HTB + (i & 1) * FRS);
    }
  };

  auto decode = [&](unsigned u, bool& isx, unsigned& src, unsigned& dst) {
    int sg, i; isx = false;
    if (u < (unsigned)U1) { sg = u / 240u; i = u % 240u; isx = i < 128; if (!isx) i -= 128; }
    else { const unsigned v = u - U1; sg = 8 + v / 112u; i = v % 112u; }
    place(isx, isx ? i >> 3 : (sg >> 3) * RCOLS + ((i >> 3) + RCOLS - rot(rank)) % RCOLS, i & 3, (sg & 7) * 2 + ((i >> 2) & 1), src, dst);
  };
  const int aid = wid == 0 ? b : NAGENT + b;
  f32x4 v[8]; bool isx = false; unsigned src = 0, dst = 0;
  if (li < 4 && wid == 0) { decode((unsigned)b, isx, src, dst); load_unit(v, true, isx, src, IC<0>{}); }
  if (tid == 0) {
    const unsigned c = __hip_atomic_fetch_add((unsigned*)(myws + WS_CTR + b * 64), 1u, __ATOMIC_RELAXED, __HIP_MEMORY_SCOPE_AGENT) + 1u;
    *(LAS unsigned*)(lds + LDS_BAD + 4) = c; *(LAS unsigned*)(lds + LDS_BAD) = 0u;
  }
  *(LAS unsigned*)(lds + LDS_FLAG + tid * 4) = 0u;
  WAIT_L(0); BAR; asm volatile("" ::: "memory");
  const unsigned call = __builtin_amdgcn_readfirstlane(*(LAS unsigned*)(lds + LDS_BAD + 4));
  const unsigned seq0 = call * SEQ;
  const size_t stg = WS_STG + (size_t)(call & 1u) * STG_BYTES;
  const char* flagw = myws + WS_FLAG + (lane & 7) * 128;

  unsigned* prog = (unsigned*)(myws + WS_PROG);
  unsigned* ticket = (unsigned*)(myws + WS_TICKET + (call & 3u) * 128u);
  auto draw = [&](unsigned* ctr, bool real) -> unsigned {
    return (unsigned)__builtin_amdgcn_raw_ptr_buffer_atomic_add_i32(1, __builtin_amdgcn_make_buffer_rsrc(ctr, 0, real ? 4 : 0, 0x00020000), (int)vd, 0, 0); };
  auto progress = [&](unsigned val) { __builtin_amdgcn_raw_buffer_store_b32(val, __builtin_amdgcn_make_buffer_rsrc(prog + aid, 0, 4, 0x00020000), vd, 0, 16); };
  auto read_go = [&]() -> unsigned { return __builtin_amdgcn_readfirstlane(__hip_atomic_load((unsigned*)(myws + WS_GO + 4), __ATOMIC_RELAXED, __HIP_MEMORY_SCOPE_AGENT)); };
  auto send_ops = [&](unsigned go, unsigned cur, unsigned tnext, unsigned limit) {
    unsigned* ctr = ticket + (go >> 31) * 8; const unsigned base = (go >> 31) ? 0u : 2u * NAGENT;
    const float mx = pow2f((int)((go >> 12) & 0xfffu) - 256), mw = pow2f((int)(go & 0xfffu) - 256);
    __amdgpu_buffer_rsrc_t rd[NR - 1];
#pragma unroll
    for (int d = 0; d < NR - 1; ++d) rd[d] = __builtin_amdgcn_make_buffer_rsrc(p.ws[(rank + 1 + (d + b + wid) % (NR - 1)) & 7] + stg, 0, STG_BYTES, 0x00020000);
    bool real = true;
#pragma unroll
    for (int j = 0; j < 8; ++j) asm volatile("" : "+v"(v[j]));
    asm volatile("" : "+v"(tnext));
    while (cur < (unsigned)NUNITS) {
      u32x4 o[4]; unsigned oo[4];
      pack_unit(v, isx ? mx : mw, isx, dst, o, oo);
      const unsigned nxt = real ? __builtin_amdgcn_readfirstlane(tnext) + base : 0xffffffffu;
      SCHED;
      decode(nxt < (unsigned)NUNITS ? nxt : 0u, isx, src, dst); load_unit(v, nxt < (unsigned)NUNITS, isx, src, IC<0>{});
      real = nxt < limit; tnext = draw(ctr, real);
      SCHED;
      SEND_UNIT(9, true);
      cur = nxt;
    }
    WAIT_V(0);
  };
  auto join_ops = [&](unsigned go, unsigned limit) {
    unsigned* ctr = ticket + (go >> 31) * 8;
    const unsigned cur = __builtin_amdgcn_readfirstlane(draw(ctr, true)) + ((go >> 31) ? 0u : 2u * NAGENT);
    decode(cur < (unsigned)NUNITS ? cur : 0u, isx, src, dst); load_unit(v, cur < (unsigned)NUNITS, isx, src, IC<0>{});
    send_ops(go, cur, draw(ctr, true), limit);
  };
  auto send_out = [&]() {
    unsigned cur = draw(ticket + 16, true), tnext = draw(ticket + 16, true);
    const bool ok = wave_wait(myws + WS_PCNT, false, call * 16u - 1u, tmo, call);
    cur = __builtin_amdgcn_readfirstlane(cur);
    __amdgpu_buffer_rsrc_t rd[NR - 1];
#pragma unroll
    for (int d = 0; d < NR - 1; ++d) rd[d] = __builtin_amdgcn_make_buffer_rsrc(p.out[(rank + 1 + (d + b + wid) % (NR - 1)) & 7], 0, GM * GN * 4, 0x00020000);
    const unsigned vb = vd + (unsigned)((NRC + rank) * 256 * 4);
    auto orow = [](unsigned u, unsigned i) -> unsigned {
      return OPERM == 0 ? (u * 4u + i) % (unsigned)GM : OPERM == 1 ? (((u * 37u) & 1023u) * 4u + i) : ((u * 37u) & 1023u) + i * 1024u; };
    auto load_out = [&](unsigned u) {
      const __amdgpu_buffer_rsrc_t rs = __builtin_amdgcn_make_buffer_rsrc(p.out[rank], 0, u < (unsigned)NOUT ? GM * GN * 4 : 0, 0x00020000);
#pragma unroll
      for (int i = 0; i < 4; ++i) v[i] = __builtin_bit_cast(f32x4, __builtin_amdgcn_raw_buffer_load_b128(rs, vb, orow(u, i) * (unsigned)(GN * 4), 16));
    };
    load_out(cur);
    if (li >= 2) wave_wait<true>(myws + WS_SENT, false, seq0 + (unsigned)(NUNITS - PSTART), tmo, call);
#pragma unroll
    for (int j = 0; j < 4; ++j) asm volatile("" : "+v"(v[j]));
    asm volatile("" : "+v"(tnext));
    bool real = true;
    while (cur < (unsigned)NOUT) {
      u32x4 o[4]; unsigned oo[4];
#pragma unroll
      for (int i = 0; i < 4; ++i) { o[i] = __builtin_bit_cast(u32x4, v[i]); if (!ok) o[i] = (u32x4){0x7fc00000u, 0x7fc00000u, 0x7fc00000u, 0x7fc00000u}; oo[i] = vb + orow(cur, i) * (unsigned)(GN * 4); asm volatile("" : "+v"(o[i])); }
      const unsigned nxt = real ? __builtin_amdgcn_readfirstlane(tnext) : 0xffffffffu;
      SCHED;
      load_out(nxt);
      real = nxt < (unsigned)NOUT; tnext = draw(ticket + 16, real);
      SCHED;
      SEND_UNIT(5, false);
      cur = nxt;
    }
    WAIT_V(0);
    if (lane < NR) __hip_atomic_fetch_add((unsigned*)(p.ws[lane] + WS_DONE + rank * 128), 1u, __ATOMIC_RELAXED, __HIP_MEMORY_SCOPE_SYSTEM);
    STAMPW(11);
  };

  bool ready = false;
  if (li < 4) {
    if (wid == 0) {
      const unsigned limit = li < 2 ? (unsigned)NUNITS : (unsigned)PLEAVE;
      send_ops((unsigned)(SHIFT0 + 256) << 12 | (unsigned)(SHIFT0 + 256), (unsigned)b, (unsigned)(b - NAGENT), limit);
      wave_wait<true>(myws + WS_GO, false, call - 1u, tmo, call);
      const unsigned go = read_go();
      if (go >> 31) { progress(seq0); WAIT_V(0); }
      if (lane == 0) __hip_atomic_fetch_add((unsigned*)(myws + WS_DCNT), 1u, __ATOMIC_RELAXED, __HIP_MEMORY_SCOPE_AGENT);
      if (go >> 31) { wave_wait<true>(myws + WS_DCNT, false, call * (unsigned)NAGENT - 1u, tmo, call); join_ops(go, limit); }
      progress(seq0 + (unsigned)(NUNITS + 8));
      STAMP(12);
      if (li < 2) send_out();
    } else if (wid == 2 && li < 2) {
      progress(seq0 + (unsigned)(SEQ - 1));
      wave_wait(myws + WS_SENT, false, seq0 + (unsigned)JOIN2, tmo, call);
      progress(seq0 + (unsigned)JOIN2);
      WAIT_V(0);
      join_ops(read_go(), (unsigned)NUNITS);
      progress(seq0 + (unsigned)(SEQ - 1));
    } else if (wid == 6) {
      wave_wait(myws + WS_SENT, false, seq0 + (unsigned)(PINAT), tmo, call);
      const float mw = pow2f((int)(read_go() & 0xfffu) - 256);
#pragma unroll 1
      for (int d = 1; d < NR; ++d) {
        const int pr = (rank + 1 + (d + b) % (NR - 1)) & 7;
        const __amdgpu_buffer_rsrc_t rp = __builtin_amdgcn_make_buffer_rsrc(p.ws[pr] + stg, 0, STG_BYTES, 0x00020000);
        f32x4 pv[2][8]; unsigned psrc, pdst[2];
#pragma unroll
        for (int e = 0; e < 2; ++e) { place(false, NRC + pr, (b >> 4) * 2 + e, b & 15, psrc, pdst[e]); load_unit(pv[e], true, false, psrc, IC<PRENT>{}); }
#pragma unroll
        for (int e = 0; e < 2; ++e) { u32x4 o[4]; unsigned oo[4]; pack_unit(pv[e], mw, false, pdst[e], o, oo);
#pragma unroll
          for (int i = 0; i < 4; ++i) __builtin_amdgcn_raw_buffer_store_b128(o[i], rp, oo[i], 0, AUX_WT); }
      }
      WAIT_V(0);
      if (lane == 0) __hip_atomic_fetch_add((unsigned*)(myws + WS_PINC), 1u, __ATOMIC_RELAXED, __HIP_MEMORY_SCOPE_AGENT);
    } else if (b == 0 && wid == 7) {
      if (lane < 3) __hip_atomic_store((unsigned*)(myws + WS_TICKET + ((call + 2u) & 3u) * 128u) + lane * 8, 0u, __ATOMIC_RELAXED, __HIP_MEMORY_SCOPE_AGENT);
      unsigned* myflag = (lane & 7) == rank ? (unsigned*)(myws + WS_SENT) : (unsigned*)(p.ws[lane & 7] + WS_FLAG + rank * 128);
      const unsigned long long t0 = __builtin_amdgcn_s_memrealtime();
      bool ok = wave_wait<true>(myws + WS_GO, false, call - 1u, tmo, call);
      if (read_go() >> 31) {
        ok = wave_wait<true>(myws + WS_LCNT2, false, call * (unsigned)NRW - 1u, tmo, call) && ok;
        ok = wave_wait<true>(myws + WS_DCNT, false, call * (unsigned)NAGENT - 1u, tmo, call) && ok;
      }
      if (lane == 0) __hip_atomic_store((unsigned*)(myws + WS_FLAG + rank * 128), seq0 + (unsigned)(SEQ - 1), __ATOMIC_RELAXED, __HIP_MEMORY_SCOPE_SYSTEM);
      unsigned last = seq0; bool pin = false;
      while (last < seq0 + (unsigned)(NUNITS + 8) || !pin) {
        unsigned m = 0xffffffffu;
#pragma unroll
        for (int j = 0; j < (NPROG + 63) / 64; ++j) { const int a = j * 64 + lane; if (a < NPROG) { const unsigned x = __hip_atomic_load(prog + a, __ATOMIC_RELAXED, __HIP_MEMORY_SCOPE_AGENT); m = x < m ? x : m; } }
        if (!pin && (unsigned)__builtin_amdgcn_readfirstlane(__hip_atomic_load((unsigned*)(myws + WS_PINC), __ATOMIC_RELAXED, __HIP_MEMORY_SCOPE_AGENT)) >= call * 32u) {
          pin = true;
          if (lane < NR) __hip_atomic_store((unsigned*)(p.ws[lane] + WS_PIN + rank * 128), call, __ATOMIC_RELAXED, __HIP_MEMORY_SCOPE_SYSTEM);
        }
#pragma unroll
        for (int sft = 32; sft >= 1; sft >>= 1) { const unsigned x = __shfl_xor(m, sft); m = x < m ? x : m; }
        m = __builtin_amdgcn_readfirstlane(m);
        if (m > last) { last = m; if (lane < NR) __hip_atomic_store(myflag, m, __ATOMIC_RELAXED, __HIP_MEMORY_SCOPE_SYSTEM); }
        if (__builtin_amdgcn_s_memrealtime() - t0 > DEADLINE) { __hip_atomic_store(tmo, call, __ATOMIC_RELAXED, __HIP_MEMORY_SCOPE_AGENT); ok = false; break; }
      }
      ok = wave_wait<true>(myws + WS_DONE + (lane & 7) * 128, false, call * (unsigned)NAGENT - 1u, tmo, call) && ok;
      ok = ok && __hip_atomic_load(tmo, __ATOMIC_RELAXED, __HIP_MEMORY_SCOPE_AGENT) != call;
      if (!ok && lane == 0) __hip_atomic_store((unsigned*)p.out[rank] + (NRC + rank) * 256, 0x7fc00000u, __ATOMIC_RELAXED, __HIP_MEMORY_SCOPE_SYSTEM);
    }
    if (li < 2) return;
  } else {
    const __amdgpu_buffer_rsrc_t rl = __builtin_amdgcn_make_buffer_rsrc(myws + stg, 0, STG_BYTES, 0x00020000);
    { const unsigned long long t0 = __builtin_amdgcn_s_memrealtime(); while (__builtin_amdgcn_s_memrealtime() - t0 < (unsigned long long)(PREDELAY * (li - 3))) __builtin_amdgcn_s_sleep(4); }
    float mx = pow2f(SHIFT0), mw = mx;
    ready = true;
    for (int pass = 0; pass < 2; ++pass) {
      unsigned ax = 0, aw = 0;
      for (int j = b - 32; j < 640; j += NRW) {
        isx = j < 128;
        if (j < 128) place(true, 2 * (j & 7) + (wid >> 2), wid & 3, j >> 3, src, dst);
        else if (j < 576) place(false, 2 * ((j - 128) % 28) + (wid >> 2), wid & 3, (j - 128) / 28, src, dst);
        else place(false, NRC + ((j - 576) >> 3), wid & 3, 2 * (j & 7) + (wid >> 2), src, dst);
        load_unit(v, true, isx, src, IC<PRENT>{});
        unsigned am = 0;
#pragma unroll
        for (int q = 0; q < 32; ++q) { unsigned a = __builtin_bit_cast(unsigned, v[q >> 2][q & 3]) & 0x7fffffffu; a = a < 0x7f800000u ? a : 0u; am = a > am ? a : am; }
        if (isx) ax = am > ax ? am : ax; else aw = am > aw ? am : aw;
        if (j < 576 || ((j - 576) >> 3) == rank) {
          u32x4 o[4]; unsigned oo[4]; pack_unit(v, isx ? mx : mw, isx, dst, o, oo);
#pragma unroll
          for (int i = 0; i < 4; ++i) __builtin_amdgcn_raw_buffer_store_b128(o[i], rl, oo[i], 0, AUX_WT);
        }
      }
      if (pass == 0) {
#pragma unroll
        for (int sft = 32; sft >= 1; sft >>= 1) { const unsigned a = __shfl_xor(ax, sft), c = __shfl_xor(aw, sft); ax = a > ax ? a : ax; aw = c > aw ? c : aw; }
        if (lane == 0) { *(LAS unsigned*)(lds + LDS_BAD + 32 + wid * 8) = ax; *(LAS unsigned*)(lds + LDS_BAD + 36 + wid * 8) = aw; }
      }
      WAIT_V(0); WAIT_L(0); BAR; asm volatile("" ::: "memory");
      if (pass == 1) break;
      if (wid == 0) {
        if (lane < 2) {
          unsigned a = 0;
#pragma unroll
          for (int q = 0; q < 8; ++q) { const unsigned c = *(LAS unsigned*)(lds + LDS_BAD + 32 + q * 8 + lane * 4); a = c > a ? c : a; }
          __hip_atomic_fetch_max((unsigned long long*)(myws + WS_LMAX) + lane, (unsigned long long)call << 32 | a, __ATOMIC_RELAXED, __HIP_MEMORY_SCOPE_AGENT);
        }
        WAIT_V(0);
        if ((unsigned)__builtin_amdgcn_readfirstlane(draw((unsigned*)(myws + WS_LCNT), true)) == call * (unsigned)NRW - 1u) {
          const unsigned long long mm = __hip_atomic_load((unsigned long long*)(myws + WS_LMAX) + (lane & 1), __ATOMIC_RELAXED, __HIP_MEMORY_SCOPE_AGENT);
          const unsigned e1 = (unsigned)mm ? 1u + ((unsigned)mm >> 23) : 0u;
          const unsigned e12 = (unsigned)__builtin_amdgcn_readlane(e1, 0) << 16 | (unsigned)__builtin_amdgcn_readlane(e1, 1);
          if (lane < NR) __hip_atomic_store((unsigned long long*)(p.ws[lane] + WS_AMX + rank * 128), (unsigned long long)e12 << 32 | call, __ATOMIC_RELAXED, __HIP_MEMORY_SCOPE_SYSTEM);
          wave_wait<true>(myws + WS_AMX + (lane & 7) * 128, false, call - 1u, tmo, call);
          const unsigned e2 = __hip_atomic_load((unsigned*)(myws + WS_AMX + (lane & 7) * 128 + 4), __ATOMIC_RELAXED, __HIP_MEMORY_SCOPE_SYSTEM);
          unsigned gx = e2 >> 16, gw = e2 & 0xffffu;
#pragma unroll
          for (int sft = 4; sft >= 1; sft >>= 1) { const unsigned a = __shfl_xor(gx, sft), c = __shfl_xor(gw, sft); gx = a > gx ? a : gx; gw = c > gw ? c : gw; }
          auto shift_of = [](unsigned g) -> int { const int e = (int)g - 128; return (g == 0u || (e >= -SHIFT0 && e < 15 - SHIFT0)) ? SHIFT0 : (14 - e > 126 ? 126 : 14 - e); };
          const int sx = shift_of(__builtin_amdgcn_readfirstlane(gx)), sw = shift_of(__builtin_amdgcn_readfirstlane(gw));
          const bool redo = sx != SHIFT0 || sw != SHIFT0;
          if (lane == 0) {
            __hip_atomic_store((unsigned long long*)(myws + WS_GO), (unsigned long long)((redo ? 0x80000000u : 0u) | (unsigned)(sx + 256) << 12 | (unsigned)(sw + 256)) << 32 | call, __ATOMIC_RELAXED, __HIP_MEMORY_SCOPE_AGENT);
            if (redo) __hip_atomic_fetch_add(ticket, 1u << 20, __ATOMIC_RELAXED, __HIP_MEMORY_SCOPE_AGENT);
          }
        }
        if (!wave_wait(myws + WS_GO, false, call - 1u, tmo, call)) *(LAS unsigned*)(lds + LDS_BAD) = 1u;
        *(LAS unsigned*)(lds + LDS_BAD + 8) = read_go();
        WAIT_L(0);
      }
      asm volatile("" ::: "memory"); BAR; asm volatile("" ::: "memory");
      const unsigned go = __builtin_amdgcn_readfirstlane(*(LAS unsigned*)(lds + LDS_BAD + 8));
      if (!(go >> 31)) break;
      mx = pow2f((int)((go >> 12) & 0xfffu) - 256); mw = pow2f((int)(go & 0xfffu) - 256); ready = false;
    }
    if (tid == 0) __hip_atomic_fetch_add((unsigned*)(myws + WS_LCNT2), 1u, __ATOMIC_RELAXED, __HIP_MEMORY_SCOPE_AGENT);
    STAMP(13);
  }

  const bool isP = li < 4;
  const int wr = wid >> 2, wc = wid & 3;
  const int lr = li - 4;
  const int mt = isP ? xcd * 2 + (li - 2) : (xcd >> 1) * 4 + (lr & 3);
  const int cr = isP ? 0 : (xcd & 1) + 2 * (lr >> 2);
  const unsigned voff = (unsigned)(wid * 2048 + lane * 16), ldsw = (unsigned)wid * 2048u;
  const char* fsrc = myws + WS_FLAG + wid * 128;
  const __amdgpu_buffer_rsrc_t ra = __builtin_amdgcn_make_buffer_rsrc(myws + stg, 0, A_BYTES, 0x00020000);
  const __amdgpu_buffer_rsrc_t rb = __builtin_amdgcn_make_buffer_rsrc(myws + stg + A_BYTES + (isP ? B_BYTES : 0u), 0, isP ? P_BYTES : B_BYTES, 0x00020000);
  const unsigned bstep = isP ? (unsigned)PIECE : B_STEP;
#define STAGE(ldsoff, rs, so) do { \
    __builtin_amdgcn_raw_ptr_buffer_load_lds(rs, (LAS void*)(lds + (ldsoff) + ldsw), 16, voff, (so), 0, LDAUX); \
    __builtin_amdgcn_raw_ptr_buffer_load_lds(rs, (LAS void*)(lds + (ldsoff) + ldsw), 16, voff, (so), 1024, LDAUX); } while (0)
#define ST1(ldsoff, rs, so, imm) do { __builtin_amdgcn_raw_ptr_buffer_load_lds(rs, (LAS void*)(lds + (ldsoff) + ldsw), 16, voff, (so), imm, LDAUX); SCHED; } while (0)
#define RA(h, m) do { _Pragma("unroll") for (int k = 0; k < 2; ++k) At[m][k] = *(const LAS f16x8*)((LAS char*)(size_t)va + (h) * HTB + (m) * FRS + k * KHS); SCHED; } while (0)
#define FLAGDMA() __builtin_amdgcn_global_load_lds((const unsigned*)fsrc, (LAS unsigned*)(lds + LDS_FLAG + wid * 256), 4, 0, FAUX)
#define LDA(dst, h) do { _Pragma("unroll") for (int m = 0; m < 4; ++m) _Pragma("unroll") for (int k = 0; k < 2; ++k) dst[m][k] = *(const LAS f16x8*)((LAS char*)(size_t)va + (h) * HTB + m * FRS + k * KHS); } while (0)
#define LDB(dst, h) do { _Pragma("unroll") for (int n = 0; n < 2; ++n) _Pragma("unroll") for (int k = 0; k < 2; ++k) dst[n][k] = *(const LAS f16x8*)((LAS char*)(size_t)vb + (h) * HTB + n * FRS + k * KHS); } while (0)
#define ACC(ai, nn, m) (128 + (((ai) * 4 + (nn)) * 4 + (m)) * 4)
#define MMA2(ai) do { MMA2_(ai); BAR; SCHED; } while (0)
#define MMA2_(ai) do { __builtin_amdgcn_s_waitcnt(0xC07F); SCHED; BAR; SCHED; __builtin_amdgcn_s_setprio(1); SCHED; \
    _Pragma("unroll") for (int m = 0; m < 4; ++m) _Pragma("unroll") for (int q = 0; q < 4; ++q) { const int nn = (m & 1) ? 3 - q : q; \
      asm volatile("v_mfma_f32_16x16x32_f16 v[%c0:%c1], %2, %3, v[%c0:%c1]\n\tv_mfma_f32_16x16x32_f16 v[%c0:%c1], %4, %5, v[%c0:%c1]" \
          :: "i"(ACC(ai, nn, m)), "i"(ACC(ai, nn, m) + 3), "v"(At[m][0]), "v"((nn >> 1) ? B1[nn & 1][0] : B0[nn & 1][0]), "v"(At[m][1]), "v"((nn >> 1) ? B1[nn & 1][1] : B0[nn & 1][1])); } \
    SCHED; __builtin_amdgcn_s_setprio(0); SCHED; } while (0)
#define ZERO_ACC() do { _Pragma("unroll") for (int i_ = 128; i_ < 256; i_ += 2) asm volatile("v_mov_b64 v[%c0:%c1], 0" :: "i"(i_), "i"(i_ + 1)); } while (0)
#define ACC_ROW(dst, ai, m, j) do { float t0_, t1_, t2_, t3_; \
    asm volatile("v_ldexp_f32 %0, v[%c4], %8\n\tv_ldexp_f32 %1, v[%c5], %8\n\tv_ldexp_f32 %2, v[%c6], %8\n\tv_ldexp_f32 %3, v[%c7], %8" \
        : "=&v"(t0_), "=&v"(t1_), "=&v"(t2_), "=&v"(t3_) : "i"(ACC(ai, 0, m) + (j)), "i"(ACC(ai, 1, m) + (j)), "i"(ACC(ai, 2, m) + (j)), "i"(ACC(ai, 3, m) + (j)), "v"(oexp)); \
    dst = (f32x4){t0_, t1_, t2_, t3_}; } while (0)
  asm volatile("" ::: "v128","v129","v130","v131","v132","v133","v134","v135","v136","v137","v138","v139","v140","v141","v142","v143","v144","v145","v146","v147","v148","v149","v150","v151","v152","v153","v154","v155","v156","v157","v158","v159","v160","v161","v162","v163","v164","v165","v166","v167","v168","v169","v170","v171","v172","v173","v174","v175","v176","v177","v178","v179","v180","v181","v182","v183","v184","v185","v186","v187","v188","v189","v190","v191","v192","v193","v194","v195","v196","v197","v198","v199","v200","v201","v202","v203","v204","v205","v206","v207","v208","v209","v210","v211","v212","v213","v214","v215","v216","v217","v218","v219","v220","v221","v222","v223","v224","v225","v226","v227","v228","v229","v230","v231","v232","v233","v234","v235","v236","v237","v238","v239","v240","v241","v242","v243","v244","v245","v246","v247","v248","v249","v250","v251","v252","v253","v254","v255");
  ZERO_ACC();
  f16x8 At[4][2], B0[2][2], B1[2][2];
  const int ph = isP ? 0 : phase_of(cr);
  const int NL = (NLRANK < 0 || rank == NLRANK) ? NLV : 0;
  const int crx = XSH ? cr ^ 1 : cr;
  const int E0 = 8 + rho0_of(cr);
  const int Tend = isP ? 64 : E0 + 192 + 56 - rho0_of(crx);
  unsigned tA, tB, tN;
  auto steps = [&](int t0) {
    const int t = t0 + lane, u = t - E0, c = u < 192 ? cr : crx, w = u - 8 * NL - 8, z = w - 56 * (1 + NL);
    const bool inl = z >= 0 && u < 192;
    const bool local = t < 8 || (u >= 0 && w < 0) || (inl && (z & 7) == 0);
    const int rho = u < 0 ? t - 8 : u >= 192 ? rho0_of(crx) + u - 24 : z < 0 ? E0 - 8 + w : E0 - 8 + w - (z >> 3) - 1, pos = rho % 7;
    const int sgm = !local ? rho / 7 : t < 8 ? (t < ph ? t : t + 24) : inl ? ph + 8 * (1 + NL) + (z >> 3) : u < 8 * NL ? ph + 8 + u : ph + (u & 7);
    const int f = first_src(c), sr = local ? rank : (f + pos + (pos >= ((rank - f) & 7) ? 1 : 0)) & 7;
    const unsigned ix = isP ? (unsigned)t : (unsigned)(sgm * 8 + sr);
    tA = t < Tend ? (unsigned)mt * PIECE + (ix & 63u) * A_STEP : 0x80000000u;
    tB = t < Tend ? (unsigned)c * PIECE + ix * bstep : 0x80000000u;
    tN = (isP || t >= Tend || local) ? 0u : seq0 + (unsigned)((sgm < 8 ? 240 * sgm + 128 : U1 - 8 * 112 + 112 * sgm) + 8 * ((c + rot(sr)) % RCOLS) + 8 + sr);
  };
  steps(0);
  if (!ready) {
    if (wid == 0) {
      if (isP && !wave_wait(myws + WS_PIN + (lane & 7) * 128, false, call - 1u, tmo, call)) *(LAS unsigned*)(lds + LDS_BAD) = 1u;
      if (!wave_wait(isP ? flagw : myws + WS_FLAG + rank * 128, false, seq0 + (unsigned)(U1 - 1), tmo, call)) *(LAS unsigned*)(lds + LDS_BAD) = 1u;
      *(LAS unsigned*)(lds + LDS_BAD + 8) = read_go(); WAIT_L(0);
    }
    asm volatile("" ::: "memory"); BAR; asm volatile("" ::: "memory");
  }
  int oexp;
  { const unsigned go = __builtin_amdgcn_readfirstlane(*(LAS unsigned*)(lds + LDS_BAD + 8)); oexp = 512 - (int)((go >> 12) & 0xfffu) - (int)(go & 0xfffu); }
  STAMP(1);
  __builtin_amdgcn_s_waitcnt(0x0F70); SCHED;
  unsigned par = 0, l1 = PIECE + HTB, a1, fl = 0;
  unsigned a2 = __builtin_amdgcn_readlane(tA, 2), b2 = __builtin_amdgcn_readlane(tB, 2), need = 0;
  unsigned va = (unsigned)(size_t)lds + wr * 4 * FRS + lane * 16, vb = (unsigned)(size_t)lds + 2 * PIECE + wc * 2 * FRS + lane * 16;
  LAS unsigned* fimg = (LAS unsigned*)(lds + LDS_FLAG + (lane & 7) * 256); asm volatile("" : "+v"(fimg));
  { const unsigned a0 = __builtin_amdgcn_readlane(tA, 0), b0 = __builtin_amdgcn_readlane(tB, 0), b1 = __builtin_amdgcn_readlane(tB, 1); a1 = __builtin_amdgcn_readlane(tA, 1);
    STAGE(2 * PIECE, rb, b0); STAGE(0, ra, a0); STAGE(2 * PIECE + HTB, rb, b0 + HTB); STAGE(HTB, ra, a0 + HTB);
    if (wr == 1) BAR;
    WAIT_V(2); BAR;
    FLAGDMA();
    STAGE(3 * PIECE, rb, b1); STAGE(PIECE, ra, a1); STAGE(3 * PIECE + HTB, rb, b1 + HTB);
    a1 += HTB;
    BAR; }
  const unsigned vpk = (unsigned)tid * 16u;
  const unsigned vring = (unsigned)(size_t)lds + LDS_RING + wid * 1024 + lane * 16;
#define SLOW_T0
#define SLOW_T1
#define STEP_STAMP
#define KLOOP(ADD, LIM) for (;;) { \
      f32x4 pt; \
      ST1(l1, ra, a1, 0); LDB(B0, 0); SCHED; ST1(l1, ra, a1, 1024); LDB(B1, 1); SCHED; \
      if (ADD) { __builtin_amdgcn_raw_ptr_buffer_load_lds(rpk, (LAS void*)(lds + LDS_RING + rs_ + wid * 1024), 16, vpk, (unsigned)(li3 - a0_) * 8192u, 0, PDAUX); SCHED; \
        rs_ = rs_ == 16384u ? 0u : rs_ + 8192u; pt = *(const LAS f32x4*)(size_t)(vring + rs_); } \
      else FLAGDMA(); \
      SCHED; LDA(At, 0); WAIT_V(10); \
      MMA2(0); \
      if ((unsigned)__builtin_amdgcn_readlane(fl, need & 63u) < need) { SLOW_T0; \
        const unsigned f2 = *(const volatile LAS unsigned*)fimg; fl = f2 > fl ? f2 : fl;     \
        if ((unsigned)__builtin_amdgcn_readlane(fl, need & 63u) < need) { \
          if (!wave_wait(flagw, (unsigned)(lane & 7) != (need & 7u), need - 1u, tmo, call, &fl)) *(LAS unsigned*)(lds + LDS_BAD) = 1u; \
          *(volatile LAS unsigned*)fimg = fl; } \
        SLOW_T1; } \
      ST1(2 * PIECE + par, rb, b2, 0); RA(1, 0); ST1(2 * PIECE + par, rb, b2, 1024); RA(1, 1); \
      ST1(par, ra, a2, 0); RA(1, 2); ST1(par, ra, a2, 1024); RA(1, 3); \
      const unsigned fn = ADD ? 0u : *fimg; \
      if (ADD && li3 - a0_ >= 2) { unsigned keep_; \
        asm volatile("s_mov_b32 %0, m0\n\ts_set_gpr_idx_on %1, gpr_idx(SRC0,DST)\n\tv_add_f32 v128, v128, %2\n\tv_add_f32 v129, v129, %3\n\tv_add_f32 v130, v130, %4\n\tv_add_f32 v131, v131, %5\n\ts_set_gpr_idx_off\n\ts_mov_b32 m0, %0" \
            : "=&s"(keep_) : "s"(((li3 - a0_ - 2) & 31) * 4), "v"(pt[0]), "v"(pt[1]), "v"(pt[2]), "v"(pt[3])); } \
      SCHED; \
      ST1(2 * PIECE + par + HTB, rb, b2 + HTB, 0); \
      l1 = par + HTB; a1 = a2 + HTB; va ^= PIECE; vb ^= PIECE; \
      a2 = __builtin_amdgcn_readlane(tA, li3); need = __builtin_amdgcn_readlane(tN, li3); \
      asm volatile("" : "+v"(va), "+v"(vb), "+s"(a2), "+s"(need), "+s"(l1), "+s"(a1)); SCHED; \
      ST1(2 * PIECE + par + HTB, rb, b2 + HTB, 1024); \
      par ^= PIECE; b2 = __builtin_amdgcn_readlane(tB, li3); ++li3; \
      STEP_STAMP \
      fl = fn > fl ? fn : fl; \
      asm volatile("" : "+s"(par), "+s"(li3), "+s"(b2), "+v"(fl)); \
      WAIT_V(9); \
      MMA2_(1); \
      if (li3 >= (LIM)) break; \
      BAR; SCHED; }
  int t = 0;
  steps(3);
  float* const myout = p.out[rank] + (size_t)(mt * 256 + wr * 64) * GN + wc * 64;
  for (int sg = isP ? 4 : -NL; sg < 5; ++sg) {
    const int Ts = sg == 4 ? Tend : sg == -NL ? E0 : t + (sg <= 0 ? 8 : (sg >= 2 && sg < 2 + NL) ? 56 : 64);
    const int slot = sg <= 0 ? sg + NL : (sg == 4 ? 0 : sg - 1);
    char* pk = myws + WS_PARK + (size_t)(slot * 256 + (XSH && sg == 4 ? b ^ 1 : b)) * 262144;
    const __amdgpu_buffer_rsrc_t rpk = __builtin_amdgcn_make_buffer_rsrc(pk, 0, 262144, 0x00020000);
    const int left = Ts - t; int li3 = 0;
    unsigned rs_ = 0;
    const bool add = !isP && (sg == 4 || (sg >= 2 && sg < 2 + NL));
    const int a1_ = sg == 4 ? left - 35 - ADDOFF : 29 - E0 - ADDOFF2, a0_ = (add && a1_ > 0) ? a1_ : 0;
    for (int part = a0_ ? 0 : 1; part < 2; ++part) {
      if (part == 1 && add) { KLOOP(true, a0_ + 34); BAR; SCHED; }
      const int lim_ = part ? left : a0_;
      KLOOP(false, lim_);
      if (!part) { BAR; SCHED; }
    }
    t = Ts;
    if (wr == 0) BAR;
    asm volatile("s_nop 15\n\ts_nop 15" ::: "memory");
    SCHED;
    STAMP(2 + 2 * r);
    if (sg <= 0) {
      unsigned vq = vpk; asm volatile("" : "+v"(vq));
      if (sg == -NL) {
#pragma unroll
      for (int i = 0; i < 32; ++i)
        asm volatile("s_nop 4\n\tglobal_store_dwordx4 %0, v[%c1:%c2], %3" PKAUX :: "v"(vq + i * 8192u), "i"(128 + 4 * i), "i"(128 + 4 * i + 3), "s"(pk) : "memory");
      } else {
#pragma unroll
      for (int i = 0; i < 32; ++i)
        asm volatile("s_nop 4\n\tglobal_store_dwordx4 %0, v[%c1:%c2], %3" PK2AUX :: "v"(vq + i * 8192u), "i"(128 + 4 * i), "i"(128 + 4 * i + 3), "s"(pk) : "memory");
      }
      asm volatile("s_nop 1");
      ZERO_ACC();
      steps(t + 3);
      if (XSH && sg == -NL) {
        WAIT_V(0);
        if (lane == 0) __hip_atomic_store((unsigned*)(myws + WS_PKF) + (size_t)(b * 8 + wid) * 2, call, __ATOMIC_RELAXED, __HIP_MEMORY_SCOPE_AGENT);
        if (XSH == 2 && !wave_wait(myws + WS_PKF + (size_t)((b ^ 1) * 8 + wid) * 8, false, call - 1u, tmo, call)) *(LAS unsigned*)(lds + LDS_BAD) = 1u;
      }
    } else {
      if (XSH == 1 && sg == 1 && !wave_wait(myws + WS_PKF + (size_t)((b ^ 1) * 8 + wid) * 8, false, call - 1u, tmo, call)) *(LAS unsigned*)(lds + LDS_BAD) = 1u;
      if (__builtin_amdgcn_readfirstlane(*(const LAS unsigned*)(lds + LDS_BAD)) != 0u) {
#pragma unroll
        for (int i = 128; i < 256; ++i) asm volatile("v_mov_b32 v[%c0], 0x7fc00000" :: "i"(i));
      }
      const int nt = isP ? NRC + rank : sg == 4 ? crx : sg * RCOLS + cr;
      const __amdgpu_buffer_rsrc_t rc = __builtin_amdgcn_make_buffer_rsrc(myout + nt * 256, 0, 192u * GN * 4u, 0x00020000);
      unsigned vo = (unsigned)((lane >> 4) * 4 * GN + (lane & 15) * 4) * 4u; asm volatile("" : "+v"(vo));
#pragma unroll
      for (int ai = 0; ai < 2; ++ai)
#pragma unroll
        for (int m = 0; m < 4; ++m) {
#pragma unroll
          for (int j = 0; j < 4; ++j) {
            f32x4 v4; ACC_ROW(v4, ai, m, j);
            __builtin_amdgcn_raw_buffer_store_b128(__builtin_bit_cast(u32x4, v4), rc, vo + (unsigned)((ai * 128 + m * 16 + j) * GN * 4), 0, AUX_NT);
          }
#pragma unroll
          for (int nn = 0; nn < 4; ++nn)
            asm volatile("v_mov_b64 v[%c0:%c1], 0\n\tv_mov_b64 v[%c2:%c3], 0" :: "i"(ACC(ai, nn, m)), "i"(ACC(ai, nn, m) + 1), "i"(ACC(ai, nn, m) + 2), "i"(ACC(ai, nn, m) + 3));
          if (ai == 0 && m == 1) {
            SCHED; steps(t + 3); asm volatile("" : "+v"(tA), "+v"(tB), "+v"(tN)); SCHED;
          }
        }
    }
    if (wr == 1) BAR;
    STAMP(3 + 2 * r);
    STAMPC(20 + r);
  }
  WAIT_V(0);
  STAMP(15);
  if (wr == 0) BAR;
  if (isP) {
    BAR; asm volatile("" ::: "memory");
    if (tid == 0) {
      __builtin_amdgcn_fence(__ATOMIC_RELEASE, "agent");
      WAIT_V(0);
      __hip_atomic_fetch_add((unsigned*)(myws + WS_PCNT), 1u, __ATOMIC_RELAXED, __HIP_MEMORY_SCOPE_AGENT);
    }
    STAMP(10);
    if (wid == 0) send_out();
  }
}
}

extern "C" void kernel_launch(int rank, int nranks, ncclComm_t comm, hipStream_t stream,
                              void* const* d_in, const int* in_sizes, int n_in,
                              void* d_out, int out_size, const int* in_dtypes, int out_dtype,
                              void* const* d_out_all, void* const* d_ws_all, size_t ws_bytes) {
  (void)comm; (void)in_dtypes; (void)out_dtype; (void)d_out;
  if (nranks != NR || (unsigned)rank >= (unsigned)NR || n_in < 2 || ws_bytes < WS_NEED || out_size != GM * GN ||
      in_sizes[0] != GM * KSH || in_sizes[1] != KSH * GN) { fprintf(stderr, "unexpected problem shape\n"); return; }
  static bool attr_set = false;
  if (!attr_set) { HIP_CHECK(hipFuncSetAttribute((const void*)fused_kernel, hipFuncAttributeMaxDynamicSharedMemorySize, LDS_BYTES)); attr_set = true; }
  Params p;
  p.x = (const float*)d_in[0]; p.w = (const float*)d_in[1]; p.rank = rank;
  for (int r = 0; r < NR; ++r) { p.out[r] = (float*)d_out_all[r]; p.ws[r] = (char*)d_ws_all[r]; }
  fused_kernel<<<256, 512, LDS_BYTES, stream>>>(p);
  HIP_CHECK(hipGetLastError());
}
```

```cpp
#include <hip/hip_runtime.h>
#include <rccl/rccl.h>
#include <cstdio>

#define HIP_CHECK(expr) do { hipError_t _e = (expr); if (_e != hipSuccess) { \
    fprintf(stderr, "HIP error %s at %s:%d\n", hipGetErrorString(_e), __FILE__, __LINE__); \
    return; } } while (0)

namespace {
#define LAS __attribute__((address_space(3)))
#define INL __attribute__((always_inline))
typedef _Float16 f16x8 __attribute__((ext_vector_type(8)));
typedef float f32x4 __attribute__((ext_vector_type(4)));
typedef unsigned u32x4 __attribute__((ext_vector_type(4)));
typedef unsigned u32x2 __attribute__((ext_vector_type(2)));

#define KKL_ 0
constexpr int NR = 8, GM = 4096, GN = 16384, KSH = 512;
constexpr int PIECE = 32768, HTB = 16384, NKS = 64;
constexpr int FRS = KKL_ ? 1024 : 2048, KHS = KKL_ ? 8192 : 1024;
constexpr int RCOLS = 14, ROUNDS = 4, NRC = RCOLS * ROUNDS;
constexpr unsigned A_STEP = 16u * PIECE, B_STEP = (unsigned)RCOLS * PIECE;
constexpr int NDUP = 16;
constexpr unsigned A_BYTES = (NKS + NDUP) * A_STEP, B_BYTES = (unsigned)(ROUNDS * NKS + NDUP) * B_STEP, P_BYTES = (unsigned)NKS * PIECE;
constexpr unsigned STG_BYTES = A_BYTES + B_BYTES + P_BYTES;
constexpr size_t WS_CTR = 0, WS_FLAG = 16384, WS_TMO = 20480, WS_TICKET = 21504, WS_PROG = 22528, WS_PIN = 24576, WS_LCNT = 26624, WS_SENT = 26752, WS_PCNT = 26880, WS_PINC = 27008, WS_LCNT2 = 27136, WS_DCNT = 27264, WS_LMAX = 27392, WS_GO = 27520, WS_AMX = 28672, WS_DONE = 29696, WS_START = 18432, WS_LATE = 27648, WS_PKF = 30720, WS_DBG = 65536, WS_STG = 1u << 20;
constexpr size_t WS_PARK = WS_STG + (size_t)STG_BYTES;
constexpr size_t WS_NEED = WS_PARK + 3 * 256 * (size_t)262144;
#define JOIN2 2950
#define PLEAVE 3050
#define PREDELAY 40
#define FAUX 16
#define PSTART 120
#define OPERM 1
#define PRENT 1
#define POLLS 32
#define PKWT 1
#define PKAUX " sc1"
#define LDAUX 16
#define NCOPY 1920u
#define HS 5
#define ADDODD 0
#define ADDOFF 8
#define ADDOFF2 0
#define PDAUX 18
#define NLV 1
#define LATEN 4
#define LATED 300
#define PK2AUX " nt"
#define ROT 1
#define XSH 1
#define PINAT 2500
#define WIN 40
constexpr int U1 = 8 * 240, NUNITS = U1 + 24 * 112, NOUT = GM / 4, SEQ = 8192, NAGENT = 32, NPROG = 48, NRW = 224;
constexpr int SHIFT0 = 8;
constexpr int LDS_FLAG = 131072, LDS_BAD = LDS_FLAG + 2048, LDS_RING = LDS_BAD + 128, LDS_BYTES = LDS_RING + 3 * 8192;
constexpr unsigned long long DEADLINE = 50000000ull;
constexpr int AUX_WT = 17, AUX_NT = 2;

template <int V> struct IC { static constexpr int value = V; };
struct Params { const float* x; const float* w; float* out[NR]; char* ws[NR]; int rank; };

#define WAIT_V(n) asm volatile("s_waitcnt vmcnt(" #n ")" ::: "memory")
#define WAIT_L(n) asm volatile("s_waitcnt lgkmcnt(" #n ")" ::: "memory")
#define BAR __builtin_amdgcn_s_barrier()
#define SCHED __builtin_amdgcn_sched_barrier(0)
#define STAMPC(i) do {} while (0)
#define STAMPW(i) do {} while (0)
#define STAMP(i) do {} while (0)

template <bool FAST = false>
__device__ __forceinline__ bool wave_wait(const char* words, bool skip, unsigned need, unsigned* tmo, unsigned call, unsigned* seen = nullptr) {
  const unsigned long long t0 = __builtin_amdgcn_s_memrealtime();
  bool ok = true;
  for (unsigned n = 1;; ++n) {
    const unsigned v = (n & 7u) ? __hip_atomic_load((const unsigned*)words, __ATOMIC_RELAXED, __HIP_MEMORY_SCOPE_AGENT)
                                : __hip_atomic_load((const unsigned*)words, __ATOMIC_RELAXED, __HIP_MEMORY_SCOPE_SYSTEM);
    if (seen) *seen = v;
    if (__builtin_amdgcn_ballot_w64(skip || v > need) == ~0ull) break;
    if ((n & 7u) == 0u &&
        ((unsigned)__builtin_amdgcn_readfirstlane(__hip_atomic_load(tmo, __ATOMIC_RELAXED, __HIP_MEMORY_SCOPE_AGENT)) == call ||
         __builtin_amdgcn_s_memrealtime() - t0 > DEADLINE)) {
      __hip_atomic_store(tmo, call, __ATOMIC_RELAXED, __HIP_MEMORY_SCOPE_AGENT);
      ok = false; break;
    }
    __builtin_amdgcn_s_sleep(FAST ? 1 : POLLS);
  }
  asm volatile("" ::: "memory");
  return ok;
}

__device__ __forceinline__ float pow2f(int e) { return __builtin_bit_cast(float, (unsigned)(e + 127) << 23); }
__device__ __forceinline__ u32x4 pack8(float m, float a0, float a1, float a2, float a3, float a4, float a5, float a6, float a7) {
  f16x8 h = {(_Float16)(a0 * m), (_Float16)(a1 * m), (_Float16)(a2 * m), (_Float16)(a3 * m), (_Float16)(a4 * m), (_Float16)(a5 * m), (_Float16)(a6 * m), (_Float16)(a7 * m)};
  return __builtin_bit_cast(u32x4, h);
}

__device__ __forceinline__ void load_send_(f32x4 (&v)[8], bool valid, bool copy, bool isx, unsigned src, unsigned dst,
                                           const void* stg, const void* xp, const void* wp, int lane) {
  const __amdgpu_buffer_rsrc_t rs = __builtin_amdgcn_make_buffer_rsrc((void*)(copy ? stg : isx ? xp : wp), 0,
                                                                      !valid ? 0u : copy ? STG_BYTES : isx ? GM * KSH * 4 : KSH * GN * 4, 0x00020000);
  const unsigned vv = copy ? (unsigned)lane * 16u : isx ? (unsigned)((lane & 15) * KSH + 4 * (lane >> 4)) * 4u : (unsigned)(4 * (lane >> 4) * GN + 4 * (lane & 15)) * 4u;
#pragma unroll
  for (int j = 0; j < 8; ++j) {
    const unsigned so = copy ? (j < 4 ? dst + (unsigned)((j >> 1) * HTB + (j & 1) * FRS) : 0x80000000u)
                             : src + (isx ? (unsigned)((j >> 1) * 16 * KSH + (j & 1) * 16) * 4u : (unsigned)(((j >> 2) * 16 + (j & 3)) * GN) * 4u);
    v[j] = __builtin_bit_cast(f32x4, __builtin_amdgcn_raw_buffer_load_b128(rs, vv, so, 16));
  }
}

#define SEND_UNIT(NPRE, REPORT) do { \
    _Pragma("unroll") for (int i = 0; i < 4; ++i) { \
      if (i == 0) asm volatile("s_waitcnt vmcnt(%0)" :: "i"((WIN) + (NPRE)) : "memory"); \
      else if (i == 1) asm volatile("s_waitcnt vmcnt(%0)" :: "i"((WIN) > 7 ? (WIN) + (NPRE) : (WIN)) : "memory"); \
      else if (i == 2) { asm volatile("s_waitcnt vmcnt(%0)" :: "i"((WIN) > 14 ? 14 + (NPRE) : (WIN)) : "memory");     \
        if (REPORT) progress(seq0 + cur); } \
      else asm volatile("s_waitcnt vmcnt(%0)" :: "i"((WIN) + 1) : "memory"); \
      SCHED; \
      _Pragma("unroll") for (int d = 0; d < NR - 1; ++d) __builtin_amdgcn_raw_buffer_store_b128(o[i], rd[d], oo[i], 0, AUX_WT); \
      SCHED; } } while (0)

__global__ void __launch_bounds__(512, 2) fused_kernel(Params p) {
  extern __shared__ __attribute__((aligned(16))) char smem[];
  LAS char* lds = (LAS char*)smem;
  const int tid = threadIdx.x, wid = __builtin_amdgcn_readfirstlane(tid >> 6), lane = tid & 63;
  const int b = blockIdx.x, xcd = b & 7, li = b >> 3, rank = p.rank;
  char* myws = p.ws[rank];
  unsigned* tmo = (unsigned*)(myws + WS_TMO);
  STAMP(0);
  const int g4 = lane >> 4, r16 = lane & 15;
  const unsigned vx = (unsigned)(r16 * KSH + 4 * g4) * 4u, vw = (unsigned)(4 * g4 * GN + 4 * r16) * 4u, vd = (unsigned)lane * 16u;
  auto rot = [](int r) INL -> int { return ROT ? (r * 7) >> 2 : 0; };
  auto first_src = [](int cr) INL -> int { return ROT && cr > 1 ? (4 * (14 - cr) + 6) / 7 : 0; };
  auto half_of = [](int cr) INL -> int { return XSH ? ((cr >> 1) * HS + 5) / 6 + 1 - (cr & 1) : cr >> 1; };
  auto early = [&](int cr) INL -> bool { return ((rank - first_src(cr)) & 7) < 4; };
  auto phase_of = [&](int cr) INL -> int { return (half_of(cr) >> 1) + ((half_of(cr) & 1) && early(cr) ? 1 : 0); };
  auto rho0_of = [&](int cr) INL -> int { return 7 * (half_of(cr) >> 1) + ((half_of(cr) & 1) ? (early(cr) ? 3 : 4) : 0); };
  unsigned odd = 0;
  auto dup_of = [](bool isx) INL -> unsigned { return isx ? NKS * A_STEP : (unsigned)(ROUNDS * NKS) * B_STEP; };
  auto place = [&](bool isx, int col, int q4, int hs, unsigned& src, unsigned& dst) INL {
    const unsigned ks = (unsigned)((hs >> 1) * 8 + rank), kk = (unsigned)((hs & 1) * KHS);
    if (isx) { src = (unsigned)((col * 256 + q4 * 64) * KSH + hs * 32) * 4u; dst = ks * A_STEP + (unsigned)col * PIECE + kk + (unsigned)((q4 >> 1) * HTB + (q4 & 1) * 4 * FRS); }
    else {
      const int rr = col / RCOLS, cc = col - rr * RCOLS;
      const int wcol = col < NRC ? ((rr + ((hs >> 1) >= phase_of(cc) ? 1 : 0)) & 3) * RCOLS + cc : col;
      src = (unsigned)(hs * 32 * GN + wcol * 256 + q4 * 64) * 4u;
      dst = A_BYTES + (col < NRC ? ((unsigned)rr * NKS + ks) * B_STEP + (unsigned)cc * PIECE : B_BYTES + ks * PIECE) + kk + (unsigned)(q4 * 2 * FRS);
    }
    if (odd && ks < (unsigned)NDUP && (isx || col < RCOLS)) dst += dup_of(isx);
  };
  auto load_unit = [&](f32x4 (&v)[8], bool valid, bool isx, unsigned src, auto aux) INL {
    const __amdgpu_buffer_rsrc_t rs = __builtin_amdgcn_make_buffer_rsrc((void*)(isx ? p.x : p.w), 0, valid ? (isx ? GM * KSH * 4 : KSH * GN * 4) : 0, 0x00020000);
    const unsigned vv = isx ? vx : vw;
#pragma unroll
    for (int j = 0; j < 8; ++j)
      v[j] = __builtin_bit_cast(f32x4, __builtin_amdgcn_raw_buffer_load_b128(rs, vv, src + (isx ? (unsigned)((j >> 1) * 16 * KSH + (j & 1) * 16) * 4u : (unsigned)(((j >> 2) * 16 + (j & 3)) * GN) * 4u), decltype(aux)::value));
  };
  auto pack_unit = [&](const f32x4 (&v)[8], float m, bool isx, unsigned dst, u32x4 (&o)[4], unsigned (&oo)[4]) INL {
#pragma unroll
    for (int i = 0; i < 4; ++i) {
      o[i] = isx ? pack8(m, v[2 * i][0], v[2 * i][1], v[2 * i][2], v[2 * i][3], v[2 * i + 1][0], v[2 * i + 1][1], v[2 * i + 1][2], v[2 * i + 1][3])
                 : pack8(m, v[0][i], v[1][i], v[2][i], v[3][i], v[4][i], v[5][i], v[6][i], v[7][i]);
      oo[i] = vd + dst + (unsigned)(isx ? i * FRS : (i >> 1) * HTB + (i & 1) * FRS);
    }
  };

  auto decode = [&](unsigned u, bool& isx, unsigned& src, unsigned& dst) INL {
    int sg, i; isx = false;
    if (u < (unsigned)U1) { sg = u / 240u; i = u % 240u; isx = i < 128; if (!isx) i -= 128; }
    else { const unsigned v = u - U1; sg = 8 + v / 112u; i = v % 112u; }
    place(isx, isx ? i >> 3 : (sg >> 3) * RCOLS + ((i >> 3) + RCOLS - rot(rank)) % RCOLS, i & 3, (sg & 7) * 2 + ((i >> 2) & 1), src, dst);
  };
  const int aid = wid == 0 ? b : NAGENT + b;
  f32x4 v[8]; bool isx = false; unsigned src = 0, dst = 0;
  if (li < 4 && wid == 0) { decode((unsigned)b, isx, src, dst); load_unit(v, true, isx, src, IC<0>{}); }
  if (tid == 0) {
    const unsigned c = __hip_atomic_fetch_add((unsigned*)(myws + WS_CTR + b * 64), 1u, __ATOMIC_RELAXED, __HIP_MEMORY_SCOPE_AGENT) + 1u;
    *(LAS unsigned*)(lds + LDS_BAD + 4) = c; *(LAS unsigned*)(lds + LDS_BAD) = 0u;
  }
  *(LAS unsigned*)(lds + LDS_FLAG + tid * 4) = 0u;
  WAIT_L(0); BAR; asm volatile("" ::: "memory");
  const unsigned call = __builtin_amdgcn_readfirstlane(*(LAS unsigned*)(lds + LDS_BAD + 4));
  const unsigned seq0 = call * SEQ;
  constexpr size_t stg = WS_STG;
  odd = call & 1u;
  if (odd) dst += dup_of(isx);
  const char* flagw = myws + WS_FLAG + (lane & 7) * 128;

  unsigned* prog = (unsigned*)(myws + WS_PROG);
  unsigned* ticket = (unsigned*)(myws + WS_TICKET + (call & 3u) * 128u);
  auto draw = [&](unsigned* ctr, bool real) INL -> unsigned {
    return (unsigned)__builtin_amdgcn_raw_ptr_buffer_atomic_add_i32(1, __builtin_amdgcn_make_buffer_rsrc(ctr, 0, real ? 4 : 0, 0x00020000), (int)vd, 0, 0); };
  auto progress = [&](unsigned val) INL { __builtin_amdgcn_raw_buffer_store_b32(val, __builtin_amdgcn_make_buffer_rsrc(prog + aid, 0, 4, 0x00020000), vd, 0, 16); };
  auto read_go = [&]() INL -> unsigned { return __builtin_amdgcn_readfirstlane(__hip_atomic_load((unsigned*)(myws + WS_GO + 4), __ATOMIC_RELAXED, __HIP_MEMORY_SCOPE_AGENT)); };
  bool copy = false;
#define load_send(valid) load_send_(v, valid, copy, isx, src, dst, myws + WS_STG, p.x, p.w, lane)
  auto send_ops = [&](unsigned go, unsigned cur, unsigned tnext, unsigned limit) INL {
    unsigned* ctr = ticket + (go >> 31) * 8; const unsigned base = (go >> 31) ? 0u : 2u * NAGENT;
    const float mx = pow2f((int)((go >> 12) & 0xfffu) - 256), mw = pow2f((int)(go & 0xfffu) - 256);
    __amdgpu_buffer_rsrc_t rd[NR - 1];
#pragma unroll
    for (int d = 0; d < NR - 1; ++d) rd[d] = __builtin_amdgcn_make_buffer_rsrc(p.ws[(rank + 1 + (d + b + wid) % (NR - 1)) & 7] + stg, 0, STG_BYTES, 0x00020000);
    bool real = true, open = go >> 31;
#pragma unroll
    for (int j = 0; j < 8; ++j) asm volatile("" : "+v"(v[j]));
    asm volatile("" : "+v"(tnext));
    while (cur < (unsigned)NUNITS) {
      u32x4 o[4]; unsigned oo[4];
      pack_unit(v, isx ? mx : mw, isx, dst, o, oo);
      if (copy) {
#pragma unroll
        for (int i = 0; i < 4; ++i) o[i] = __builtin_bit_cast(u32x4, v[i]);
      }
      const unsigned raw = __builtin_amdgcn_readfirstlane(tnext) + base, nxt = real ? raw & 0xffffffu : 0xffffffffu;
      open = open || (real && ((raw >> 24) & 1u));
      if (!open && cur >= (unsigned)(NDUP / 8 * 240)) { wave_wait<true>(myws + WS_START + (lane & 7) * 128, false, call - 1u, tmo, call); open = true; }
      SCHED;
      decode(nxt < (unsigned)NUNITS ? nxt : 0u, isx, src, dst);
      copy = (raw >> 25) && nxt >= NCOPY;
      load_send(nxt < (unsigned)NUNITS);
      real = nxt < limit; tnext = draw(ctr, real);
      SCHED;
      SEND_UNIT(9, true);
      cur = nxt;
    }
    WAIT_V(0);
  };
  auto join_ops = [&](unsigned go, unsigned limit) INL {
    unsigned* ctr = ticket + (go >> 31) * 8;
    const unsigned cur = (__builtin_amdgcn_readfirstlane(draw(ctr, true)) & 0xffffffu) + ((go >> 31) ? 0u : 2u * NAGENT);
    decode(cur < (unsigned)NUNITS ? cur : 0u, isx, src, dst); copy = false; load_send(cur < (unsigned)NUNITS);
    send_ops(go, cur, draw(ctr, true), limit);
  };
  auto send_out = [&]() INL {
    unsigned cur = draw(ticket + 16, true), tnext = draw(ticket + 16, true);
    const bool ok = wave_wait(myws + WS_PCNT, false, call * 16u - 1u, tmo, call);
    cur = __builtin_amdgcn_readfirstlane(cur);
    __amdgpu_buffer_rsrc_t rd[NR - 1];
#pragma unroll
    for (int d = 0; d < NR - 1; ++d) rd[d] = __builtin_amdgcn_make_buffer_rsrc(p.out[(rank + 1 + (d + b + wid) % (NR - 1)) & 7], 0, GM * GN * 4, 0x00020000);
    const unsigned vb = vd + (unsigned)((NRC + rank) * 256 * 4);
    auto orow = [](unsigned u, unsigned i) INL -> unsigned {
      return OPERM == 0 ? (u * 4u + i) % (unsigned)GM : OPERM == 1 ? (((u * 37u) & 1023u) * 4u + i) : ((u * 37u) & 1023u) + i * 1024u; };
    auto load_out = [&](unsigned u) INL {
      const __amdgpu_buffer_rsrc_t rs = __builtin_amdgcn_make_buffer_rsrc(p.out[rank], 0, u < (unsigned)NOUT ? GM * GN * 4 : 0, 0x00020000);
#pragma unroll
      for (int i = 0; i < 4; ++i) v[i] = __builtin_bit_cast(f32x4, __builtin_amdgcn_raw_buffer_load_b128(rs, vb, orow(u, i) * (unsigned)(GN * 4), 16));
    };
    load_out(cur);
    if (li >= 2) wave_wait<true>(myws + WS_SENT, false, seq0 + (unsigned)(NUNITS - PSTART), tmo, call);
#pragma unroll
    for (int j = 0; j < 4; ++j) asm volatile("" : "+v"(v[j]));
    asm volatile("" : "+v"(tnext));
    bool real = true;
    while (cur < (unsigned)NOUT) {
      u32x4 o[4]; unsigned oo[4];
#pragma unroll
      for (int i = 0; i < 4; ++i) { o[i] = __builtin_bit_cast(u32x4, v[i]); if (!ok) o[i] = (u32x4){0x7fc00000u, 0x7fc00000u, 0x7fc00000u, 0x7fc00000u}; oo[i] = vb + orow(cur, i) * (unsigned)(GN * 4); asm volatile("" : "+v"(o[i])); }
      const unsigned nxt = real ? __builtin_amdgcn_readfirstlane(tnext) : 0xffffffffu;
      SCHED;
      load_out(nxt);
      real = nxt < (unsigned)NOUT; tnext = draw(ticket + 16, real);
      SCHED;
      SEND_UNIT(5, false);
      cur = nxt;
    }
    WAIT_V(0);
    if (lane < NR) __hip_atomic_fetch_add((unsigned*)(p.ws[lane] + WS_DONE + rank * 128), 1u, __ATOMIC_RELAXED, __HIP_MEMORY_SCOPE_SYSTEM);
    STAMPW(11);
  };

  bool ready = false;
  if (li < 4) {
    if (wid == 0) {
      const unsigned limit = li < 2 ? (unsigned)NUNITS : (unsigned)PLEAVE;
      send_ops((unsigned)(SHIFT0 + 256) << 12 | (unsigned)(SHIFT0 + 256), (unsigned)b, (unsigned)(b - NAGENT), limit);
      wave_wait<true>(myws + WS_GO, false, call - 1u, tmo, call);
      const unsigned go = read_go();
      if (go >> 31) { progress(seq0); WAIT_V(0); }
      if (lane == 0) __hip_atomic_fetch_add((unsigned*)(myws + WS_DCNT), 1u, __ATOMIC_RELAXED, __HIP_MEMORY_SCOPE_AGENT);
      if (go >> 31) { wave_wait<true>(myws + WS_DCNT, false, call * (unsigned)NAGENT - 1u, tmo, call); join_ops(go, limit); }
      progress(seq0 + (unsigned)(NUNITS + 8));
      STAMP(12);
      if (li < 2) send_out();
    } else if (wid == 2 && li < 2) {
      progress(seq0 + (unsigned)(SEQ - 1));
      wave_wait(myws + WS_SENT, false, seq0 + (unsigned)JOIN2, tmo, call);
      progress(seq0 + (unsigned)JOIN2);
      WAIT_V(0);
      join_ops(read_go(), (unsigned)NUNITS);
      progress(seq0 + (unsigned)(SEQ - 1));
    } else if (wid == 6) {
      wave_wait(myws + WS_SENT, false, seq0 + (unsigned)(PINAT), tmo, call);
      const float mw = pow2f((int)(read_go() & 0xfffu) - 256);
#pragma unroll 1
      for (int d = 1; d < NR; ++d) {
        const int pr = (rank + 1 + (d + b) % (NR - 1)) & 7;
        const __amdgpu_buffer_rsrc_t rp = __builtin_amdgcn_make_buffer_rsrc(p.ws[pr] + stg, 0, STG_BYTES, 0x00020000);
        f32x4 pv[2][8]; unsigned psrc, pdst[2];
#pragma unroll
        for (int e = 0; e < 2; ++e) { place(false, NRC + pr, (b >> 4) * 2 + e, b & 15, psrc, pdst[e]); load_unit(pv[e], true, false, psrc, IC<0>{}); }
#pragma unroll
        for (int e = 0; e < 2; ++e) { u32x4 o[4]; unsigned oo[4]; pack_unit(pv[e], mw, false, pdst[e], o, oo);
#pragma unroll
          for (int i = 0; i < 4; ++i) __builtin_amdgcn_raw_buffer_store_b128(o[i], rp, oo[i], 0, AUX_WT); }
      }
      WAIT_V(0);
      if (lane == 0) __hip_atomic_fetch_add((unsigned*)(myws + WS_PINC), 1u, __ATOMIC_RELAXED, __HIP_MEMORY_SCOPE_AGENT);
    } else if (b == 0 && wid == 7) {
      if (lane < 3) __hip_atomic_store((unsigned*)(myws + WS_TICKET + ((call + 2u) & 3u) * 128u) + lane * 8, 0u, __ATOMIC_RELAXED, __HIP_MEMORY_SCOPE_AGENT);
      unsigned* myflag = (lane & 7) == rank ? (unsigned*)(myws + WS_SENT) : (unsigned*)(p.ws[lane & 7] + WS_FLAG + rank * 128);
      const unsigned long long t0 = __builtin_amdgcn_s_memrealtime();
      { const unsigned st = __hip_atomic_load((unsigned*)(myws + WS_START + (lane & 7) * 128), __ATOMIC_RELAXED, __HIP_MEMORY_SCOPE_SYSTEM);
        const int ahead = __builtin_popcountll(__builtin_amdgcn_ballot_w64(lane < NR && st >= call));
        if (lane == 0) __hip_atomic_store((unsigned*)(myws + WS_LATE), ahead >= LATEN ? call : 0u, __ATOMIC_RELAXED, __HIP_MEMORY_SCOPE_AGENT);
        while (__builtin_amdgcn_s_memrealtime() - t0 < (unsigned long long)LATED) __builtin_amdgcn_s_sleep(2);
        if (lane < NR) __hip_atomic_store((unsigned*)(p.ws[lane] + WS_START + rank * 128), call, __ATOMIC_RELAXED, __HIP_MEMORY_SCOPE_SYSTEM); }
      bool ok = wave_wait<true>(myws + WS_START + (lane & 7) * 128, false, call - 1u, tmo, call);
      if (ok && lane == 0) __hip_atomic_fetch_add(ticket, 1u << 24, __ATOMIC_RELAXED, __HIP_MEMORY_SCOPE_AGENT);
      ok = wave_wait<true>(myws + WS_GO, false, call - 1u, tmo, call) && ok;
      const unsigned redo = read_go() >> 31;
      if (redo) {
        ok = wave_wait<true>(myws + WS_LCNT2, false, call * (unsigned)NRW - 1u, tmo, call) && ok;
        ok = wave_wait<true>(myws + WS_DCNT, false, call * (unsigned)NAGENT - 1u, tmo, call) && ok;
      }
      if (lane == 0) __hip_atomic_store((unsigned*)(myws + WS_FLAG + rank * 128), seq0 + (unsigned)(SEQ - 1), __ATOMIC_RELAXED, __HIP_MEMORY_SCOPE_SYSTEM);
      if (ok && lane == 0) __hip_atomic_fetch_add(ticket + redo * 8, 1u << 25, __ATOMIC_RELAXED, __HIP_MEMORY_SCOPE_AGENT);
      unsigned last = seq0; bool pin = false;
      while (last < seq0 + (unsigned)(NUNITS + 8) || !pin) {
        unsigned m = 0xffffffffu;
#pragma unroll
        for (int j = 0; j < (NPROG + 63) / 64; ++j) { const int a = j * 64 + lane; if (a < NPROG) { const unsigned x = __hip_atomic_load(prog + a, __ATOMIC_RELAXED, __HIP_MEMORY_SCOPE_AGENT); m = x < m ? x : m; } }
        if (!pin && (unsigned)__builtin_amdgcn_readfirstlane(__hip_atomic_load((unsigned*)(myws + WS_PINC), __ATOMIC_RELAXED, __HIP_MEMORY_SCOPE_AGENT)) >= call * 32u) {
          pin = true;
          if (lane < NR) __hip_atomic_store((unsigned*)(p.ws[lane] + WS_PIN + rank * 128), call, __ATOMIC_RELAXED, __HIP_MEMORY_SCOPE_SYSTEM);
        }
#pragma unroll
        for (int sft = 32; sft >= 1; sft >>= 1) { const unsigned x = __shfl_xor(m, sft); m = x < m ? x : m; }
        m = __builtin_amdgcn_readfirstlane(m);
        if (m > last) { last = m; if (lane < NR) __hip_atomic_store(myflag, m, __ATOMIC_RELAXED, __HIP_MEMORY_SCOPE_SYSTEM); }
        if (__builtin_amdgcn_s_memrealtime() - t0 > DEADLINE) { __hip_atomic_store(tmo, call, __ATOMIC_RELAXED, __HIP_MEMORY_SCOPE_AGENT); ok = false; break; }
      }
      ok = wave_wait<true>(myws + WS_DONE + (lane & 7) * 128, false, call * (unsigned)NAGENT - 1u, tmo, call) && ok;
      ok = ok && __hip_atomic_load(tmo, __ATOMIC_RELAXED, __HIP_MEMORY_SCOPE_AGENT) != call;
      if (!ok && lane == 0) __hip_atomic_store((unsigned*)p.out[rank] + (NRC + rank) * 256, 0x7fc00000u, __ATOMIC_RELAXED, __HIP_MEMORY_SCOPE_SYSTEM);
    }
    if (li < 2) return;
  } else {
    const __amdgpu_buffer_rsrc_t rl = __builtin_amdgcn_make_buffer_rsrc(myws + stg, 0, STG_BYTES, 0x00020000);
    { const unsigned long long t0 = __builtin_amdgcn_s_memrealtime(); while (__builtin_amdgcn_s_memrealtime() - t0 < (unsigned long long)(PREDELAY * (li - 3))) __builtin_amdgcn_s_sleep(4); }
    float mx = pow2f(SHIFT0), mw = mx;
    ready = true;
    for (int pass = 0; pass < 2; ++pass) {
      unsigned ax = 0, aw = 0;
      for (int j = b - 32; j < 640; j += NRW) {
        isx = j < 128;
        if (j < 128) place(true, 2 * (j & 7) + (wid >> 2), wid & 3, j >> 3, src, dst);
        else if (j < 576) place(false, 2 * ((j - 128) % 28) + (wid >> 2), wid & 3, (j - 128) / 28, src, dst);
        else place(false, NRC + ((j - 576) >> 3), wid & 3, 2 * (j & 7) + (wid >> 2), src, dst);
        if (PRENT && j >= 128 && j < 576 && (j - 128) % 28 >= 7) load_unit(v, true, isx, src, IC<2>{}); else load_unit(v, true, isx, src, IC<0>{});
        unsigned am = 0;
#pragma unroll
        for (int q = 0; q < 32; ++q) { unsigned a = __builtin_bit_cast(unsigned, v[q >> 2][q & 3]) & 0x7fffffffu; a = a < 0x7f800000u ? a : 0u; am = a > am ? a : am; }
        if (isx) ax = am > ax ? am : ax; else aw = am > aw ? am : aw;
        if (j < 576 || ((j - 576) >> 3) == rank) {
          u32x4 o[4]; unsigned oo[4]; pack_unit(v, isx ? mx : mw, isx, dst, o, oo);
#pragma unroll
          for (int i = 0; i < 4; ++i) __builtin_amdgcn_raw_buffer_store_b128(o[i], rl, oo[i], 0, AUX_WT);
        }
      }
      if (pass == 0) {
#pragma unroll
        for (int sft = 32; sft >= 1; sft >>= 1) { const unsigned a = __shfl_xor(ax, sft), c = __shfl_xor(aw, sft); ax = a > ax ? a : ax; aw = c > aw ? c : aw; }
        if (lane == 0) { *(LAS unsigned*)(lds + LDS_BAD + 32 + wid * 8) = ax; *(LAS unsigned*)(lds + LDS_BAD + 36 + wid * 8) = aw; }
      }
      WAIT_V(0); WAIT_L(0); BAR; asm volatile("" ::: "memory");
      if (pass == 1) break;
      if (wid == 0) {
        if (lane < 2) {
          unsigned a = 0;
#pragma unroll
          for (int q = 0; q < 8; ++q) { const unsigned c = *(LAS unsigned*)(lds + LDS_BAD + 32 + q * 8 + lane * 4); a = c > a ? c : a; }
          __hip_atomic_fetch_max((unsigned long long*)(myws + WS_LMAX) + lane, (unsigned long long)call << 32 | a, __ATOMIC_RELAXED, __HIP_MEMORY_SCOPE_AGENT);
        }
        WAIT_V(0);
        if ((unsigned)__builtin_amdgcn_readfirstlane(draw((unsigned*)(myws + WS_LCNT), true)) == call * (unsigned)NRW - 1u) {
          const unsigned long long mm = __hip_atomic_load((unsigned long long*)(myws + WS_LMAX) + (lane & 1), __ATOMIC_RELAXED, __HIP_MEMORY_SCOPE_AGENT);
          const unsigned e1 = (unsigned)mm ? 1u + ((unsigned)mm >> 23) : 0u;
          const unsigned e12 = (unsigned)__builtin_amdgcn_readlane(e1, 0) << 16 | (unsigned)__builtin_amdgcn_readlane(e1, 1);
          if (lane < NR) __hip_atomic_store((unsigned long long*)(p.ws[lane] + WS_AMX + rank * 128), (unsigned long long)e12 << 32 | call, __ATOMIC_RELAXED, __HIP_MEMORY_SCOPE_SYSTEM);
          wave_wait<true>(myws + WS_AMX + (lane & 7) * 128, false, call - 1u, tmo, call);
          const unsigned e2 = __hip_atomic_load((unsigned*)(myws + WS_AMX + (lane & 7) * 128 + 4), __ATOMIC_RELAXED, __HIP_MEMORY_SCOPE_SYSTEM);
          unsigned gx = e2 >> 16, gw = e2 & 0xffffu;
#pragma unroll
          for (int sft = 4; sft >= 1; sft >>= 1) { const unsigned a = __shfl_xor(gx, sft), c = __shfl_xor(gw, sft); gx = a > gx ? a : gx; gw = c > gw ? c : gw; }
          auto shift_of = [](unsigned g) INL -> int { const int e = (int)g - 128; return (g == 0u || (e >= -SHIFT0 && e < 15 - SHIFT0)) ? SHIFT0 : (14 - e > 126 ? 126 : 14 - e); };
          const int sx = shift_of(__builtin_amdgcn_readfirstlane(gx)), sw = shift_of(__builtin_amdgcn_readfirstlane(gw));
          const bool redo = sx != SHIFT0 || sw != SHIFT0;
          const bool late = __hip_atomic_load((unsigned*)(myws + WS_LATE), __ATOMIC_RELAXED, __HIP_MEMORY_SCOPE_AGENT) == call;
          if (lane == 0) {
            __hip_atomic_store((unsigned long long*)(myws + WS_GO), (unsigned long long)((redo ? 0x80000000u : 0u) | (late ? 0x40000000u : 0u) | (unsigned)(sx + 256) << 12 | (unsigned)(sw + 256)) << 32 | call, __ATOMIC_RELAXED, __HIP_MEMORY_SCOPE_AGENT);
            if (redo) __hip_atomic_fetch_add(ticket, 1u << 20, __ATOMIC_RELAXED, __HIP_MEMORY_SCOPE_AGENT);
          }
        }
        if (!wave_wait(myws + WS_GO, false, call - 1u, tmo, call)) *(LAS unsigned*)(lds + LDS_BAD) = 1u;
        *(LAS unsigned*)(lds + LDS_BAD + 8) = read_go();
        WAIT_L(0);
      }
      asm volatile("" ::: "memory"); BAR; asm volatile("" ::: "memory");
      const unsigned go = __builtin_amdgcn_readfirstlane(*(LAS unsigned*)(lds + LDS_BAD + 8));
      if (!(go >> 31)) break;
      mx = pow2f((int)((go >> 12) & 0xfffu) - 256); mw = pow2f((int)(go & 0xfffu) - 256); ready = false;
    }
    if (tid == 0) __hip_atomic_fetch_add((unsigned*)(myws + WS_LCNT2), 1u, __ATOMIC_RELAXED, __HIP_MEMORY_SCOPE_AGENT);
    STAMP(13);
  }

  const bool isP = li < 4;
  const int wr = wid >> 2, wc = wid & 3;
  const int lr = li - 4;
  const int mt = isP ? xcd * 2 + (li - 2) : (xcd >> 1) * 4 + (lr & 3);
  const int cr = isP ? 0 : (xcd & 1) + 2 * (lr >> 2);
  const unsigned voff = (unsigned)(wid * 2048 + lane * 16), ldsw = (unsigned)wid * 2048u;
  const char* fsrc = myws + WS_FLAG + wid * 128;
  const __amdgpu_buffer_rsrc_t ra = __builtin_amdgcn_make_buffer_rsrc(myws + stg, 0, A_BYTES, 0x00020000);
  const __amdgpu_buffer_rsrc_t rb = __builtin_amdgcn_make_buffer_rsrc(myws + stg + A_BYTES + (isP ? B_BYTES : 0u), 0, isP ? P_BYTES : B_BYTES, 0x00020000);
  const unsigned bstep = isP ? (unsigned)PIECE : B_STEP;
#define STAGE(ldsoff, rs, so) do { \
    __builtin_amdgcn_raw_ptr_buffer_load_lds(rs, (LAS void*)(lds + (ldsoff) + ldsw), 16, voff, (so), 0, LDAUX); \
    __builtin_amdgcn_raw_ptr_buffer_load_lds(rs, (LAS void*)(lds + (ldsoff) + ldsw), 16, voff, (so), 1024, LDAUX); } while (0)
#define ST1(ldsoff, rs, so, imm) do { __builtin_amdgcn_raw_ptr_buffer_load_lds(rs, (LAS void*)(lds + (ldsoff) + ldsw), 16, voff, (so), imm, LDAUX); SCHED; } while (0)
#define RA(h, m) do { _Pragma("unroll") for (int k = 0; k < 2; ++k) At[m][k] = *(const LAS f16x8*)((LAS char*)(size_t)va + (h) * HTB + (m) * FRS + k * KHS); SCHED; } while (0)
#define FLAGDMA() __builtin_amdgcn_global_load_lds((const unsigned*)fsrc, (LAS unsigned*)(lds + LDS_FLAG + wid * 256), 4, 0, FAUX)
#define LDA(dst, h) do { _Pragma("unroll") for (int m = 0; m < 4; ++m) _Pragma("unroll") for (int k = 0; k < 2; ++k) dst[m][k] = *(const LAS f16x8*)((LAS char*)(size_t)va + (h) * HTB + m * FRS + k * KHS); } while (0)
#define LDB(dst, h) do { _Pragma("unroll") for (int n = 0; n < 2; ++n) _Pragma("unroll") for (int k = 0; k < 2; ++k) dst[n][k] = *(const LAS f16x8*)((LAS char*)(size_t)vb + (h) * HTB + n * FRS + k * KHS); } while (0)
#define ACC(ai, nn, m) (128 + (((ai) * 4 + (nn)) * 4 + (m)) * 4)
#define MMA2(ai) do { MMA2_(ai); BAR; SCHED; } while (0)
#define MMA2_(ai) do { __builtin_amdgcn_s_waitcnt(0xC07F); SCHED; BAR; SCHED; __builtin_amdgcn_s_setprio(1); SCHED; \
    _Pragma("unroll") for (int m = 0; m < 4; ++m) _Pragma("unroll") for (int q = 0; q < 4; ++q) { const int nn = (m & 1) ? 3 - q : q; \
      asm volatile("v_mfma_f32_16x16x32_f16 v[%c0:%c1], %2, %3, v[%c0:%c1]\n\tv_mfma_f32_16x16x32_f16 v[%c0:%c1], %4, %5, v[%c0:%c1]" \
          :: "i"(ACC(ai, nn, m)), "i"(ACC(ai, nn, m) + 3), "v"(At[m][0]), "v"((nn >> 1) ? B1[nn & 1][0] : B0[nn & 1][0]), "v"(At[m][1]), "v"((nn >> 1) ? B1[nn & 1][1] : B0[nn & 1][1])); } \
    SCHED; __builtin_amdgcn_s_setprio(0); SCHED; } while (0)
#define ZERO_ACC() do { _Pragma("unroll") for (int i_ = 128; i_ < 256; i_ += 2) asm volatile("v_mov_b64 v[%c0:%c1], 0" :: "i"(i_), "i"(i_ + 1)); } while (0)
#define ACC_ROW(dst, ai, m, j) do { float t0_, t1_, t2_, t3_; \
    asm volatile("v_ldexp_f32 %0, v[%c4], %8\n\tv_ldexp_f32 %1, v[%c5], %8\n\tv_ldexp_f32 %2, v[%c6], %8\n\tv_ldexp_f32 %3, v[%c7], %8" \
        : "=&v"(t0_), "=&v"(t1_), "=&v"(t2_), "=&v"(t3_) : "i"(ACC(ai, 0, m) + (j)), "i"(ACC(ai, 1, m) + (j)), "i"(ACC(ai, 2, m) + (j)), "i"(ACC(ai, 3, m) + (j)), "v"(oexp)); \
    dst = (f32x4){t0_, t1_, t2_, t3_}; } while (0)
  asm volatile("" ::: "v128","v129","v130","v131","v132","v133","v134","v135","v136","v137","v138","v139","v140","v141","v142","v143","v144","v145","v146","v147","v148","v149","v150","v151","v152","v153","v154","v155","v156","v157","v158","v159","v160","v161","v162","v163","v164","v165","v166","v167","v168","v169","v170","v171","v172","v173","v174","v175","v176","v177","v178","v179","v180","v181","v182","v183","v184","v185","v186","v187","v188","v189","v190","v191","v192","v193","v194","v195","v196","v197","v198","v199","v200","v201","v202","v203","v204","v205","v206","v207","v208","v209","v210","v211","v212","v213","v214","v215","v216","v217","v218","v219","v220","v221","v222","v223","v224","v225","v226","v227","v228","v229","v230","v231","v232","v233","v234","v235","v236","v237","v238","v239","v240","v241","v242","v243","v244","v245","v246","v247","v248","v249","v250","v251","v252","v253","v254","v255");
  ZERO_ACC();
  f16x8 At[4][2], B0[2][2], B1[2][2];
  const int ph = isP ? 0 : phase_of(cr);
  if (!ready) {
    if (wid == 0) {
      if (isP && !wave_wait(myws + WS_PIN + (lane & 7) * 128, false, call - 1u, tmo, call)) *(LAS unsigned*)(lds + LDS_BAD) = 1u;
      if (!wave_wait(isP ? flagw : myws + WS_FLAG + rank * 128, false, seq0 + (unsigned)(U1 - 1), tmo, call)) *(LAS unsigned*)(lds + LDS_BAD) = 1u;
      *(LAS unsigned*)(lds + LDS_BAD + 8) = read_go(); WAIT_L(0);
    }
    asm volatile("" ::: "memory"); BAR; asm volatile("" ::: "memory");
  }
  const unsigned gow = __builtin_amdgcn_readfirstlane(*(LAS unsigned*)(lds + LDS_BAD + 8));
  const int oexp = 512 - (int)((gow >> 12) & 0xfffu) - (int)(gow & 0xfffu);
  const int NL = (gow >> 30) & 1u ? 0 : NLV;
  const int crx = XSH ? cr ^ 1 : cr;
  const int E0 = 8 + rho0_of(cr);
  const int Tend = isP ? 64 : E0 + 192 + 56 - rho0_of(crx);
  unsigned tA, tB, tN;
  auto steps = [&](int t0) INL {
    const int t = t0 + lane, u = t - E0, c = u < 192 ? cr : crx, w = u - 8 * NL - 8, z = w - 56 * (1 + NL);
    const bool inl = z >= 0 && u < 192;
    const bool local = t < 8 || (u >= 0 && w < 0) || (inl && (z & 7) == 0);
    const int rho = u < 0 ? t - 8 : u >= 192 ? rho0_of(crx) + u - 24 : z < 0 ? E0 - 8 + w : E0 - 8 + w - (z >> 3) - 1, pos = rho % 7;
    const int sgm = !local ? rho / 7 : t < 8 ? (t < ph ? t : t + 24) : inl ? ph + 8 * (1 + NL) + (z >> 3) : u < 8 * NL ? ph + 8 + u : ph + (u & 7);
    const int f = first_src(c), sr = local ? rank : (f + pos + (pos >= ((rank - f) & 7) ? 1 : 0)) & 7;
    const unsigned ix = isP ? (unsigned)t : (unsigned)(sgm * 8 + sr);
    tA = t < Tend ? (unsigned)mt * PIECE + (ix & 63u) * A_STEP + (odd && (ix & 63u) < (unsigned)NDUP ? dup_of(true) : 0u) : 0x80000000u;
    tB = t < Tend ? (unsigned)c * PIECE + ix * bstep + (odd && !isP && ix < (unsigned)NDUP ? dup_of(false) : 0u) : 0x80000000u;
    tN = (isP || t >= Tend || local) ? 0u : seq0 + (unsigned)((sgm < 8 ? 240 * sgm + 128 : U1 - 8 * 112 + 112 * sgm) + 8 * ((c + rot(sr)) % RCOLS) + 8 + sr);
  };
  steps(0);
  STAMP(1);
  __builtin_amdgcn_s_waitcnt(0x0F70); SCHED;
  unsigned par = 0, l1 = PIECE + HTB, a1, fl = 0;
  unsigned a2 = __builtin_amdgcn_readlane(tA, 2), b2 = __builtin_amdgcn_readlane(tB, 2), need = 0;
  unsigned va = (unsigned)(size_t)lds + wr * 4 * FRS + lane * 16, vb = (unsigned)(size_t)lds + 2 * PIECE + wc * 2 * FRS + lane * 16;
  LAS unsigned* fimg = (LAS unsigned*)(lds + LDS_FLAG + (lane & 7) * 256); asm volatile("" : "+v"(fimg));
  { const unsigned a0 = __builtin_amdgcn_readlane(tA, 0), b0 = __builtin_amdgcn_readlane(tB, 0), b1 = __builtin_amdgcn_readlane(tB, 1); a1 = __builtin_amdgcn_readlane(tA, 1);
    STAGE(2 * PIECE, rb, b0); STAGE(0, ra, a0); STAGE(2 * PIECE + HTB, rb, b0 + HTB); STAGE(HTB, ra, a0 + HTB);
    if (wr == 1) BAR;
    WAIT_V(2); BAR;
    FLAGDMA();
    STAGE(3 * PIECE, rb, b1); STAGE(PIECE, ra, a1); STAGE(3 * PIECE + HTB, rb, b1 + HTB);
    a1 += HTB;
    BAR; }
  const unsigned vpk = (unsigned)tid * 16u;
  const unsigned vring = (unsigned)(size_t)lds + LDS_RING + wid * 1024 + lane * 16;
#define SLOW_T0
#define SLOW_T1
#define STEP_STAMP
#define KLOOP(ADD, LIM) for (;;) { \
      f32x4 pt; \
      ST1(l1, ra, a1, 0); LDB(B0, 0); SCHED; ST1(l1, ra, a1, 1024); LDB(B1, 1); SCHED; \
      if (ADD) { __builtin_amdgcn_raw_ptr_buffer_load_lds(rpk, (LAS void*)(lds + LDS_RING + rs_ + wid * 1024), 16, vpk, (unsigned)(li3 - a0_) * 8192u, 0, PDAUX); SCHED; \
        rs_ = rs_ == 16384u ? 0u : rs_ + 8192u; pt = *(const LAS f32x4*)(size_t)(vring + rs_); } \
      else FLAGDMA(); \
      SCHED; LDA(At, 0); WAIT_V(10); \
      MMA2(0); \
      if ((unsigned)__builtin_amdgcn_readlane(fl, need & 63u) < need) { SLOW_T0; \
        const unsigned f2 = *(const volatile LAS unsigned*)fimg; fl = f2 > fl ? f2 : fl;     \
        if ((unsigned)__builtin_amdgcn_readlane(fl, need & 63u) < need) { \
          if (!wave_wait(flagw, (unsigned)(lane & 7) != (need & 7u), need - 1u, tmo, call, &fl)) *(LAS unsigned*)(lds + LDS_BAD) = 1u; \
          *(volatile LAS unsigned*)fimg = fl; } \
        SLOW_T1; } \
      ST1(2 * PIECE + par, rb, b2, 0); RA(1, 0); ST1(2 * PIECE + par, rb, b2, 1024); RA(1, 1); \
      ST1(par, ra, a2, 0); RA(1, 2); ST1(par, ra, a2, 1024); RA(1, 3); \
      const unsigned fn = ADD ? 0u : *fimg; \
      if (ADD && li3 - a0_ >= 2) { unsigned keep_; \
        asm volatile("s_mov_b32 %0, m0\n\ts_set_gpr_idx_on %1, gpr_idx(SRC0,DST)\n\tv_add_f32 v128, v128, %2\n\tv_add_f32 v129, v129, %3\n\tv_add_f32 v130, v130, %4\n\tv_add_f32 v131, v131, %5\n\ts_set_gpr_idx_off\n\ts_mov_b32 m0, %0" \
            : "=&s"(keep_) : "s"(((li3 - a0_ - 2) & 31) * 4), "v"(pt[0]), "v"(pt[1]), "v"(pt[2]), "v"(pt[3])); } \
      SCHED; \
      ST1(2 * PIECE + par + HTB, rb, b2 + HTB, 0); \
      l1 = par + HTB; a1 = a2 + HTB; va ^= PIECE; vb ^= PIECE; \
      a2 = __builtin_amdgcn_readlane(tA, li3); need = __builtin_amdgcn_readlane(tN, li3); \
      asm volatile("" : "+v"(va), "+v"(vb), "+s"(a2), "+s"(need), "+s"(l1), "+s"(a1)); SCHED; \
      ST1(2 * PIECE + par + HTB, rb, b2 + HTB, 1024); \
      par ^= PIECE; b2 = __builtin_amdgcn_readlane(tB, li3); ++li3; \
      STEP_STAMP \
      fl = fn > fl ? fn : fl; \
      asm volatile("" : "+s"(par), "+s"(li3), "+s"(b2), "+v"(fl)); \
      WAIT_V(9); \
      MMA2_(1); \
      if (li3 >= (LIM)) break; \
      BAR; SCHED; }
  int t = 0;
  steps(3);
  float* const myout = p.out[rank] + (size_t)(mt * 256 + wr * 64) * GN + wc * 64;
  for (int sg = isP ? 4 : -NL; sg < 5; ++sg) {
    const int Ts = sg == 4 ? Tend : sg == -NL ? E0 : t + (sg <= 0 ? 8 : (sg >= 2 && sg < 2 + NL) ? 56 : 64);
    const int slot = sg <= 0 ? sg + NL : (sg == 4 ? 0 : sg - 1);
    char* pk = myws + WS_PARK + (size_t)(slot * 256 + (XSH && sg == 4 ? b ^ 1 : b)) * 262144;
    const __amdgpu_buffer_rsrc_t rpk = __builtin_amdgcn_make_buffer_rsrc(pk, 0, 262144, 0x00020000);
    const int left = Ts - t; int li3 = 0;
    unsigned rs_ = 0;
    const bool add = !isP && (sg == 4 || (sg >= 2 && sg < 2 + NL));
    const int a1_ = (7 * (HS + 1) + 1) / 2 + 8 - E0 - (sg == 4 ? ADDOFF : ADDOFF2) - (xcd & 1) * ADDODD, a0_ = (add && a1_ > 0) ? (a1_ < left - 35 ? a1_ : left - 35) : 0;
    for (int part = a0_ ? 0 : 1; part < 2; ++part) {
      if (part == 1 && add) { KLOOP(true, a0_ + 34); BAR; SCHED; }
      const int lim_ = part ? left : a0_;
      KLOOP(false, lim_);
      if (!part) { BAR; SCHED; }
    }
    t = Ts;
    if (wr == 0) BAR;
    asm volatile("s_nop 15\n\ts_nop 15" ::: "memory");
    SCHED;
    STAMP(2 + 2 * r);
    if (sg <= 0) {
      unsigned vq = vpk; asm volatile("" : "+v"(vq));
      if (sg == -NL) {
#pragma unroll
      for (int i = 0; i < 32; ++i)
        asm volatile("s_nop 4\n\tglobal_store_dwordx4 %0, v[%c1:%c2], %3" PKAUX :: "v"(vq + i * 8192u), "i"(128 + 4 * i), "i"(128 + 4 * i + 3), "s"(pk) : "memory");
      } else {
#pragma unroll
      for (int i = 0; i < 32; ++i)
        asm volatile("s_nop 4\n\tglobal_store_dwordx4 %0, v[%c1:%c2], %3" PK2AUX :: "v"(vq + i * 8192u), "i"(128 + 4 * i), "i"(128 + 4 * i + 3), "s"(pk) : "memory");
      }
      asm volatile("s_nop 1");
      ZERO_ACC();
      steps(t + 3);
      if (XSH && sg == -NL) {
        WAIT_V(0);
        if (lane == 0) __hip_atomic_store((unsigned*)(myws + WS_PKF) + (size_t)(b * 8 + wid) * 2, call, __ATOMIC_RELAXED, __HIP_MEMORY_SCOPE_AGENT);
        if (XSH == 2 && !wave_wait(myws + WS_PKF + (size_t)((b ^ 1) * 8 + wid) * 8, false, call - 1u, tmo, call)) *(LAS unsigned*)(lds + LDS_BAD) = 1u;
      }
    } else {
      if (XSH == 1 && sg == 1 && !wave_wait(myws + WS_PKF + (size_t)((b ^ 1) * 8 + wid) * 8, false, call - 1u, tmo, call)) *(LAS unsigned*)(lds + LDS_BAD) = 1u;
      if (__builtin_amdgcn_readfirstlane(*(const LAS unsigned*)(lds + LDS_BAD)) != 0u) {
#pragma unroll
        for (int i = 128; i < 256; ++i) asm volatile("v_mov_b32 v[%c0], 0x7fc00000" :: "i"(i));
      }
      const int nt = isP ? NRC + rank : sg == 4 ? crx : sg * RCOLS + cr;
      const __amdgpu_buffer_rsrc_t rc = __builtin_amdgcn_make_buffer_rsrc(myout + nt * 256, 0, 192u * GN * 4u, 0x00020000);
      unsigned vo = (unsigned)((lane >> 4) * 4 * GN + (lane & 15) * 4) * 4u; asm volatile("" : "+v"(vo));
#pragma unroll
      for (int ai = 0; ai < 2; ++ai)
#pragma unroll
        for (int m = 0; m < 4; ++m) {
#pragma unroll
          for (int j = 0; j < 4; ++j) {
            f32x4 v4; ACC_ROW(v4, ai, m, j);
            __builtin_amdgcn_raw_buffer_store_b128(__builtin_bit_cast(u32x4, v4), rc, vo + (unsigned)((ai * 128 + m * 16 + j) * GN * 4), 0, AUX_NT);
          }
#pragma unroll
          for (int nn = 0; nn < 4; ++nn)
            asm volatile("v_mov_b64 v[%c0:%c1], 0\n\tv_mov_b64 v[%c2:%c3], 0" :: "i"(ACC(ai, nn, m)), "i"(ACC(ai, nn, m) + 1), "i"(ACC(ai, nn, m) + 2), "i"(ACC(ai, nn, m) + 3));
          if (ai == 0 && m == 1) {
            SCHED; steps(t + 3); asm volatile("" : "+v"(tA), "+v"(tB), "+v"(tN)); SCHED;
          }
        }
    }
    if (wr == 1) BAR;
    STAMP(3 + 2 * r);
    STAMPC(20 + r);
  }
  WAIT_V(0);
  STAMP(15);
  if (wr == 0) BAR;
  if (isP) {
    BAR; asm volatile("" ::: "memory");
    if (tid == 0) {
      __builtin_amdgcn_fence(__ATOMIC_RELEASE, "agent");
      WAIT_V(0);
      __hip_atomic_fetch_add((unsigned*)(myws + WS_PCNT), 1u, __ATOMIC_RELAXED, __HIP_MEMORY_SCOPE_AGENT);
    }
    STAMP(10);
    if (wid == 0) send_out();
  }
}
}

extern "C" void kernel_launch(int rank, int nranks, ncclComm_t comm, hipStream_t stream,
                              void* const* d_in, const int* in_sizes, int n_in,
                              void* d_out, int out_size, const int* in_dtypes, int out_dtype,
                              void* const* d_out_all, void* const* d_ws_all, size_t ws_bytes) {
  (void)comm; (void)in_dtypes; (void)out_dtype; (void)d_out;
  if (nranks != NR || (unsigned)rank >= (unsigned)NR || n_in < 2 || ws_bytes < WS_NEED || out_size != GM * GN ||
      in_sizes[0] != GM * KSH || in_sizes[1] != KSH * GN) { fprintf(stderr, "unexpected problem shape\n"); return; }
  static bool attr_set = false;
  if (!attr_set) { HIP_CHECK(hipFuncSetAttribute((const void*)fused_kernel, hipFuncAttributeMaxDynamicSharedMemorySize, LDS_BYTES)); attr_set = true; }
  Params p;
  p.x = (const float*)d_in[0]; p.w = (const float*)d_in[1]; p.rank = rank;
  for (int r = 0; r < NR; ++r) { p.out[r] = (float*)d_out_all[r]; p.ws[r] = (char*)d_ws_all[r]; }
  fused_kernel<<<256, 512, LDS_BYTES, stream>>>(p);
  HIP_CHECK(hipGetLastError());
}
```

```cpp
#include <hip/hip_runtime.h>
#include <rccl/rccl.h>
#include <cstdio>

#define HIP_CHECK(expr) do { hipError_t _e = (expr); if (_e != hipSuccess) { \
    fprintf(stderr, "HIP error %s at %s:%d\n", hipGetErrorString(_e), __FILE__, __LINE__); \
    return; } } while (0)

namespace {
#define LAS __attribute__((address_space(3)))
#define INL __attribute__((always_inline))
typedef _Float16 f16x8 __attribute__((ext_vector_type(8)));
typedef float f32x4 __attribute__((ext_vector_type(4)));
typedef unsigned u32x4 __attribute__((ext_vector_type(4)));

constexpr int NR = 8, GM = 4096, GN = 16384, KSH = 512;
constexpr int PIECE = 32768, HTB = 16384, NKS = 64;
constexpr int FRS = 2048, KHS = 1024;
constexpr int RCOLS = 14, ROUNDS = 4, NRC = RCOLS * ROUNDS;
constexpr unsigned A_STEP = 16u * PIECE, B_STEP = (unsigned)RCOLS * PIECE;
constexpr int NDUP = 16;
constexpr unsigned A_BYTES = (NKS + NDUP) * A_STEP, B_BYTES = (unsigned)(ROUNDS * NKS + NDUP) * B_STEP, P_BYTES = (unsigned)NKS * PIECE;
constexpr unsigned STG_BYTES = A_BYTES + B_BYTES + P_BYTES;
constexpr size_t WS_CTR = 0;
constexpr size_t WS_FLAG = 16384, WS_START = 18432, WS_PIN = 24576, WS_AMX = 28672, WS_DONE = 29696;
constexpr size_t WS_TMO = 20480, WS_TICKET = 21504, WS_PROG = 22528, WS_LCNT = 26624, WS_SENT = 26752, WS_PCNT = 26880, WS_PINC = 27008,
                 WS_LCNT2 = 27136, WS_DCNT = 27264, WS_LMAX = 27392, WS_GO = 27520, WS_LATE = 27648, WS_MAP = 27776;
constexpr size_t WS_PKC = 30720, WS_SPD = 31744;
constexpr size_t WS_DBG = 65536, WS_STG = 1u << 20;
constexpr size_t WS_PARK = WS_STG + (size_t)STG_BYTES;
constexpr size_t WS_NEED = WS_PARK + 2 * 256 * (size_t)262144;
#define PINAT 2200
#define PLEAVE1 2600
#define PLEAVE 3150
#define JOIN2 2600
#define PSTART 200
#define PREDELAY 40
#define ADDOFF 8
#define ADDOFF2 4
#define SWAPT 1012u
constexpr int POLLS = 32, LATEN = 4, LATED = 300;
constexpr int U1 = 8 * 240, NUNITS = U1 + 24 * 112, NOUT = GM / 4, SEQ = 8192, NAGENT = 32, NPROG = 48, NRW = 224;
constexpr int SHIFT0 = 8;
constexpr int LDS_FLAG = 131072, LDS_BAD = LDS_FLAG + 2048, LDS_RING = LDS_BAD + 128, LDS_BYTES = LDS_RING + 3 * 8192;
constexpr unsigned long long DEADLINE = 80000000ull;
constexpr int AUX_WT = 17, AUX_NT = 2, AUX_SC1 = 16;

template <int V> struct IC { static constexpr int value = V; };
struct Params { const float* x; const float* w; float* out[NR]; char* ws[NR]; int rank; };

#define WAIT_V(n) asm volatile("s_waitcnt vmcnt(" #n ")" ::: "memory")
#define WAIT_L(n) asm volatile("s_waitcnt lgkmcnt(" #n ")" ::: "memory")
#define BAR __builtin_amdgcn_s_barrier()
#define SCHED __builtin_amdgcn_sched_barrier(0)
#define STAMPC(i) do {} while (0)
#define STAMPW(i) do {} while (0)
#define STAMP(i) do {} while (0)

template <bool FAST = false>
__device__ __forceinline__ bool wave_wait(const char* words, bool skip, unsigned need, unsigned* tmo, unsigned call, unsigned* seen = nullptr) {
  const unsigned long long t0 = __builtin_amdgcn_s_memrealtime();
  bool ok = true;
  for (unsigned n = 1;; ++n) {
    const unsigned v = (n & 7u) ? __hip_atomic_load((const unsigned*)words, __ATOMIC_RELAXED, __HIP_MEMORY_SCOPE_AGENT)
                                : __hip_atomic_load((const unsigned*)words, __ATOMIC_RELAXED, __HIP_MEMORY_SCOPE_SYSTEM);
    if (seen) *seen = v;
    if (__builtin_amdgcn_ballot_w64(skip || v > need) == ~0ull) break;
    if ((n & 7u) == 0u &&
        ((unsigned)__builtin_amdgcn_readfirstlane(__hip_atomic_load(tmo, __ATOMIC_RELAXED, __HIP_MEMORY_SCOPE_AGENT)) == call ||
         __builtin_amdgcn_s_memrealtime() - t0 > DEADLINE)) {
      __hip_atomic_store(tmo, call, __ATOMIC_RELAXED, __HIP_MEMORY_SCOPE_AGENT);
      ok = false; break;
    }
    __builtin_amdgcn_s_sleep(FAST ? 1 : POLLS);
  }
  asm volatile("" ::: "memory");
  return ok;
}

__device__ __forceinline__ float pow2f(int e) { return __builtin_bit_cast(float, (unsigned)(e + 127) << 23); }
__device__ __forceinline__ u32x4 pack8(float m, float a0, float a1, float a2, float a3, float a4, float a5, float a6, float a7) {
  f16x8 h = {(_Float16)(a0 * m), (_Float16)(a1 * m), (_Float16)(a2 * m), (_Float16)(a3 * m), (_Float16)(a4 * m), (_Float16)(a5 * m), (_Float16)(a6 * m), (_Float16)(a7 * m)};
  return __builtin_bit_cast(u32x4, h);
}

__device__ __forceinline__ void load_send_(f32x4 (&v)[8], bool valid, bool copy, bool isx, unsigned src, unsigned dst,
                                           const void* stg, const void* xp, const void* wp, int lane) {
  const __amdgpu_buffer_rsrc_t rs = __builtin_amdgcn_make_buffer_rsrc((void*)(copy ? stg : isx ? xp : wp), 0,
                                                                      !valid ? 0u : copy ? STG_BYTES : isx ? GM * KSH * 4 : KSH * GN * 4, 0x00020000);
  const unsigned vv = copy ? (unsigned)lane * 16u : isx ? (unsigned)((lane & 15) * KSH + 4 * (lane >> 4)) * 4u : (unsigned)(4 * (lane >> 4) * GN + 4 * (lane & 15)) * 4u;
#pragma unroll
  for (int j = 0; j < 8; ++j) {
    const unsigned so = copy ? (j < 4 ? dst + (unsigned)((j >> 1) * HTB + (j & 1) * FRS) : 0x80000000u)
                             : src + (isx ? (unsigned)((j >> 1) * 16 * KSH + (j & 1) * 16) * 4u : (unsigned)(((j >> 2) * 16 + (j & 3)) * GN) * 4u);
    v[j] = __builtin_bit_cast(f32x4, __builtin_amdgcn_raw_buffer_load_b128(rs, vv, so, 16));
  }
}

#define SEND_UNIT(NPRE, REPORT) do { \
    _Pragma("unroll") for (int i = 0; i < 4; ++i) { \
      if (i == 2) { asm volatile("s_waitcnt vmcnt(%0)" :: "i"(14 + (NPRE)) : "memory"); if (REPORT) progress(seq0 + cur); } \
      SCHED; \
      _Pragma("unroll") for (int d = 0; d < NR - 1; ++d) __builtin_amdgcn_raw_buffer_store_b128(o[i], rd[d], oo[i], 0, AUX_WT); \
      SCHED; } } while (0)

__global__ void __launch_bounds__(512, 2) fused_kernel(Params p) {
  extern __shared__ __attribute__((aligned(16))) char smem[];
  LAS char* lds = (LAS char*)smem;
  const int tid = threadIdx.x, wid = __builtin_amdgcn_readfirstlane(tid >> 6), lane = tid & 63;
  const int b = blockIdx.x, xcd = b & 7, li = b >> 3, rank = p.rank;
  char* myws = p.ws[rank];
  unsigned* tmo = (unsigned*)(myws + WS_TMO);
  STAMP(0);
  const int g4 = lane >> 4, r16 = lane & 15;
  const unsigned vx = (unsigned)(r16 * KSH + 4 * g4) * 4u, vw = (unsigned)(4 * g4 * GN + 4 * r16) * 4u, vd = (unsigned)lane * 16u;
  auto rot = [](int r) INL -> int { return (r * 7) >> 2; };
  auto first_src = [](int cr) INL -> int { return cr > 1 ? (4 * (14 - cr) + 6) / 7 : 0; };
  auto half_of = [](int cr) INL -> int { return ((cr >> 1) < 5 ? cr >> 1 : 5) + 1 - (cr & 1); };
  auto early = [&](int cr) INL -> bool { return ((rank - first_src(cr)) & 7) < 4; };
  auto phase_of = [&](int cr) INL -> int { return (half_of(cr) >> 1) + ((half_of(cr) & 1) && early(cr) ? 1 : 0); };
  auto rho0_of = [&](int cr) INL -> int { return 7 * (half_of(cr) >> 1) + ((half_of(cr) & 1) ? (early(cr) ? 3 : 4) : 0); };
  unsigned odd = 0;
  auto dup_of = [](bool isx) INL -> unsigned { return isx ? NKS * A_STEP : (unsigned)(ROUNDS * NKS) * B_STEP; };
  auto place = [&](bool isx, int col, int q4, int hs, unsigned& src, unsigned& dst) INL {
    const unsigned ks = (unsigned)((hs >> 1) * 8 + rank), kk = (unsigned)((hs & 1) * KHS);
    if (isx) { src = (unsigned)((col * 256 + q4 * 64) * KSH + hs * 32) * 4u; dst = ks * A_STEP + (unsigned)col * PIECE + kk + (unsigned)((q4 >> 1) * HTB + (q4 & 1) * 4 * FRS); }
    else {
      const int rr = col / RCOLS, cc = col - rr * RCOLS;
      const int wcol = col < NRC ? ((rr + ((hs >> 1) >= phase_of(cc) ? 1 : 0)) & 3) * RCOLS + cc : col;
      src = (unsigned)(hs * 32 * GN + wcol * 256 + q4 * 64) * 4u;
      dst = A_BYTES + (col < NRC ? ((unsigned)rr * NKS + ks) * B_STEP + (unsigned)cc * PIECE : B_BYTES + ks * PIECE) + kk + (unsigned)(q4 * 2 * FRS);
    }
    if (odd && ks < (unsigned)NDUP && (isx || col < RCOLS)) dst += dup_of(isx);
  };
  auto load_unit = [&](f32x4 (&v)[8], bool valid, bool isx, unsigned src, auto aux) INL {
    const __amdgpu_buffer_rsrc_t rs = __builtin_amdgcn_make_buffer_rsrc((void*)(isx ? p.x : p.w), 0, valid ? (isx ? GM * KSH * 4 : KSH * GN * 4) : 0, 0x00020000);
    const unsigned vv = isx ? vx : vw;
#pragma unroll
    for (int j = 0; j < 8; ++j)
      v[j] = __builtin_bit_cast(f32x4, __builtin_amdgcn_raw_buffer_load_b128(rs, vv, src + (isx ? (unsigned)((j >> 1) * 16 * KSH + (j & 1) * 16) * 4u : (unsigned)(((j >> 2) * 16 + (j & 3)) * GN) * 4u), decltype(aux)::value));
  };
  auto pack_unit = [&](const f32x4 (&v)[8], float m, bool isx, unsigned dst, u32x4 (&o)[4], unsigned (&oo)[4]) INL {
#pragma unroll
    for (int i = 0; i < 4; ++i) {
      o[i] = isx ? pack8(m, v[2 * i][0], v[2 * i][1], v[2 * i][2], v[2 * i][3], v[2 * i + 1][0], v[2 * i + 1][1], v[2 * i + 1][2], v[2 * i + 1][3])
                 : pack8(m, v[0][i], v[1][i], v[2][i], v[3][i], v[4][i], v[5][i], v[6][i], v[7][i]);
      oo[i] = vd + dst + (unsigned)(isx ? i * FRS : (i >> 1) * HTB + (i & 1) * FRS);
    }
  };

  auto decode = [&](unsigned u, bool& isx, unsigned& src, unsigned& dst) INL {
    int sg, i; isx = false;
    if (u < (unsigned)U1) { sg = u / 240u; i = u % 240u; isx = i < 128; if (!isx) i -= 128; }
    else { const unsigned v = u - U1; sg = 8 + v / 112u; i = v % 112u; }
    place(isx, isx ? i >> 3 : (sg >> 3) * RCOLS + ((i >> 3) + RCOLS - rot(rank)) % RCOLS, i & 3, (sg & 7) * 2 + ((i >> 2) & 1), src, dst);
  };
  const int aid = wid == 0 ? b : NAGENT + b;
  f32x4 v[8]; bool isx = false; unsigned src = 0, dst = 0;
  if (li < 4 && wid == 0) { decode((unsigned)b, isx, src, dst); load_unit(v, true, isx, src, IC<0>{}); }
  if (tid == 0) {
    const unsigned c = __hip_atomic_fetch_add((unsigned*)(myws + WS_CTR + b * 64), 1u, __ATOMIC_RELAXED, __HIP_MEMORY_SCOPE_AGENT) + 1u;
    *(LAS unsigned*)(lds + LDS_BAD + 4) = c; *(LAS unsigned*)(lds + LDS_BAD) = 0u;
  }
  *(LAS unsigned*)(lds + LDS_FLAG + tid * 4) = 0u;
  WAIT_L(0); BAR; asm volatile("" ::: "memory");
  const unsigned call = __builtin_amdgcn_readfirstlane(*(LAS unsigned*)(lds + LDS_BAD + 4));
  const unsigned seq0 = call * SEQ;
  constexpr size_t stg = WS_STG;
  odd = call & 1u;
  if (odd) dst += dup_of(isx);
  const char* flagw = myws + WS_FLAG + (lane & 7) * 128;

  unsigned* prog = (unsigned*)(myws + WS_PROG);
  unsigned* ticket = (unsigned*)(myws + WS_TICKET + (call & 3u) * 128u);
  auto draw = [&](unsigned* ctr, bool real) INL -> unsigned {
    return (unsigned)__builtin_amdgcn_raw_ptr_buffer_atomic_add_i32(1, __builtin_amdgcn_make_buffer_rsrc(ctr, 0, real ? 4 : 0, 0x00020000), (int)vd, 0, 0); };
  auto progress = [&](unsigned val) INL { __builtin_amdgcn_raw_buffer_store_b32(val, __builtin_amdgcn_make_buffer_rsrc(prog + aid, 0, 4, 0x00020000), vd, 0, 16); };
  auto read_go = [&]() INL -> unsigned { return __builtin_amdgcn_readfirstlane(__hip_atomic_load((unsigned*)(myws + WS_GO + 4), __ATOMIC_RELAXED, __HIP_MEMORY_SCOPE_AGENT)); };
  bool copy = false, staged = false;
#define load_send(valid) load_send_(v, valid, copy, isx, src, dst, myws + WS_STG, p.x, p.w, lane)
  auto send_ops = [&](unsigned go, unsigned cur, unsigned tnext, unsigned limit) INL {
    unsigned* ctr = ticket + (go >> 31) * 8; const unsigned base = (go >> 31) ? 0u : 2u * NAGENT;
    const float mx = pow2f((int)((go >> 12) & 0xfffu) - 256), mw = pow2f((int)(go & 0xfffu) - 256);
    __amdgpu_buffer_rsrc_t rd[NR - 1];
#pragma unroll
    for (int d = 0; d < NR - 1; ++d) rd[d] = __builtin_amdgcn_make_buffer_rsrc(p.ws[(rank + 1 + (d + b + wid) % (NR - 1)) & 7] + stg, 0, STG_BYTES, 0x00020000);
    bool real = true, open = go >> 31;
#pragma unroll
    for (int j = 0; j < 8; ++j) asm volatile("" : "+v"(v[j]));
    asm volatile("" : "+v"(tnext));
    while (cur < (unsigned)NUNITS) {
      u32x4 o[4]; unsigned oo[4];
      pack_unit(v, isx ? mx : mw, isx, dst, o, oo);
      if (copy) {
#pragma unroll
        for (int i = 0; i < 4; ++i) o[i] = __builtin_bit_cast(u32x4, v[i]);
      }
      const unsigned raw = __builtin_amdgcn_readfirstlane(tnext) + base, nxt = real ? raw & 0xffffffu : 0xffffffffu;
      open = open || (real && ((raw >> 24) & 1u));
      if (!open && cur >= (unsigned)(NDUP / 8 * 240)) { wave_wait<true>(myws + WS_START + (lane & 7) * 128, false, call - 1u, tmo, call); open = true; }
      SCHED;
      decode(nxt < (unsigned)NUNITS ? nxt : 0u, isx, src, dst);
      staged = staged || (real && (raw >> 25));
      copy = staged && nxt >= (unsigned)U1;
      load_send(nxt < (unsigned)NUNITS);
      real = nxt < limit; tnext = draw(ctr, real);
      SCHED;
      SEND_UNIT(9, true);
      cur = nxt;
    }
    WAIT_V(0);
  };
  auto join_ops = [&](unsigned go, unsigned limit) INL {
    unsigned* ctr = ticket + (go >> 31) * 8;
    const unsigned cur = (__builtin_amdgcn_readfirstlane(draw(ctr, true)) & 0xffffffu) + ((go >> 31) ? 0u : 2u * NAGENT);
    decode(cur < (unsigned)NUNITS ? cur : 0u, isx, src, dst); copy = false; load_send(cur < (unsigned)NUNITS);
    send_ops(go, cur, draw(ctr, true), limit);
  };
  auto send_out = [&]() INL {
    unsigned cur = draw(ticket + 16, true), tnext = draw(ticket + 16, true);
    const bool ok = wave_wait(myws + WS_PCNT, false, call * 16u - 1u, tmo, call);
    cur = __builtin_amdgcn_readfirstlane(cur);
    __amdgpu_buffer_rsrc_t rd[NR - 1];
#pragma unroll
    for (int d = 0; d < NR - 1; ++d) rd[d] = __builtin_amdgcn_make_buffer_rsrc(p.out[(rank + 1 + (d + b + wid) % (NR - 1)) & 7], 0, GM * GN * 4, 0x00020000);
    const unsigned vb = vd + (unsigned)((NRC + rank) * 256 * 4);
    auto orow = [](unsigned u, unsigned i) INL -> unsigned { return ((u * 37u) & 1023u) * 4u + i; };
    auto load_out = [&](unsigned u) INL {
      const __amdgpu_buffer_rsrc_t rs = __builtin_amdgcn_make_buffer_rsrc(p.out[rank], 0, u < (unsigned)NOUT ? GM * GN * 4 : 0, 0x00020000);
#pragma unroll
      for (int i = 0; i < 4; ++i) v[i] = __builtin_bit_cast(f32x4, __builtin_amdgcn_raw_buffer_load_b128(rs, vb, orow(u, i) * (unsigned)(GN * 4), 16));
    };
    load_out(cur);
    if (li >= 2) wave_wait<true>(myws + WS_SENT, false, seq0 + (unsigned)(NUNITS - PSTART), tmo, call);
#pragma unroll
    for (int j = 0; j < 4; ++j) asm volatile("" : "+v"(v[j]));
    asm volatile("" : "+v"(tnext));
    bool real = true;
    while (cur < (unsigned)NOUT) {
      u32x4 o[4]; unsigned oo[4];
#pragma unroll
      for (int i = 0; i < 4; ++i) { o[i] = __builtin_bit_cast(u32x4, v[i]); if (!ok) o[i] = (u32x4){0x7fc00000u, 0x7fc00000u, 0x7fc00000u, 0x7fc00000u}; oo[i] = vb + orow(cur, i) * (unsigned)(GN * 4); asm volatile("" : "+v"(o[i])); }
      const unsigned nxt = real ? __builtin_amdgcn_readfirstlane(tnext) : 0xffffffffu;
      SCHED;
      load_out(nxt);
      real = nxt < (unsigned)NOUT; tnext = draw(ticket + 16, real);
      SCHED;
      SEND_UNIT(5, false);
      cur = nxt;
    }
    WAIT_V(0);
    if (lane < NR) __hip_atomic_fetch_add((unsigned*)(p.ws[lane] + WS_DONE + rank * 128), 1u, __ATOMIC_RELAXED, __HIP_MEMORY_SCOPE_SYSTEM);
    STAMPW(11);
  };

  bool ready = false;
  if (li < 4) {
    if (wid == 0) {
      const unsigned limit = li < 2 ? (unsigned)NUNITS : li == 2 ? (unsigned)PLEAVE1 : (unsigned)PLEAVE;
      send_ops((unsigned)(SHIFT0 + 256) << 12 | (unsigned)(SHIFT0 + 256), (unsigned)b, (unsigned)(b - NAGENT), limit);
      unsigned go = 0;
      if (!staged) { wave_wait<true>(myws + WS_GO, false, call - 1u, tmo, call); go = read_go(); }
      if (go >> 31) { progress(seq0); WAIT_V(0); }
      if (lane == 0) __hip_atomic_fetch_add((unsigned*)(myws + WS_DCNT), 1u, __ATOMIC_RELAXED, __HIP_MEMORY_SCOPE_AGENT);
      if (go >> 31) { wave_wait<true>(myws + WS_DCNT, false, call * (unsigned)NAGENT - 1u, tmo, call); join_ops(go, limit); }
      progress(seq0 + (unsigned)(NUNITS + 8));
      STAMP(12);
      if (li < 2) send_out();
    } else if (wid == 2 && li < 2) {
      progress(seq0 + (unsigned)(SEQ - 1));
      wave_wait(myws + WS_SENT, false, seq0 + (unsigned)JOIN2, tmo, call);
      progress(seq0 + (unsigned)JOIN2);
      WAIT_V(0);
      join_ops(read_go(), (unsigned)NUNITS);
      progress(seq0 + (unsigned)(SEQ - 1));
    } else if (wid == 6) {
      wave_wait(myws + WS_SENT, false, seq0 + (unsigned)(PINAT), tmo, call);
      const float mw = pow2f((int)(read_go() & 0xfffu) - 256);
#pragma unroll 1
      for (int d = 1; d < NR; ++d) {
        const int pr = (rank + 1 + (d + b) % (NR - 1)) & 7;
        const __amdgpu_buffer_rsrc_t rp = __builtin_amdgcn_make_buffer_rsrc(p.ws[pr] + stg, 0, STG_BYTES, 0x00020000);
        f32x4 pv[2][8]; unsigned psrc, pdst[2];
#pragma unroll
        for (int e = 0; e < 2; ++e) { place(false, NRC + pr, (b >> 4) * 2 + e, b & 15, psrc, pdst[e]); load_unit(pv[e], true, false, psrc, IC<0>{}); }
#pragma unroll
        for (int e = 0; e < 2; ++e) { u32x4 o[4]; unsigned oo[4]; pack_unit(pv[e], mw, false, pdst[e], o, oo);
#pragma unroll
          for (int i = 0; i < 4; ++i) __builtin_amdgcn_raw_buffer_store_b128(o[i], rp, oo[i], 0, AUX_WT); }
      }
      WAIT_V(0);
      if (lane == 0) __hip_atomic_fetch_add((unsigned*)(myws + WS_PINC), 1u, __ATOMIC_RELAXED, __HIP_MEMORY_SCOPE_AGENT);
    } else if (b == 1 && wid == 7) {
      wave_wait(myws + WS_PKC + (lane & 7) * 128, false, call * (unsigned)NRW - 1u, tmo, call);
      wave_wait(myws + WS_SPD + (lane & 7) * 128, false, call - 1u, tmo, call);
      const unsigned f = __hip_atomic_load((unsigned*)(myws + WS_SPD + (lane & 7) * 128 + 4), __ATOMIC_RELAXED, __HIP_MEMORY_SCOPE_AGENT);
      const unsigned rl = lane & 7;
      unsigned kth = 0, task = rl, word = 0;
#pragma unroll
      for (int j = 0; j < 8; ++j) { const unsigned fj = __builtin_amdgcn_readlane(f, j), rj = __builtin_amdgcn_readlane(rl, j);
        if (((rj ^ rl) & 1u) == 0u && (fj == f ? j < (lane & 7) : (rl & 1u) ? fj < f : fj > f)) ++kth; }
#pragma unroll
      for (int j = 0; j < 8; ++j) { const unsigned fj = __builtin_amdgcn_readlane(f, j), rj = __builtin_amdgcn_readlane(rl, j), kj = __builtin_amdgcn_readlane(kth, j);
        if (((rj ^ rl) & 1u) && kj == kth && ((rl & 1u) ? fj * 1000u > f * SWAPT : f * 1000u > fj * SWAPT)) task = rj; }
#pragma unroll
      for (int j = 0; j < 8; ++j) word |= (unsigned)__builtin_amdgcn_readlane(task, j) << (3 * j);
      if (lane == 0) __hip_atomic_store((unsigned long long*)(myws + WS_MAP), (unsigned long long)word << 32 | call, __ATOMIC_RELAXED, __HIP_MEMORY_SCOPE_AGENT);
    } else if (b == 0 && wid == 7) {
      if (lane < 3) __hip_atomic_store((unsigned*)(myws + WS_TICKET + ((call + 2u) & 3u) * 128u) + lane * 8, 0u, __ATOMIC_RELAXED, __HIP_MEMORY_SCOPE_AGENT);
      unsigned* myflag = (lane & 7) == rank ? (unsigned*)(myws + WS_SENT) : (unsigned*)(p.ws[lane & 7] + WS_FLAG + rank * 128);
      const unsigned long long t0 = __builtin_amdgcn_s_memrealtime();
      { const unsigned st = __hip_atomic_load((unsigned*)(myws + WS_START + (lane & 7) * 128), __ATOMIC_RELAXED, __HIP_MEMORY_SCOPE_SYSTEM);
        const int ahead = __builtin_popcountll(__builtin_amdgcn_ballot_w64(lane < NR && st >= call));
        if (lane == 0) __hip_atomic_store((unsigned*)(myws + WS_LATE), ahead >= LATEN ? call : 0u, __ATOMIC_RELAXED, __HIP_MEMORY_SCOPE_AGENT);
        while (__builtin_amdgcn_s_memrealtime() - t0 < (unsigned long long)LATED) __builtin_amdgcn_s_sleep(2);
        if (lane < NR) __hip_atomic_store((unsigned*)(p.ws[lane] + WS_START + rank * 128), call, __ATOMIC_RELAXED, __HIP_MEMORY_SCOPE_SYSTEM); }
      bool ok = wave_wait<true>(myws + WS_START + (lane & 7) * 128, false, call - 1u, tmo, call);
      if (ok && lane == 0) __hip_atomic_fetch_add(ticket, 1u << 24, __ATOMIC_RELAXED, __HIP_MEMORY_SCOPE_AGENT);
      ok = wave_wait<true>(myws + WS_GO, false, call - 1u, tmo, call) && ok;
      const unsigned redo = read_go() >> 31;
      if (redo) {
        ok = wave_wait<true>(myws + WS_LCNT2, false, call * (unsigned)NRW - 1u, tmo, call) && ok;
        ok = wave_wait<true>(myws + WS_DCNT, false, call * (unsigned)NAGENT - 1u, tmo, call) && ok;
      }
      if (lane == 0) __hip_atomic_store((unsigned*)(myws + WS_FLAG + rank * 128), seq0 + (unsigned)(SEQ - 1), __ATOMIC_RELAXED, __HIP_MEMORY_SCOPE_SYSTEM);
      if (ok && lane == 0) __hip_atomic_fetch_add(ticket + redo * 8, 1u << 25, __ATOMIC_RELAXED, __HIP_MEMORY_SCOPE_AGENT);
      unsigned last = seq0; bool pin = false;
      const unsigned long long t1 = ok ? __builtin_amdgcn_s_memrealtime() : t0;
      while (last < seq0 + (unsigned)(NUNITS + 8) || !pin) {
        unsigned m = 0xffffffffu;
#pragma unroll
        for (int j = 0; j < (NPROG + 63) / 64; ++j) { const int a = j * 64 + lane; if (a < NPROG) { const unsigned x = __hip_atomic_load(prog + a, __ATOMIC_RELAXED, __HIP_MEMORY_SCOPE_AGENT); m = x < m ? x : m; } }
        if (!pin && last > seq0 + (unsigned)(PINAT) && (unsigned)__builtin_amdgcn_readfirstlane(__hip_atomic_load((unsigned*)(myws + WS_PINC), __ATOMIC_RELAXED, __HIP_MEMORY_SCOPE_AGENT)) >= call * 32u) {
          pin = true;
          if (lane < NR) __hip_atomic_store((unsigned*)(p.ws[lane] + WS_PIN + rank * 128), call, __ATOMIC_RELAXED, __HIP_MEMORY_SCOPE_SYSTEM);
        }
#pragma unroll
        for (int sft = 32; sft >= 1; sft >>= 1) { const unsigned x = __shfl_xor(m, sft); m = x < m ? x : m; }
        m = __builtin_amdgcn_readfirstlane(m);
        if (m > last) { last = m; if (lane < NR) __hip_atomic_store(myflag, m, __ATOMIC_RELAXED, __HIP_MEMORY_SCOPE_SYSTEM); }
        if (__builtin_amdgcn_s_memrealtime() - t1 > DEADLINE) { __hip_atomic_store(tmo, call, __ATOMIC_RELAXED, __HIP_MEMORY_SCOPE_AGENT); ok = false; break; }
      }
      ok = wave_wait<true>(myws + WS_DONE + (lane & 7) * 128, false, call * (unsigned)NAGENT - 1u, tmo, call) && ok;
      ok = ok && __hip_atomic_load(tmo, __ATOMIC_RELAXED, __HIP_MEMORY_SCOPE_AGENT) != call;
      if (!ok && lane == 0) __hip_atomic_store((unsigned*)p.out[rank] + (NRC + rank) * 256, 0x7fc00000u, __ATOMIC_RELAXED, __HIP_MEMORY_SCOPE_SYSTEM);
    }
    if (li < 2) return;
  } else {
    const __amdgpu_buffer_rsrc_t rl = __builtin_amdgcn_make_buffer_rsrc(myws + stg, 0, STG_BYTES, 0x00020000);
    { const unsigned long long t0 = __builtin_amdgcn_s_memrealtime(); while (__builtin_amdgcn_s_memrealtime() - t0 < (unsigned long long)(PREDELAY * (li - 3))) __builtin_amdgcn_s_sleep(4); }
    float mx = pow2f(SHIFT0), mw = mx;
    ready = true;
    for (int pass = 0; pass < 2; ++pass) {
      unsigned ax = 0, aw = 0;
      for (int j = b - 32; j < 640; j += NRW) {
        isx = j < 128;
        if (j < 128) place(true, 2 * (j & 7) + (wid >> 2), wid & 3, j >> 3, src, dst);
        else if (j < 576) place(false, 2 * ((j - 128) % 28) + (wid >> 2), wid & 3, (j - 128) / 28, src, dst);
        else place(false, NRC + ((j - 576) >> 3), wid & 3, 2 * (j & 7) + (wid >> 2), src, dst);
        if (j >= 128 && j < 576 && (j - 128) % 28 >= 7) load_unit(v, true, isx, src, IC<AUX_NT>{});
        else load_unit(v, true, isx, src, IC<0>{});
        unsigned am = 0;
#pragma unroll
        for (int q = 0; q < 32; ++q) { unsigned a = __builtin_bit_cast(unsigned, v[q >> 2][q & 3]) & 0x7fffffffu; a = a < 0x7f800000u ? a : 0u; am = a > am ? a : am; }
        if (isx) ax = am > ax ? am : ax; else aw = am > aw ? am : aw;
        if (j < 576 || ((j - 576) >> 3) == rank) {
          u32x4 o[4]; unsigned oo[4]; pack_unit(v, isx ? mx : mw, isx, dst, o, oo);
#pragma unroll
          for (int i = 0; i < 4; ++i) __builtin_amdgcn_raw_buffer_store_b128(o[i], rl, oo[i], 0, AUX_WT);
        }
      }
      if (pass == 0) {
#pragma unroll
        for (int sft = 32; sft >= 1; sft >>= 1) { const unsigned a = __shfl_xor(ax, sft), c = __shfl_xor(aw, sft); ax = a > ax ? a : ax; aw = c > aw ? c : aw; }
        if (lane == 0) { *(LAS unsigned*)(lds + LDS_BAD + 32 + wid * 8) = ax; *(LAS unsigned*)(lds + LDS_BAD + 36 + wid * 8) = aw; }
      }
      WAIT_V(0); WAIT_L(0); BAR; asm volatile("" ::: "memory");
      if (pass == 1) break;
      if (wid == 0) {
        if (lane < 2) {
          unsigned a = 0;
#pragma unroll
          for (int q = 0; q < 8; ++q) { const unsigned c = *(LAS unsigned*)(lds + LDS_BAD + 32 + q * 8 + lane * 4); a = c > a ? c : a; }
          __hip_atomic_fetch_max((unsigned long long*)(myws + WS_LMAX) + lane, (unsigned long long)call << 32 | a, __ATOMIC_RELAXED, __HIP_MEMORY_SCOPE_AGENT);
        }
        WAIT_V(0);
        if ((unsigned)__builtin_amdgcn_readfirstlane(draw((unsigned*)(myws + WS_LCNT), true)) == call * (unsigned)NRW - 1u) {
          const unsigned long long mm = __hip_atomic_load((unsigned long long*)(myws + WS_LMAX) + (lane & 1), __ATOMIC_RELAXED, __HIP_MEMORY_SCOPE_AGENT);
          const unsigned e1 = (unsigned)mm ? 1u + ((unsigned)mm >> 23) : 0u;
          const unsigned e12 = (unsigned)__builtin_amdgcn_readlane(e1, 0) << 16 | (unsigned)__builtin_amdgcn_readlane(e1, 1);
          if (lane < NR) __hip_atomic_store((unsigned long long*)(p.ws[lane] + WS_AMX + rank * 128), (unsigned long long)e12 << 32 | call, __ATOMIC_RELAXED, __HIP_MEMORY_SCOPE_SYSTEM);
          wave_wait<true>(myws + WS_AMX + (lane & 7) * 128, false, call - 1u, tmo, call);
          const unsigned e2 = __hip_atomic_load((unsigned*)(myws + WS_AMX + (lane & 7) * 128 + 4), __ATOMIC_RELAXED, __HIP_MEMORY_SCOPE_SYSTEM);
          unsigned gx = e2 >> 16, gw = e2 & 0xffffu;
#pragma unroll
          for (int sft = 4; sft >= 1; sft >>= 1) { const unsigned a = __shfl_xor(gx, sft), c = __shfl_xor(gw, sft); gx = a > gx ? a : gx; gw = c > gw ? c : gw; }
          auto shift_of = [](unsigned g) INL -> int { const int e = (int)g - 128; return (g == 0u || (e >= -SHIFT0 && e < 15 - SHIFT0)) ? SHIFT0 : (14 - e > 126 ? 126 : 14 - e); };
          const int sx = shift_of(__builtin_amdgcn_readfirstlane(gx)), sw = shift_of(__builtin_amdgcn_readfirstlane(gw));
          const bool redo = sx != SHIFT0 || sw != SHIFT0;
          const bool late = __hip_atomic_load((unsigned*)(myws + WS_LATE), __ATOMIC_RELAXED, __HIP_MEMORY_SCOPE_AGENT) == call;
          if (lane == 0) {
            __hip_atomic_store((unsigned long long*)(myws + WS_GO), (unsigned long long)((redo ? 0x80000000u : 0u) | (late ? 0x40000000u : 0u) | (unsigned)(sx + 256) << 12 | (unsigned)(sw + 256)) << 32 | call, __ATOMIC_RELAXED, __HIP_MEMORY_SCOPE_AGENT);
            if (redo) __hip_atomic_fetch_add(ticket, 1u << 20, __ATOMIC_RELAXED, __HIP_MEMORY_SCOPE_AGENT);
          }
        }
        if (!wave_wait(myws + WS_GO, false, call - 1u, tmo, call)) *(LAS unsigned*)(lds + LDS_BAD) = 1u;
        *(LAS unsigned*)(lds + LDS_BAD + 8) = read_go();
        WAIT_L(0);
      }
      asm volatile("" ::: "memory"); BAR; asm volatile("" ::: "memory");
      const unsigned go = __builtin_amdgcn_readfirstlane(*(LAS unsigned*)(lds + LDS_BAD + 8));
      if (!(go >> 31)) break;
      mx = pow2f((int)((go >> 12) & 0xfffu) - 256); mw = pow2f((int)(go & 0xfffu) - 256); ready = false;
    }
    if (tid == 0) __hip_atomic_fetch_add((unsigned*)(myws + WS_LCNT2), 1u, __ATOMIC_RELAXED, __HIP_MEMORY_SCOPE_AGENT);
    STAMP(13);
  }

  const bool isP = li < 4;
  const int wr = wid >> 2, wc = wid & 3;
  const int lr = li - 4;
  const int role = xcd;
  const int mt = isP ? xcd * 2 + (li - 2) : (role >> 1) * 4 + (lr & 3);
  const int cr = isP ? 0 : (role & 1) + 2 * (lr >> 2);
  const unsigned voff = (unsigned)(wid * 2048 + lane * 16), ldsw = (unsigned)wid * 2048u;
  const char* fsrc = myws + WS_FLAG + wid * 128;
  const __amdgpu_buffer_rsrc_t ra = __builtin_amdgcn_make_buffer_rsrc(myws + stg, 0, A_BYTES, 0x00020000);
  const __amdgpu_buffer_rsrc_t rb = __builtin_amdgcn_make_buffer_rsrc(myws + stg + A_BYTES + (isP ? B_BYTES : 0u), 0, isP ? P_BYTES : B_BYTES, 0x00020000);
  const unsigned bstep = isP ? (unsigned)PIECE : B_STEP;
#define STAGE(ldsoff, rs, so) do { \
    __builtin_amdgcn_raw_ptr_buffer_load_lds(rs, (LAS void*)(lds + (ldsoff) + ldsw), 16, voff, (so), 0, AUX_SC1); \
    __builtin_amdgcn_raw_ptr_buffer_load_lds(rs, (LAS void*)(lds + (ldsoff) + ldsw), 16, voff, (so), 1024, AUX_SC1); } while (0)
#define ST1(ldsoff, rs, so, imm) do { __builtin_amdgcn_raw_ptr_buffer_load_lds(rs, (LAS void*)(lds + (ldsoff) + ldsw), 16, voff, (so), imm, AUX_SC1); SCHED; } while (0)
#define RA(h, m) do { _Pragma("unroll") for (int k = 0; k < 2; ++k) At[m][k] = *(const LAS f16x8*)((LAS char*)(size_t)va + (h) * HTB + (m) * FRS + k * KHS); SCHED; } while (0)
#define FLAGDMA() __builtin_amdgcn_global_load_lds((const unsigned*)fsrc, (LAS unsigned*)(lds + LDS_FLAG + wid * 256), 4, 0, AUX_SC1)
#define LDA(dst, h) do { _Pragma("unroll") for (int m = 0; m < 4; ++m) _Pragma("unroll") for (int k = 0; k < 2; ++k) dst[m][k] = *(const LAS f16x8*)((LAS char*)(size_t)va + (h) * HTB + m * FRS + k * KHS); } while (0)
#define LDB(dst, h) do { _Pragma("unroll") for (int n = 0; n < 2; ++n) _Pragma("unroll") for (int k = 0; k < 2; ++k) dst[n][k] = *(const LAS f16x8*)((LAS char*)(size_t)vb + (h) * HTB + n * FRS + k * KHS); } while (0)
#define ACC(ai, nn, m) (128 + (((ai) * 4 + (nn)) * 4 + (m)) * 4)
#define MMA2(ai) do { MMA2_(ai); BAR; SCHED; } while (0)
#define MMA2_(ai) do { __builtin_amdgcn_s_waitcnt(0xC07F); SCHED; BAR; SCHED; __builtin_amdgcn_s_setprio(1); SCHED; \
    _Pragma("unroll") for (int m = 0; m < 4; ++m) _Pragma("unroll") for (int q = 0; q < 4; ++q) { const int nn = (m & 1) ? 3 - q : q; \
      asm volatile("v_mfma_f32_16x16x32_f16 v[%c0:%c1], %2, %3, v[%c0:%c1]\n\tv_mfma_f32_16x16x32_f16 v[%c0:%c1], %4, %5, v[%c0:%c1]" \
          :: "i"(ACC(ai, nn, m)), "i"(ACC(ai, nn, m) + 3), "v"(At[m][0]), "v"((nn >> 1) ? B1[nn & 1][0] : B0[nn & 1][0]), "v"(At[m][1]), "v"((nn >> 1) ? B1[nn & 1][1] : B0[nn & 1][1])); } \
    SCHED; __builtin_amdgcn_s_setprio(0); SCHED; } while (0)
#define ZERO_ACC() do { _Pragma("unroll") for (int i_ = 128; i_ < 256; i_ += 2) asm volatile("v_mov_b64 v[%c0:%c1], 0" :: "i"(i_), "i"(i_ + 1)); } while (0)
#define ACC_ROW(dst, ai, m, j) do { float t0_, t1_, t2_, t3_; \
    asm volatile("v_ldexp_f32 %0, v[%c4], %8\n\tv_ldexp_f32 %1, v[%c5], %8\n\tv_ldexp_f32 %2, v[%c6], %8\n\tv_ldexp_f32 %3, v[%c7], %8" \
        : "=&v"(t0_), "=&v"(t1_), "=&v"(t2_), "=&v"(t3_) : "i"(ACC(ai, 0, m) + (j)), "i"(ACC(ai, 1, m) + (j)), "i"(ACC(ai, 2, m) + (j)), "i"(ACC(ai, 3, m) + (j)), "v"(oexp)); \
    dst = (f32x4){t0_, t1_, t2_, t3_}; } while (0)
  asm volatile("" ::: "v128","v129","v130","v131","v132","v133","v134","v135","v136","v137","v138","v139","v140","v141","v142","v143","v144","v145","v146","v147","v148","v149","v150","v151","v152","v153","v154","v155","v156","v157","v158","v159","v160","v161","v162","v163","v164","v165","v166","v167","v168","v169","v170","v171","v172","v173","v174","v175","v176","v177","v178","v179","v180","v181","v182","v183","v184","v185","v186","v187","v188","v189","v190","v191","v192","v193","v194","v195","v196","v197","v198","v199","v200","v201","v202","v203","v204","v205","v206","v207","v208","v209","v210","v211","v212","v213","v214","v215","v216","v217","v218","v219","v220","v221","v222","v223","v224","v225","v226","v227","v228","v229","v230","v231","v232","v233","v234","v235","v236","v237","v238","v239","v240","v241","v242","v243","v244","v245","v246","v247","v248","v249","v250","v251","v252","v253","v254","v255");
  ZERO_ACC();
  f16x8 At[4][2], B0[2][2], B1[2][2];
  const int ph = isP ? 0 : phase_of(cr);
  if (!ready) {
    if (wid == 0) {
      if (isP && !wave_wait(myws + WS_PIN + (lane & 7) * 128, false, call - 1u, tmo, call)) *(LAS unsigned*)(lds + LDS_BAD) = 1u;
      if (!wave_wait(isP ? flagw : myws + WS_FLAG + rank * 128, false, seq0 + (unsigned)(U1 - 1), tmo, call)) *(LAS unsigned*)(lds + LDS_BAD) = 1u;
      *(LAS unsigned*)(lds + LDS_BAD + 8) = read_go(); WAIT_L(0);
    }
    asm volatile("" ::: "memory"); BAR; asm volatile("" ::: "memory");
  }
  const unsigned gow = __builtin_amdgcn_readfirstlane(*(LAS unsigned*)(lds + LDS_BAD + 8));
  const int oexp = 512 - (int)((gow >> 12) & 0xfffu) - (int)(gow & 0xfffu);
  const int NL = (gow >> 30) & 1u ? 0 : 1;
  int rolex = role, mtx = mt, crx = cr;
  const int E0 = 8 + rho0_of(cr);
  int Tend = isP ? 64 : 256;
  unsigned tA, tB, tN;
  auto steps = [&](int t0) INL {
    const int t = t0 + lane, u = t - E0, c = u < 192 ? cr : crx, w = u - 8 * NL - 8;
    const int z = w - 56, zn = 128 - 64 * NL;
    const bool inl = z >= 0 && z < zn;
    const bool local = t < 8 || (u >= 0 && w < 0) || (inl && (z & 7) == 0);
    const int rho = u < 0 ? t - 8 : u >= 192 ? rho0_of(crx) + u - 24 : E0 - 8 + w - (z < 0 ? 0 : inl ? (z >> 3) + 1 : zn >> 3), pos = rho % 7;
    const int sgm = !local ? rho / 7 : t < 8 ? (t < ph ? t : t + 24) : inl ? ph + 8 + (z >> 3) : u < 8 * NL ? ph + 16 + u : ph + (u & 7);
    const int f = first_src(c), sr = local ? rank : (f + pos + (pos >= ((rank - f) & 7) ? 1 : 0)) & 7;
    const unsigned ix = isP ? (unsigned)t : (unsigned)(sgm * 8 + sr);
    tA = t < Tend ? (unsigned)(u < 192 ? mt : mtx) * PIECE + (ix & 63u) * A_STEP + (odd && (ix & 63u) < (unsigned)NDUP ? dup_of(true) : 0u) : 0x80000000u;
    tB = t < Tend ? (unsigned)c * PIECE + ix * bstep + (odd && !isP && ix < (unsigned)NDUP ? dup_of(false) : 0u) : 0x80000000u;
    tN = (isP || t >= Tend || local) ? 0u : seq0 + (unsigned)((sgm < 8 ? 240 * sgm + 128 : U1 - 8 * 112 + 112 * sgm) + 8 * ((c + rot(sr)) % RCOLS) + 8 + sr);
  };
  steps(0);
  STAMP(1);
  __builtin_amdgcn_s_waitcnt(0x0F70); SCHED;
  unsigned par = 0, l1 = PIECE + HTB, a1, fl = 0;
  unsigned a2 = __builtin_amdgcn_readlane(tA, 2), b2 = __builtin_amdgcn_readlane(tB, 2), need = 0;
  unsigned va = (unsigned)(size_t)lds + wr * 4 * FRS + lane * 16, vb = (unsigned)(size_t)lds + 2 * PIECE + wc * 2 * FRS + lane * 16;
  LAS unsigned* fimg = (LAS unsigned*)(lds + LDS_FLAG + (lane & 7) * 256); asm volatile("" : "+v"(fimg));
  { const unsigned a0 = __builtin_amdgcn_readlane(tA, 0), b0 = __builtin_amdgcn_readlane(tB, 0), b1 = __builtin_amdgcn_readlane(tB, 1); a1 = __builtin_amdgcn_readlane(tA, 1);
    STAGE(2 * PIECE, rb, b0); STAGE(0, ra, a0); STAGE(2 * PIECE + HTB, rb, b0 + HTB); STAGE(HTB, ra, a0 + HTB);
    if (wr == 1) BAR;
    WAIT_V(2); BAR;
    FLAGDMA();
    STAGE(3 * PIECE, rb, b1); STAGE(PIECE, ra, a1); STAGE(3 * PIECE + HTB, rb, b1 + HTB);
    a1 += HTB;
    BAR; }
  const unsigned vpk = (unsigned)tid * 16u;
  const unsigned vring = (unsigned)(size_t)lds + LDS_RING + wid * 1024 + lane * 16;
#define SLOW_T0
#define SLOW_T1
#define STEP_STAMP
#define KLOOP(ADD, LIM) for (;;) { \
      f32x4 pt; \
      ST1(l1, ra, a1, 0); LDB(B0, 0); SCHED; ST1(l1, ra, a1, 1024); LDB(B1, 1); SCHED; \
      if (ADD) { __builtin_amdgcn_raw_ptr_buffer_load_lds(rpk, (LAS void*)(lds + LDS_RING + rs_ + wid * 1024), 16, vpk, (unsigned)(li3 - a0_) * 8192u, 0, AUX_SC1 | AUX_NT); SCHED; \
        rs_ = rs_ == 16384u ? 0u : rs_ + 8192u; pt = *(const LAS f32x4*)(size_t)(vring + rs_); } \
      else FLAGDMA(); \
      SCHED; LDA(At, 0); WAIT_V(10); \
      MMA2(0); \
      if ((unsigned)__builtin_amdgcn_readlane(fl, need & 63u) < need) { SLOW_T0; \
        const unsigned f2 = *(const volatile LAS unsigned*)fimg; fl = f2 > fl ? f2 : fl;     \
        if ((unsigned)__builtin_amdgcn_readlane(fl, need & 63u) < need) { \
          if (!wave_wait(flagw, (unsigned)(lane & 7) != (need & 7u), need - 1u, tmo, call, &fl)) *(LAS unsigned*)(lds + LDS_BAD) = 1u; \
          *(volatile LAS unsigned*)fimg = fl; } \
        SLOW_T1; } \
      ST1(2 * PIECE + par, rb, b2, 0); RA(1, 0); ST1(2 * PIECE + par, rb, b2, 1024); RA(1, 1); \
      ST1(par, ra, a2, 0); RA(1, 2); ST1(par, ra, a2, 1024); RA(1, 3); \
      const unsigned fn = ADD ? 0u : *fimg; \
      if (ADD && li3 - a0_ >= 2) { unsigned keep_; \
        asm volatile("s_mov_b32 %0, m0\n\ts_set_gpr_idx_on %1, gpr_idx(SRC0,DST)\n\tv_add_f32 v128, v128, %2\n\tv_add_f32 v129, v129, %3\n\tv_add_f32 v130, v130, %4\n\tv_add_f32 v131, v131, %5\n\ts_set_gpr_idx_off\n\ts_mov_b32 m0, %0" \
            : "=&s"(keep_) : "s"(((li3 - a0_ - 2) & 31) * 4), "v"(pt[0]), "v"(pt[1]), "v"(pt[2]), "v"(pt[3])); } \
      SCHED; \
      ST1(2 * PIECE + par + HTB, rb, b2 + HTB, 0); \
      l1 = par + HTB; a1 = a2 + HTB; va ^= PIECE; vb ^= PIECE; \
      a2 = __builtin_amdgcn_readlane(tA, li3); need = __builtin_amdgcn_readlane(tN, li3); \
      asm volatile("" : "+v"(va), "+v"(vb), "+s"(a2), "+s"(need), "+s"(l1), "+s"(a1)); SCHED; \
      ST1(2 * PIECE + par + HTB, rb, b2 + HTB, 1024); \
      par ^= PIECE; b2 = __builtin_amdgcn_readlane(tB, li3); ++li3; \
      STEP_STAMP \
      fl = fn > fl ? fn : fl; \
      asm volatile("" : "+s"(par), "+s"(li3), "+s"(b2), "+v"(fl)); \
      WAIT_V(9); \
      MMA2_(1); \
      if (li3 >= (LIM)) break; \
      BAR; SCHED; }
  int t = 0;
  steps(3);
  float* const myout = p.out[rank] + (size_t)(wr * 64) * GN + wc * 64;
  for (int sg = isP ? 4 : -NL; sg < 5; ++sg) {
    const int Ts = sg == 4 ? Tend : sg == -NL ? E0 : t + (sg <= 0 ? 8 : (NL && sg == 3) ? 56 : 64);
    const int slot = sg <= 0 ? sg + NL : (sg == 4 ? 0 : 1);
    char* pk = myws + WS_PARK + (size_t)(slot * 256 + (sg == 4 ? rolex : role) * 32 + li) * 262144;
    const __amdgpu_buffer_rsrc_t rpk = __builtin_amdgcn_make_buffer_rsrc(pk, 0, 262144, 0x00020000);
    const int left = Ts - t; int li3 = 0;
    unsigned rs_ = 0;
    const bool add = !isP && (sg == 4 || (NL && sg == 3));
    const bool ref = sg == 2 && lr == ((role & 1) ? 4 : 0);
    const int a1_ = 29 - E0 - (sg == 4 ? ADDOFF : ADDOFF2),
              a0_ = ref ? 32 : (add && a1_ > 0) ? (a1_ < left - 35 ? a1_ : left - 35) : 0;
    const unsigned r0_ = (unsigned)__builtin_amdgcn_s_memrealtime(); __builtin_amdgcn_s_waitcnt(0xC07F);
    for (int part = a0_ ? 0 : 1; part < 2; ++part) {
      if (part == 1 && add) { KLOOP(true, a0_ + 34); BAR; SCHED; }
      const int lim_ = part ? left : a0_;
      KLOOP(false, lim_);
      if (!part) {
        BAR; SCHED;
        if (ref && tid == 0) __hip_atomic_store((unsigned long long*)(myws + WS_SPD + xcd * 128),
            (unsigned long long)((1u << 28) / __builtin_elementwise_max((unsigned)__builtin_amdgcn_s_memrealtime() - r0_, 256u)) << 32 | call, __ATOMIC_RELAXED, __HIP_MEMORY_SCOPE_AGENT);
      }
    }
    t = Ts;
    if (wr == 0) BAR;
    asm volatile("s_nop 15\n\ts_nop 15" ::: "memory");
    SCHED;
    STAMP(2 + 2 * r);
    if (sg <= 0) {
      unsigned vq = vpk; asm volatile("" : "+v"(vq));
      if (sg == -NL) {
#pragma unroll
      for (int i = 0; i < 32; ++i)
        asm volatile("s_nop 4\n\tglobal_store_dwordx4 %0, v[%c1:%c2], %3" " sc1" :: "v"(vq + i * 8192u), "i"(128 + 4 * i), "i"(128 + 4 * i + 3), "s"(pk) : "memory");
      } else {
#pragma unroll
      for (int i = 0; i < 32; ++i)
        asm volatile("s_nop 4\n\tglobal_store_dwordx4 %0, v[%c1:%c2], %3" " nt" :: "v"(vq + i * 8192u), "i"(128 + 4 * i), "i"(128 + 4 * i + 3), "s"(pk) : "memory");
      }
      asm volatile("s_nop 1");
      ZERO_ACC();
      steps(t + 3);
      if (sg == -NL) {
        WAIT_V(0);
        if (lane == 0) __hip_atomic_fetch_add((unsigned*)(myws + WS_PKC + xcd * 128), 1u, __ATOMIC_RELAXED, __HIP_MEMORY_SCOPE_AGENT);
      }
    } else {
      if (sg == 2) {
        unsigned mp = 0;
        if (!wave_wait(myws + WS_MAP + (lane & 1) * 4, lane & 1, call - 1u, tmo, call, &mp)) *(LAS unsigned*)(lds + LDS_BAD) = 1u;
        else rolex = (int)(((unsigned)__builtin_amdgcn_readlane(mp, 1) >> (3 * xcd)) & 7u);
        mtx = (rolex >> 1) * 4 + (lr & 3); crx = (rolex & 1) + 2 * (lr >> 2); Tend = E0 + 192 + 56 - rho0_of(crx);
      }
      if (__builtin_amdgcn_readfirstlane(*(const LAS unsigned*)(lds + LDS_BAD)) != 0u) {
#pragma unroll
        for (int i = 128; i < 256; ++i) asm volatile("v_mov_b32 v[%c0], 0x7fc00000" :: "i"(i));
      }
      const int nt = isP ? NRC + rank : sg == 4 ? crx : sg * RCOLS + cr;
      const __amdgpu_buffer_rsrc_t rc = __builtin_amdgcn_make_buffer_rsrc(myout + (size_t)((sg == 4 ? mtx : mt) * 256) * GN + nt * 256, 0, 192u * GN * 4u, 0x00020000);
      unsigned vo = (unsigned)((lane >> 4) * 4 * GN + (lane & 15) * 4) * 4u; asm volatile("" : "+v"(vo));
#pragma unroll
      for (int ai = 0; ai < 2; ++ai)
#pragma unroll
        for (int m = 0; m < 4; ++m) {
#pragma unroll
          for (int j = 0; j < 4; ++j) {
            f32x4 v4; ACC_ROW(v4, ai, m, j);
            __builtin_amdgcn_raw_buffer_store_b128(__builtin_bit_cast(u32x4, v4), rc, vo + (unsigned)((ai * 128 + m * 16 + j) * GN * 4), 0, AUX_NT);
          }
#pragma unroll
          for (int nn = 0; nn < 4; ++nn)
            asm volatile("v_mov_b64 v[%c0:%c1], 0\n\tv_mov_b64 v[%c2:%c3], 0" :: "i"(ACC(ai, nn, m)), "i"(ACC(ai, nn, m) + 1), "i"(ACC(ai, nn, m) + 2), "i"(ACC(ai, nn, m) + 3));
          if (ai == 0 && m == 1) {
            SCHED; steps(t + 3); asm volatile("" : "+v"(tA), "+v"(tB), "+v"(tN)); SCHED;
          }
        }
    }
    if (wr == 1) BAR;
    STAMP(3 + 2 * r);
    STAMPC(20 + r);
  }
  WAIT_V(0);
  STAMP(15);
  if (wr == 0) BAR;
  if (isP) {
    BAR; asm volatile("" ::: "memory");
    if (tid == 0) {
      __builtin_amdgcn_fence(__ATOMIC_RELEASE, "agent");
      WAIT_V(0);
      __hip_atomic_fetch_add((unsigned*)(myws + WS_PCNT), 1u, __ATOMIC_RELAXED, __HIP_MEMORY_SCOPE_AGENT);
    }
    STAMP(10);
    if (wid == 0) {
      if (li == 2) {
        progress(seq0 + (unsigned)PLEAVE1);
        WAIT_V(0);
        join_ops(gow, (unsigned)NUNITS);
        progress(seq0 + (unsigned)(NUNITS + 8));
      }
      send_out();
    }
  }
}
}

extern "C" void kernel_launch(int rank, int nranks, ncclComm_t comm, hipStream_t stream,
                              void* const* d_in, const int* in_sizes, int n_in,
                              void* d_out, int out_size, const int* in_dtypes, int out_dtype,
                              void* const* d_out_all, void* const* d_ws_all, size_t ws_bytes) {
  (void)comm; (void)in_dtypes; (void)out_dtype; (void)d_out;
  if (nranks != NR || (unsigned)rank >= (unsigned)NR || n_in < 2 || ws_bytes < WS_NEED || out_size != GM * GN ||
      in_sizes[0] != GM * KSH || in_sizes[1] != KSH * GN) { fprintf(stderr, "unexpected problem shape\n"); return; }
  static bool attr_set = false;
  if (!attr_set) { HIP_CHECK(hipFuncSetAttribute((const void*)fused_kernel, hipFuncAttributeMaxDynamicSharedMemorySize, LDS_BYTES)); attr_set = true; }
  Params p;
  p.x = (const float*)d_in[0]; p.w = (const float*)d_in[1]; p.rank = rank;
  for (int r = 0; r < NR; ++r) { p.out[r] = (float*)d_out_all[r]; p.ws[r] = (char*)d_ws_all[r]; }
  fused_kernel<<<256, 512, LDS_BYTES, stream>>>(p);
  HIP_CHECK(hipGetLastError());
}
```

```cpp
#include <hip/hip_runtime.h>
#include <rccl/rccl.h>
#include <cstdio>

#define HIP_CHECK(expr) do { hipError_t _e = (expr); if (_e != hipSuccess) { \
    fprintf(stderr, "HIP error %s at %s:%d\n", hipGetErrorString(_e), __FILE__, __LINE__); \
    return; } } while (0)

namespace {
#define LAS __attribute__((address_space(3)))
#define INL __attribute__((always_inline))
typedef _Float16 f16x8 __attribute__((ext_vector_type(8)));
typedef float f32x4 __attribute__((ext_vector_type(4)));
typedef unsigned u32x4 __attribute__((ext_vector_type(4)));

constexpr int NR = 8, GM = 4096, GN = 16384, KSH = 512;
constexpr int PIECE = 32768, HTB = 16384, NKS = 64;
constexpr int FRS = 2048, KHS = 1024;
constexpr int RCOLS = 14, ROUNDS = 4, NRC = RCOLS * ROUNDS;
constexpr unsigned A_STEP = 16u * PIECE, B_STEP = (unsigned)RCOLS * PIECE;
constexpr int NDUP = 16;
constexpr unsigned A_BYTES = (NKS + NDUP) * A_STEP, B_BYTES = (unsigned)(ROUNDS * NKS + NDUP) * B_STEP, P_BYTES = (unsigned)NKS * PIECE;
constexpr unsigned STG_BYTES = A_BYTES + B_BYTES + P_BYTES;
constexpr size_t WS_CTR = 0;
constexpr size_t WS_FLAG = 16384, WS_START = 18432, WS_PIN = 24576, WS_AMX = 28672, WS_DONE = 29696;
constexpr size_t WS_TMO = 20480, WS_TICKET = 21504, WS_PROG = 22528, WS_LCNT = 26624, WS_SENT = 26752, WS_PCNT = 26880, WS_PINC = 27008,
                 WS_LCNT2 = 27136, WS_DCNT = 27264, WS_LMAX = 27392, WS_GO = 27520, WS_LATE = 27648, WS_MAP = 27776;
constexpr size_t WS_PKC = 30720, WS_SPD = 31744;
constexpr size_t WS_DBG = 65536, WS_STG = 1u << 20;
constexpr size_t WS_PARK = WS_STG + (size_t)STG_BYTES;
constexpr size_t WS_NEED = WS_PARK + 2 * 256 * (size_t)262144;
#define PINAT 2200
#define PLEAVE1 2600
#define PLEAVE 3150
#define JOIN2 2600
#define PSTART 200
#define PREDELAY 40
#define ADDOFF 8
#define ADDOFF2 4
#define SWAPT 1012u
constexpr int POLLS = 32, LATEN = 4, LATED = 300;
constexpr int U1 = 8 * 240, NUNITS = U1 + 24 * 112, NOUT = GM / 4, SEQ = 8192, NAGENT = 32, NPROG = 48, NRW = 224;
constexpr int SHIFT0 = 8;
constexpr int LDS_FLAG = 131072, LDS_BAD = LDS_FLAG + 2048, LDS_RING = LDS_BAD + 128, LDS_BYTES = LDS_RING + 3 * 8192;
constexpr unsigned long long DEADLINE = 80000000ull;
constexpr int AUX_WT = 17, AUX_NT = 2, AUX_SC1 = 16;

template <int V> struct IC { static constexpr int value = V; };
struct Params { const float* x; const float* w; float* out[NR]; char* ws[NR]; int rank; };

#define WAIT_V(n) asm volatile("s_waitcnt vmcnt(" #n ")" ::: "memory")
#define WAIT_L(n) asm volatile("s_waitcnt lgkmcnt(" #n ")" ::: "memory")
#define BAR __builtin_amdgcn_s_barrier()
#define SCHED __builtin_amdgcn_sched_barrier(0)
#define STAMPC(i) do {} while (0)
#define STAMPW(i) do {} while (0)
#define STAMP(i) do {} while (0)

template <bool FAST = false>
__device__ __forceinline__ bool wave_wait(const char* words, bool skip, unsigned need, unsigned* tmo, unsigned call, unsigned* seen = nullptr) {
  const unsigned long long t0 = __builtin_amdgcn_s_memrealtime();
  bool ok = true;
  for (unsigned n = 1;; ++n) {
    const unsigned v = (n & 7u) ? __hip_atomic_load((const unsigned*)words, __ATOMIC_RELAXED, __HIP_MEMORY_SCOPE_AGENT)
                                : __hip_atomic_load((const unsigned*)words, __ATOMIC_RELAXED, __HIP_MEMORY_SCOPE_SYSTEM);
    if (seen) *seen = v;
    if (__builtin_amdgcn_ballot_w64(skip || v > need) == ~0ull) break;
    if ((n & 7u) == 0u &&
        ((unsigned)__builtin_amdgcn_readfirstlane(__hip_atomic_load(tmo, __ATOMIC_RELAXED, __HIP_MEMORY_SCOPE_AGENT)) == call ||
         __builtin_amdgcn_s_memrealtime() - t0 > DEADLINE)) {
      __hip_atomic_store(tmo, call, __ATOMIC_RELAXED, __HIP_MEMORY_SCOPE_AGENT);
      ok = false; break;
    }
    __builtin_amdgcn_s_sleep(FAST ? 1 : POLLS);
  }
  asm volatile("" ::: "memory");
  return ok;
}

__device__ __forceinline__ float pow2f(int e) { return __builtin_bit_cast(float, (unsigned)(e + 127) << 23); }
__device__ __forceinline__ u32x4 pack8(float m, float a0, float a1, float a2, float a3, float a4, float a5, float a6, float a7) {
  f16x8 h = {(_Float16)(a0 * m), (_Float16)(a1 * m), (_Float16)(a2 * m), (_Float16)(a3 * m), (_Float16)(a4 * m), (_Float16)(a5 * m), (_Float16)(a6 * m), (_Float16)(a7 * m)};
  return __builtin_bit_cast(u32x4, h);
}

__device__ __forceinline__ void load_send_(f32x4 (&v)[8], bool valid, bool copy, bool isx, unsigned src, unsigned dst,
                                           const void* stg, const void* xp, const void* wp, int lane) {
  const __amdgpu_buffer_rsrc_t rs = __builtin_amdgcn_make_buffer_rsrc((void*)(copy ? stg : isx ? xp : wp), 0,
                                                                      !valid ? 0u : copy ? STG_BYTES : isx ? GM * KSH * 4 : KSH * GN * 4, 0x00020000);
  const unsigned vv = copy ? (unsigned)lane * 16u : isx ? (unsigned)((lane & 15) * KSH + 4 * (lane >> 4)) * 4u : (unsigned)(4 * (lane >> 4) * GN + 4 * (lane & 15)) * 4u;
#pragma unroll
  for (int j = 0; j < 8; ++j) {
    const unsigned so = copy ? (j < 4 ? dst + (unsigned)((j >> 1) * HTB + (j & 1) * FRS) : 0x80000000u)
                             : src + (isx ? (unsigned)((j >> 1) * 16 * KSH + (j & 1) * 16) * 4u : (unsigned)(((j >> 2) * 16 + (j & 3)) * GN) * 4u);
    v[j] = __builtin_bit_cast(f32x4, __builtin_amdgcn_raw_buffer_load_b128(rs, vv, so, 16));
  }
}

#define SEND_UNIT(NPRE, REPORT) do { \
    _Pragma("unroll") for (int i = 0; i < 4; ++i) { \
      if (i == 2) { asm volatile("s_waitcnt vmcnt(%0)" :: "i"(14 + (NPRE)) : "memory"); if (REPORT) progress(seq0 + cur); } \
      SCHED; \
      _Pragma("unroll") for (int d = 0; d < NR - 1; ++d) __builtin_amdgcn_raw_buffer_store_b128(o[i], rd[d], oo[i], 0, AUX_WT); \
      SCHED; } } while (0)

__global__ void __launch_bounds__(512, 2) fused_kernel(Params p) {
  extern __shared__ __attribute__((aligned(16))) char smem[];
  LAS char* lds = (LAS char*)smem;
  const int tid = threadIdx.x, wid = __builtin_amdgcn_readfirstlane(tid >> 6), lane = tid & 63;
  const int b = blockIdx.x, xcd = b & 7, li = b >> 3, rank = p.rank;
  char* myws = p.ws[rank];
  unsigned* tmo = (unsigned*)(myws + WS_TMO);
  STAMP(0);
  const int g4 = lane >> 4, r16 = lane & 15;
  const unsigned vx = (unsigned)(r16 * KSH + 4 * g4) * 4u, vw = (unsigned)(4 * g4 * GN + 4 * r16) * 4u, vd = (unsigned)lane * 16u;
  auto rot = [](int r) INL -> int { return (r * 7) >> 2; };
  auto first_src = [](int cr) INL -> int { return cr > 1 ? (4 * (14 - cr) + 6) / 7 : 0; };
  auto half_of = [](int cr) INL -> int { return ((cr >> 1) < 5 ? cr >> 1 : 5) + 1 - (cr & 1); };
  auto early = [&](int cr) INL -> bool { return ((rank - first_src(cr)) & 7) < 4; };
  auto phase_of = [&](int cr) INL -> int { return (half_of(cr) >> 1) + ((half_of(cr) & 1) && early(cr) ? 1 : 0); };
  auto rho0_of = [&](int cr) INL -> int { return 7 * (half_of(cr) >> 1) + ((half_of(cr) & 1) ? (early(cr) ? 3 : 4) : 0); };
  unsigned odd = 0;
  auto dup_of = [](bool isx) INL -> unsigned { return isx ? NKS * A_STEP : (unsigned)(ROUNDS * NKS) * B_STEP; };
  auto place = [&](bool isx, int col, int q4, int hs, unsigned& src, unsigned& dst) INL {
    const unsigned ks = (unsigned)((hs >> 1) * 8 + rank), kk = (unsigned)((hs & 1) * KHS);
    if (isx) { src = (unsigned)((col * 256 + q4 * 64) * KSH + hs * 32) * 4u; dst = ks * A_STEP + (unsigned)col * PIECE + kk + (unsigned)((q4 >> 1) * HTB + (q4 & 1) * 4 * FRS); }
    else {
      const int rr = col / RCOLS, cc = col - rr * RCOLS;
      const int wcol = col < NRC ? ((rr + ((hs >> 1) >= phase_of(cc) ? 1 : 0)) & 3) * RCOLS + cc : col;
      src = (unsigned)(hs * 32 * GN + wcol * 256 + q4 * 64) * 4u;
      dst = A_BYTES + (col < NRC ? ((unsigned)rr * NKS + ks) * B_STEP + (unsigned)cc * PIECE : B_BYTES + ks * PIECE) + kk + (unsigned)(q4 * 2 * FRS);
    }
    if (odd && ks < (unsigned)NDUP && (isx || col < RCOLS)) dst += dup_of(isx);
  };
  auto load_unit = [&](f32x4 (&v)[8], bool valid, bool isx, unsigned src, auto aux) INL {
    const __amdgpu_buffer_rsrc_t rs = __builtin_amdgcn_make_buffer_rsrc((void*)(isx ? p.x : p.w), 0, valid ? (isx ? GM * KSH * 4 : KSH * GN * 4) : 0, 0x00020000);
    const unsigned vv = isx ? vx : vw;
#pragma unroll
    for (int j = 0; j < 8; ++j)
      v[j] = __builtin_bit_cast(f32x4, __builtin_amdgcn_raw_buffer_load_b128(rs, vv, src + (isx ? (unsigned)((j >> 1) * 16 * KSH + (j & 1) * 16) * 4u : (unsigned)(((j >> 2) * 16 + (j & 3)) * GN) * 4u), decltype(aux)::value));
  };
  auto pack_unit = [&](const f32x4 (&v)[8], float m, bool isx, unsigned dst, u32x4 (&o)[4], unsigned (&oo)[4]) INL {
#pragma unroll
    for (int i = 0; i < 4; ++i) {
      o[i] = isx ? pack8(m, v[2 * i][0], v[2 * i][1], v[2 * i][2], v[2 * i][3], v[2 * i + 1][0], v[2 * i + 1][1], v[2 * i + 1][2], v[2 * i + 1][3])
                 : pack8(m, v[0][i], v[1][i], v[2][i], v[3][i], v[4][i], v[5][i], v[6][i], v[7][i]);
      oo[i] = vd + dst + (unsigned)(isx ? i * FRS : (i >> 1) * HTB + (i & 1) * FRS);
    }
  };

  auto decode = [&](unsigned u, bool& isx, unsigned& src, unsigned& dst) INL {
    int sg, i; isx = false;
    if (u < (unsigned)U1) { sg = u / 240u; i = u % 240u; isx = i < 128; if (!isx) i -= 128; }
    else { const unsigned v = u - U1; sg = 8 + v / 112u; i = v % 112u; }
    place(isx, isx ? i >> 3 : (sg >> 3) * RCOLS + ((i >> 3) + RCOLS - rot(rank)) % RCOLS, i & 3, (sg & 7) * 2 + ((i >> 2) & 1), src, dst);
  };
  const int aid = wid == 0 ? b : NAGENT + b;
  f32x4 v[8]; bool isx = false; unsigned src = 0, dst = 0;
  if (li < 4 && wid == 0) { decode((unsigned)b, isx, src, dst); load_unit(v, true, isx, src, IC<0>{}); }
  if (tid == 0) {
    const unsigned c = __hip_atomic_fetch_add((unsigned*)(myws + WS_CTR + b * 64), 1u, __ATOMIC_RELAXED, __HIP_MEMORY_SCOPE_AGENT) + 1u;
    *(LAS unsigned*)(lds + LDS_BAD + 4) = c; *(LAS unsigned*)(lds + LDS_BAD) = 0u;
  }
  *(LAS unsigned*)(lds + LDS_FLAG + tid * 4) = 0u;
  WAIT_L(0); BAR; asm volatile("" ::: "memory");
  const unsigned call = __builtin_amdgcn_readfirstlane(*(LAS unsigned*)(lds + LDS_BAD + 4));
  const unsigned seq0 = call * SEQ;
  constexpr size_t stg = WS_STG;
  odd = call & 1u;
  if (odd) dst += dup_of(isx);
  const char* flagw = myws + WS_FLAG + (lane & 7) * 128;

  unsigned* prog = (unsigned*)(myws + WS_PROG);
  unsigned* ticket = (unsigned*)(myws + WS_TICKET + (call & 3u) * 128u);
  auto draw = [&](unsigned* ctr, bool real) INL -> unsigned {
    return (unsigned)__builtin_amdgcn_raw_ptr_buffer_atomic_add_i32(1, __builtin_amdgcn_make_buffer_rsrc(ctr, 0, real ? 4 : 0, 0x00020000), (int)vd, 0, 0); };
  auto progress = [&](unsigned val) INL { __builtin_amdgcn_raw_buffer_store_b32(val, __builtin_amdgcn_make_buffer_rsrc(prog + aid, 0, 4, 0x00020000), vd, 0, 16); };
  auto read_go = [&]() INL -> unsigned { return __builtin_amdgcn_readfirstlane(__hip_atomic_load((unsigned*)(myws + WS_GO + 4), __ATOMIC_RELAXED, __HIP_MEMORY_SCOPE_AGENT)); };
  bool copy = false, staged = false;
#define load_send(valid) load_send_(v, valid, copy, isx, src, dst, myws + WS_STG, p.x, p.w, lane)
  auto send_ops = [&](unsigned go, unsigned cur, unsigned tnext, unsigned limit) INL {
    unsigned* ctr = ticket + (go >> 31) * 8; const unsigned base = (go >> 31) ? 0u : 2u * NAGENT;
    const float mx = pow2f((int)((go >> 12) & 0xfffu) - 256), mw = pow2f((int)(go & 0xfffu) - 256);
    __amdgpu_buffer_rsrc_t rd[NR - 1];
#pragma unroll
    for (int d = 0; d < NR - 1; ++d) rd[d] = __builtin_amdgcn_make_buffer_rsrc(p.ws[(rank + 1 + (d + b + wid) % (NR - 1)) & 7] + stg, 0, STG_BYTES, 0x00020000);
    bool real = true, open = go >> 31;
#pragma unroll
    for (int j = 0; j < 8; ++j) asm volatile("" : "+v"(v[j]));
    asm volatile("" : "+v"(tnext));
    while (cur < (unsigned)NUNITS) {
      u32x4 o[4]; unsigned oo[4];
      pack_unit(v, isx ? mx : mw, isx, dst, o, oo);
      if (copy) {
#pragma unroll
        for (int i = 0; i < 4; ++i) o[i] = __builtin_bit_cast(u32x4, v[i]);
      }
      const unsigned raw = __builtin_amdgcn_readfirstlane(tnext) + base, nxt = real ? raw & 0xffffffu : 0xffffffffu;
      open = open || (real && ((raw >> 24) & 1u));
      if (!open && cur >= (unsigned)(NDUP / 8 * 240)) { wave_wait<true>(myws + WS_START + (lane & 7) * 128, false, call - 1u, tmo, call); open = true; }
      SCHED;
      decode(nxt < (unsigned)NUNITS ? nxt : 0u, isx, src, dst);
      staged = staged || (real && (raw >> 25));
      copy = staged && nxt >= (unsigned)U1;
      load_send(nxt < (unsigned)NUNITS);
      real = nxt < limit; tnext = draw(ctr, real);
      SCHED;
      SEND_UNIT(9, true);
      cur = nxt;
    }
    WAIT_V(0);
  };
  auto join_ops = [&](unsigned go, unsigned limit) INL {
    unsigned* ctr = ticket + (go >> 31) * 8;
    const unsigned cur = (__builtin_amdgcn_readfirstlane(draw(ctr, true)) & 0xffffffu) + ((go >> 31) ? 0u : 2u * NAGENT);
    progress(seq0 + (cur < (unsigned)NUNITS ? cur : (unsigned)NUNITS));
    decode(cur < (unsigned)NUNITS ? cur : 0u, isx, src, dst); copy = false; load_send(cur < (unsigned)NUNITS);
    send_ops(go, cur, draw(ctr, true), limit);
  };
  auto send_out = [&]() INL {
    unsigned cur = draw(ticket + 16, true), tnext = draw(ticket + 16, true);
    const bool ok = wave_wait(myws + WS_PCNT, false, call * 16u - 1u, tmo, call);
    cur = __builtin_amdgcn_readfirstlane(cur);
    __amdgpu_buffer_rsrc_t rd[NR - 1];
#pragma unroll
    for (int d = 0; d < NR - 1; ++d) rd[d] = __builtin_amdgcn_make_buffer_rsrc(p.out[(rank + 1 + (d + b + wid) % (NR - 1)) & 7], 0, GM * GN * 4, 0x00020000);
    const unsigned vb = vd + (unsigned)((NRC + rank) * 256 * 4);
    auto orow = [](unsigned u, unsigned i) INL -> unsigned { return ((u * 37u) & 1023u) * 4u + i; };
    auto load_out = [&](unsigned u) INL {
      const __amdgpu_buffer_rsrc_t rs = __builtin_amdgcn_make_buffer_rsrc(p.out[rank], 0, u < (unsigned)NOUT ? GM * GN * 4 : 0, 0x00020000);
#pragma unroll
      for (int i = 0; i < 4; ++i) v[i] = __builtin_bit_cast(f32x4, __builtin_amdgcn_raw_buffer_load_b128(rs, vb, orow(u, i) * (unsigned)(GN * 4), 16));
    };
    load_out(cur);
    if (li >= 2) wave_wait<true>(myws + WS_SENT, false, seq0 + (unsigned)(NUNITS - PSTART), tmo, call);
#pragma unroll
    for (int j = 0; j < 4; ++j) asm volatile("" : "+v"(v[j]));
    asm volatile("" : "+v"(tnext));
    bool real = true;
    while (cur < (unsigned)NOUT) {
      u32x4 o[4]; unsigned oo[4];
#pragma unroll
      for (int i = 0; i < 4; ++i) { o[i] = __builtin_bit_cast(u32x4, v[i]); if (!ok) o[i] = (u32x4){0x7fc00000u, 0x7fc00000u, 0x7fc00000u, 0x7fc00000u}; oo[i] = vb + orow(cur, i) * (unsigned)(GN * 4); asm volatile("" : "+v"(o[i])); }
      const unsigned nxt = real ? __builtin_amdgcn_readfirstlane(tnext) : 0xffffffffu;
      SCHED;
      load_out(nxt);
      real = nxt < (unsigned)NOUT; tnext = draw(ticket + 16, real);
      SCHED;
      SEND_UNIT(5, false);
      cur = nxt;
    }
    WAIT_V(0);
    if (lane < NR) __hip_atomic_fetch_add((unsigned*)(p.ws[lane] + WS_DONE + rank * 128), 1u, __ATOMIC_RELAXED, __HIP_MEMORY_SCOPE_SYSTEM);
    STAMPW(11);
  };

  bool ready = false;
  if (li < 4) {
    if (wid == 0) {
      const unsigned limit = li < 2 ? (unsigned)NUNITS : li == 2 ? (unsigned)PLEAVE1 : (unsigned)PLEAVE;
      send_ops((unsigned)(SHIFT0 + 256) << 12 | (unsigned)(SHIFT0 + 256), (unsigned)b, (unsigned)(b - NAGENT), limit);
      unsigned go = 0;
      if (!staged) { wave_wait<true>(myws + WS_GO, false, call - 1u, tmo, call); go = read_go(); }
      if (go >> 31) { progress(seq0); WAIT_V(0); }
      if (lane == 0) __hip_atomic_fetch_add((unsigned*)(myws + WS_DCNT), 1u, __ATOMIC_RELAXED, __HIP_MEMORY_SCOPE_AGENT);
      if (go >> 31) { wave_wait<true>(myws + WS_DCNT, false, call * (unsigned)NAGENT - 1u, tmo, call); join_ops(go, limit); }
      progress(seq0 + (unsigned)(NUNITS + 8));
      STAMP(12);
      if (li < 2) send_out();
    } else if (wid == 2 && li < 2) {
      progress(seq0 + (unsigned)(SEQ - 1));
      wave_wait(myws + WS_SENT, false, seq0 + (unsigned)JOIN2, tmo, call);
      progress(seq0 + (unsigned)JOIN2);
      WAIT_V(0);
      join_ops(read_go(), (unsigned)NUNITS);
      progress(seq0 + (unsigned)(SEQ - 1));
    } else if (wid == 6) {
      wave_wait(myws + WS_SENT, false, seq0 + (unsigned)(PINAT), tmo, call);
      const float mw = pow2f((int)(read_go() & 0xfffu) - 256);
#pragma unroll 1
      for (int d = 1; d < NR; ++d) {
        const int pr = (rank + 1 + (d + b) % (NR - 1)) & 7;
        const __amdgpu_buffer_rsrc_t rp = __builtin_amdgcn_make_buffer_rsrc(p.ws[pr] + stg, 0, STG_BYTES, 0x00020000);
        f32x4 pv[2][8]; unsigned psrc, pdst[2];
#pragma unroll
        for (int e = 0; e < 2; ++e) { place(false, NRC + pr, (b >> 4) * 2 + e, b & 15, psrc, pdst[e]); load_unit(pv[e], true, false, psrc, IC<0>{}); }
#pragma unroll
        for (int e = 0; e < 2; ++e) { u32x4 o[4]; unsigned oo[4]; pack_unit(pv[e], mw, false, pdst[e], o, oo);
#pragma unroll
          for (int i = 0; i < 4; ++i) __builtin_amdgcn_raw_buffer_store_b128(o[i], rp, oo[i], 0, AUX_WT); }
      }
      WAIT_V(0);
      if (lane == 0) __hip_atomic_fetch_add((unsigned*)(myws + WS_PINC), 1u, __ATOMIC_RELAXED, __HIP_MEMORY_SCOPE_AGENT);
    } else if (b == 1 && wid == 7) {
      wave_wait(myws + WS_PKC + (lane & 7) * 128, false, call * (unsigned)NRW - 1u, tmo, call);
      wave_wait(myws + WS_SPD + (lane & 7) * 128, false, call - 1u, tmo, call);
      const unsigned f = __hip_atomic_load((unsigned*)(myws + WS_SPD + (lane & 7) * 128 + 4), __ATOMIC_RELAXED, __HIP_MEMORY_SCOPE_AGENT);
      const unsigned rl = lane & 7;
      unsigned kth = 0, task = rl, word = 0;
#pragma unroll
      for (int j = 0; j < 8; ++j) { const unsigned fj = __builtin_amdgcn_readlane(f, j), rj = __builtin_amdgcn_readlane(rl, j);
        if (((rj ^ rl) & 1u) == 0u && (fj == f ? j < (lane & 7) : (rl & 1u) ? fj < f : fj > f)) ++kth; }
#pragma unroll
      for (int j = 0; j < 8; ++j) { const unsigned fj = __builtin_amdgcn_readlane(f, j), rj = __builtin_amdgcn_readlane(rl, j), kj = __builtin_amdgcn_readlane(kth, j);
        if (((rj ^ rl) & 1u) && kj == kth && ((rl & 1u) ? fj * 1000u > f * SWAPT : f * 1000u > fj * SWAPT)) task = rj; }
#pragma unroll
      for (int j = 0; j < 8; ++j) word |= (unsigned)__builtin_amdgcn_readlane(task, j) << (3 * j);
      if (lane == 0) __hip_atomic_store((unsigned long long*)(myws + WS_MAP), (unsigned long long)word << 32 | call, __ATOMIC_RELAXED, __HIP_MEMORY_SCOPE_AGENT);
    } else if (b == 0 && wid == 7) {
      if (lane < 3) __hip_atomic_store((unsigned*)(myws + WS_TICKET + ((call + 2u) & 3u) * 128u) + lane * 8, 0u, __ATOMIC_RELAXED, __HIP_MEMORY_SCOPE_AGENT);
      unsigned* myflag = (lane & 7) == rank ? (unsigned*)(myws + WS_SENT) : (unsigned*)(p.ws[lane & 7] + WS_FLAG + rank * 128);
      const unsigned long long t0 = __builtin_amdgcn_s_memrealtime();
      { const unsigned st = __hip_atomic_load((unsigned*)(myws + WS_START + (lane & 7) * 128), __ATOMIC_RELAXED, __HIP_MEMORY_SCOPE_SYSTEM);
        const int ahead = __builtin_popcountll(__builtin_amdgcn_ballot_w64(lane < NR && st >= call));
        if (lane == 0) __hip_atomic_store((unsigned*)(myws + WS_LATE), ahead >= LATEN ? call : 0u, __ATOMIC_RELAXED, __HIP_MEMORY_SCOPE_AGENT);
        while (__builtin_amdgcn_s_memrealtime() - t0 < (unsigned long long)LATED) __builtin_amdgcn_s_sleep(2);
        if (lane < NR) __hip_atomic_store((unsigned*)(p.ws[lane] + WS_START + rank * 128), call, __ATOMIC_RELAXED, __HIP_MEMORY_SCOPE_SYSTEM); }
      bool ok = wave_wait<true>(myws + WS_START + (lane & 7) * 128, false, call - 1u, tmo, call);
      if (ok && lane == 0) __hip_atomic_fetch_add(ticket, 1u << 24, __ATOMIC_RELAXED, __HIP_MEMORY_SCOPE_AGENT);
      ok = wave_wait<true>(myws + WS_GO, false, call - 1u, tmo, call) && ok;
      const unsigned redo = read_go() >> 31;
      if (redo) {
        ok = wave_wait<true>(myws + WS_LCNT2, false, call * (unsigned)NRW - 1u, tmo, call) && ok;
        ok = wave_wait<true>(myws + WS_DCNT, false, call * (unsigned)NAGENT - 1u, tmo, call) && ok;
      }
      if (lane == 0) __hip_atomic_store((unsigned*)(myws + WS_FLAG + rank * 128), seq0 + (unsigned)(SEQ - 1), __ATOMIC_RELAXED, __HIP_MEMORY_SCOPE_SYSTEM);
      if (ok && lane == 0) __hip_atomic_fetch_add(ticket + redo * 8, 1u << 25, __ATOMIC_RELAXED, __HIP_MEMORY_SCOPE_AGENT);
      unsigned last = seq0; bool pin = false;
      const unsigned long long t1 = ok ? __builtin_amdgcn_s_memrealtime() : t0;
      while (last < seq0 + (unsigned)(NUNITS + 8) || !pin) {
        unsigned m = 0xffffffffu;
#pragma unroll
        for (int j = 0; j < (NPROG + 63) / 64; ++j) { const int a = j * 64 + lane; if (a < NPROG) { const unsigned x = __hip_atomic_load(prog + a, __ATOMIC_RELAXED, __HIP_MEMORY_SCOPE_AGENT); m = x < m ? x : m; } }
        if (!pin && last > seq0 + (unsigned)(PINAT) && (unsigned)__builtin_amdgcn_readfirstlane(__hip_atomic_load((unsigned*)(myws + WS_PINC), __ATOMIC_RELAXED, __HIP_MEMORY_SCOPE_AGENT)) >= call * 32u) {
          pin = true;
          if (lane < NR) __hip_atomic_store((unsigned*)(p.ws[lane] + WS_PIN + rank * 128), call, __ATOMIC_RELAXED, __HIP_MEMORY_SCOPE_SYSTEM);
        }
#pragma unroll
        for (int sft = 32; sft >= 1; sft >>= 1) { const unsigned x = __shfl_xor(m, sft); m = x < m ? x : m; }
        m = __builtin_amdgcn_readfirstlane(m);
        if (m > last) {
          auto crossed = [&](int x) INL -> bool { return last <= seq0 + (unsigned)x && m > seq0 + (unsigned)x; };
          const bool own = crossed(PINAT) || crossed(JOIN2) || crossed(NUNITS - PSTART);
          if (lane < NR && (own || lane != rank)) __hip_atomic_store(myflag, m, __ATOMIC_RELAXED, __HIP_MEMORY_SCOPE_SYSTEM);
          last = m;
        }
        if (__builtin_amdgcn_s_memrealtime() - t1 > DEADLINE) { __hip_atomic_store(tmo, call, __ATOMIC_RELAXED, __HIP_MEMORY_SCOPE_AGENT); ok = false; break; }
      }
      ok = wave_wait<true>(myws + WS_DONE + (lane & 7) * 128, false, call * (unsigned)NAGENT - 1u, tmo, call) && ok;
      ok = ok && __hip_atomic_load(tmo, __ATOMIC_RELAXED, __HIP_MEMORY_SCOPE_AGENT) != call;
      if (!ok && lane == 0) __hip_atomic_store((unsigned*)p.out[rank] + (NRC + rank) * 256, 0x7fc00000u, __ATOMIC_RELAXED, __HIP_MEMORY_SCOPE_SYSTEM);
    }
    if (li < 2) return;
  } else {
    const __amdgpu_buffer_rsrc_t rl = __builtin_amdgcn_make_buffer_rsrc(myws + stg, 0, STG_BYTES, 0x00020000);
    { const unsigned long long t0 = __builtin_amdgcn_s_memrealtime(); while (__builtin_amdgcn_s_memrealtime() - t0 < (unsigned long long)(PREDELAY * (li - 3))) __builtin_amdgcn_s_sleep(4); }
    float mx = pow2f(SHIFT0), mw = mx;
    ready = true;
    for (int pass = 0; pass < 2; ++pass) {
      unsigned ax = 0, aw = 0;
      for (int j = b - 32; j < 640; j += NRW) {
        isx = j < 128;
        if (j < 128) place(true, 2 * (j & 7) + (wid >> 2), wid & 3, j >> 3, src, dst);
        else if (j < 576) place(false, 2 * ((j - 128) % 28) + (wid >> 2), wid & 3, (j - 128) / 28, src, dst);
        else place(false, NRC + ((j - 576) >> 3), wid & 3, 2 * (j & 7) + (wid >> 2), src, dst);
        if (j >= 128 && j < 576 && (j - 128) % 28 >= 7) load_unit(v, true, isx, src, IC<AUX_NT>{});
        else load_unit(v, true, isx, src, IC<0>{});
        unsigned am = 0;
#pragma unroll
        for (int q = 0; q < 32; ++q) { unsigned a = __builtin_bit_cast(unsigned, v[q >> 2][q & 3]) & 0x7fffffffu; a = a < 0x7f800000u ? a : 0u; am = a > am ? a : am; }
        if (isx) ax = am > ax ? am : ax; else aw = am > aw ? am : aw;
        if (j < 576 || ((j - 576) >> 3) == rank) {
          u32x4 o[4]; unsigned oo[4]; pack_unit(v, isx ? mx : mw, isx, dst, o, oo);
#pragma unroll
          for (int i = 0; i < 4; ++i) __builtin_amdgcn_raw_buffer_store_b128(o[i], rl, oo[i], 0, AUX_WT);
        }
      }
      if (pass == 0) {
#pragma unroll
        for (int sft = 32; sft >= 1; sft >>= 1) { const unsigned a = __shfl_xor(ax, sft), c = __shfl_xor(aw, sft); ax = a > ax ? a : ax; aw = c > aw ? c : aw; }
        if (lane == 0) { *(LAS unsigned*)(lds + LDS_BAD + 32 + wid * 8) = ax; *(LAS unsigned*)(lds + LDS_BAD + 36 + wid * 8) = aw; }
      }
      WAIT_V(0); WAIT_L(0); BAR; asm volatile("" ::: "memory");
      if (pass == 1) break;
      if (wid == 0) {
        if (lane < 2) {
          unsigned a = 0;
#pragma unroll
          for (int q = 0; q < 8; ++q) { const unsigned c = *(LAS unsigned*)(lds + LDS_BAD + 32 + q * 8 + lane * 4); a = c > a ? c : a; }
          __hip_atomic_fetch_max((unsigned long long*)(myws + WS_LMAX) + lane, (unsigned long long)call << 32 | a, __ATOMIC_RELAXED, __HIP_MEMORY_SCOPE_AGENT);
        }
        WAIT_V(0);
        if ((unsigned)__builtin_amdgcn_readfirstlane(draw((unsigned*)(myws + WS_LCNT), true)) == call * (unsigned)NRW - 1u) {
          const unsigned long long mm = __hip_atomic_load((unsigned long long*)(myws + WS_LMAX) + (lane & 1), __ATOMIC_RELAXED, __HIP_MEMORY_SCOPE_AGENT);
          const unsigned e1 = (unsigned)mm ? 1u + ((unsigned)mm >> 23) : 0u;
          const unsigned e12 = (unsigned)__builtin_amdgcn_readlane(e1, 0) << 16 | (unsigned)__builtin_amdgcn_readlane(e1, 1);
          if (lane < NR) __hip_atomic_store((unsigned long long*)(p.ws[lane] + WS_AMX + rank * 128), (unsigned long long)e12 << 32 | call, __ATOMIC_RELAXED, __HIP_MEMORY_SCOPE_SYSTEM);
          wave_wait<true>(myws + WS_AMX + (lane & 7) * 128, false, call - 1u, tmo, call);
          const unsigned e2 = __hip_atomic_load((unsigned*)(myws + WS_AMX + (lane & 7) * 128 + 4), __ATOMIC_RELAXED, __HIP_MEMORY_SCOPE_SYSTEM);
          unsigned gx = e2 >> 16, gw = e2 & 0xffffu;
#pragma unroll
          for (int sft = 4; sft >= 1; sft >>= 1) { const unsigned a = __shfl_xor(gx, sft), c = __shfl_xor(gw, sft); gx = a > gx ? a : gx; gw = c > gw ? c : gw; }
          auto shift_of = [](unsigned g) INL -> int { const int e = (int)g - 128; return (g == 0u || (e >= -SHIFT0 && e < 15 - SHIFT0)) ? SHIFT0 : (14 - e > 126 ? 126 : 14 - e); };
          const int sx = shift_of(__builtin_amdgcn_readfirstlane(gx)), sw = shift_of(__builtin_amdgcn_readfirstlane(gw));
          const bool redo = sx != SHIFT0 || sw != SHIFT0;
          const bool late = __hip_atomic_load((unsigned*)(myws + WS_LATE), __ATOMIC_RELAXED, __HIP_MEMORY_SCOPE_AGENT) == call;
          if (lane == 0) {
            __hip_atomic_store((unsigned long long*)(myws + WS_GO), (unsigned long long)((redo ? 0x80000000u : 0u) | (late ? 0x40000000u : 0u) | (unsigned)(sx + 256) << 12 | (unsigned)(sw + 256)) << 32 | call, __ATOMIC_RELAXED, __HIP_MEMORY_SCOPE_AGENT);
            if (redo) __hip_atomic_fetch_add(ticket, 1u << 20, __ATOMIC_RELAXED, __HIP_MEMORY_SCOPE_AGENT);
          }
        }
        if (!wave_wait(myws + WS_GO, false, call - 1u, tmo, call)) *(LAS unsigned*)(lds + LDS_BAD) = 1u;
        *(LAS unsigned*)(lds + LDS_BAD + 8) = read_go();
        WAIT_L(0);
      }
      asm volatile("" ::: "memory"); BAR; asm volatile("" ::: "memory");
      const unsigned go = __builtin_amdgcn_readfirstlane(*(LAS unsigned*)(lds + LDS_BAD + 8));
      if (!(go >> 31)) break;
      mx = pow2f((int)((go >> 12) & 0xfffu) - 256); mw = pow2f((int)(go & 0xfffu) - 256); ready = false;
    }
    if (tid == 0) __hip_atomic_fetch_add((unsigned*)(myws + WS_LCNT2), 1u, __ATOMIC_RELAXED, __HIP_MEMORY_SCOPE_AGENT);
    STAMP(13);
  }

  const bool isP = li < 4;
  const int wr = wid >> 2, wc = wid & 3;
  const int lr = li - 4;
  const int role = xcd;
  const int mt = isP ? xcd * 2 + (li - 2) : (role >> 1) * 4 + (lr & 3);
  const int cr = isP ? 0 : (role & 1) + 2 * (lr >> 2);
  const unsigned voff = (unsigned)(wid * 2048 + lane * 16), ldsw = (unsigned)wid * 2048u;
  const char* fsrc = myws + WS_FLAG + wid * 128;
  const __amdgpu_buffer_rsrc_t ra = __builtin_amdgcn_make_buffer_rsrc(myws + stg, 0, A_BYTES, 0x00020000);
  const __amdgpu_buffer_rsrc_t rb = __builtin_amdgcn_make_buffer_rsrc(myws + stg + A_BYTES + (isP ? B_BYTES : 0u), 0, isP ? P_BYTES : B_BYTES, 0x00020000);
  const unsigned bstep = isP ? (unsigned)PIECE : B_STEP;
#define STAGE(ldsoff, rs, so) do { \
    __builtin_amdgcn_raw_ptr_buffer_load_lds(rs, (LAS void*)(lds + (ldsoff) + ldsw), 16, voff, (so), 0, AUX_SC1); \
    __builtin_amdgcn_raw_ptr_buffer_load_lds(rs, (LAS void*)(lds + (ldsoff) + ldsw), 16, voff, (so), 1024, AUX_SC1); } while (0)
#define ST1(ldsoff, rs, so, imm) do { __builtin_amdgcn_raw_ptr_buffer_load_lds(rs, (LAS void*)(lds + (ldsoff) + ldsw), 16, voff, (so), imm, AUX_SC1); SCHED; } while (0)
#define RA(h, m) do { _Pragma("unroll") for (int k = 0; k < 2; ++k) At[m][k] = *(const LAS f16x8*)((LAS char*)(size_t)va + (h) * HTB + (m) * FRS + k * KHS); SCHED; } while (0)
#define FLAGDMA() __builtin_amdgcn_global_load_lds((const unsigned*)fsrc, (LAS unsigned*)(lds + LDS_FLAG + wid * 256), 4, 0, AUX_SC1)
#define LDA(dst, h) do { _Pragma("unroll") for (int m = 0; m < 4; ++m) _Pragma("unroll") for (int k = 0; k < 2; ++k) dst[m][k] = *(const LAS f16x8*)((LAS char*)(size_t)va + (h) * HTB + m * FRS + k * KHS); } while (0)
#define LDB(dst, h) do { _Pragma("unroll") for (int n = 0; n < 2; ++n) _Pragma("unroll") for (int k = 0; k < 2; ++k) dst[n][k] = *(const LAS f16x8*)((LAS char*)(size_t)vb + (h) * HTB + n * FRS + k * KHS); } while (0)
#define ACC(ai, nn, m) (128 + (((ai) * 4 + (nn)) * 4 + (m)) * 4)
#define MMA2(ai) do { MMA2_(ai); BAR; SCHED; } while (0)
#define MMA2_(ai) do { __builtin_amdgcn_s_waitcnt(0xC07F); SCHED; BAR; SCHED; __builtin_amdgcn_s_setprio(1); SCHED; \
    _Pragma("unroll") for (int m = 0; m < 4; ++m) _Pragma("unroll") for (int q = 0; q < 4; ++q) { const int nn = (m & 1) ? 3 - q : q; \
      asm volatile("v_mfma_f32_16x16x32_f16 v[%c0:%c1], %2, %3, v[%c0:%c1]\n\tv_mfma_f32_16x16x32_f16 v[%c0:%c1], %4, %5, v[%c0:%c1]" \
          :: "i"(ACC(ai, nn, m)), "i"(ACC(ai, nn, m) + 3), "v"(At[m][0]), "v"((nn >> 1) ? B1[nn & 1][0] : B0[nn & 1][0]), "v"(At[m][1]), "v"((nn >> 1) ? B1[nn & 1][1] : B0[nn & 1][1])); } \
    SCHED; __builtin_amdgcn_s_setprio(0); SCHED; } while (0)
#define ZERO_ACC() do { _Pragma("unroll") for (int i_ = 128; i_ < 256; i_ += 2) asm volatile("v_mov_b64 v[%c0:%c1], 0" :: "i"(i_), "i"(i_ + 1)); } while (0)
#define ACC_ROW(dst, ai, m, j) do { float t0_, t1_, t2_, t3_; \
    asm volatile("v_ldexp_f32 %0, v[%c4], %8\n\tv_ldexp_f32 %1, v[%c5], %8\n\tv_ldexp_f32 %2, v[%c6], %8\n\tv_ldexp_f32 %3, v[%c7], %8" \
        : "=&v"(t0_), "=&v"(t1_), "=&v"(t2_), "=&v"(t3_) : "i"(ACC(ai, 0, m) + (j)), "i"(ACC(ai, 1, m) + (j)), "i"(ACC(ai, 2, m) + (j)), "i"(ACC(ai, 3, m) + (j)), "v"(oexp)); \
    dst = (f32x4){t0_, t1_, t2_, t3_}; } while (0)
  asm volatile("" ::: "v128","v129","v130","v131","v132","v133","v134","v135","v136","v137","v138","v139","v140","v141","v142","v143","v144","v145","v146","v147","v148","v149","v150","v151","v152","v153","v154","v155","v156","v157","v158","v159","v160","v161","v162","v163","v164","v165","v166","v167","v168","v169","v170","v171","v172","v173","v174","v175","v176","v177","v178","v179","v180","v181","v182","v183","v184","v185","v186","v187","v188","v189","v190","v191","v192","v193","v194","v195","v196","v197","v198","v199","v200","v201","v202","v203","v204","v205","v206","v207","v208","v209","v210","v211","v212","v213","v214","v215","v216","v217","v218","v219","v220","v221","v222","v223","v224","v225","v226","v227","v228","v229","v230","v231","v232","v233","v234","v235","v236","v237","v238","v239","v240","v241","v242","v243","v244","v245","v246","v247","v248","v249","v250","v251","v252","v253","v254","v255");
  ZERO_ACC();
  f16x8 At[4][2], B0[2][2], B1[2][2];
  const int ph = isP ? 0 : phase_of(cr);
  if (!ready) {
    if (wid == 0) {
      if (isP && !wave_wait(myws + WS_PIN + (lane & 7) * 128, false, call - 1u, tmo, call)) *(LAS unsigned*)(lds + LDS_BAD) = 1u;
      if (!wave_wait(isP ? flagw : myws + WS_FLAG + rank * 128, false, seq0 + (unsigned)(U1 - 1), tmo, call)) *(LAS unsigned*)(lds + LDS_BAD) = 1u;
      *(LAS unsigned*)(lds + LDS_BAD + 8) = read_go(); WAIT_L(0);
    }
    asm volatile("" ::: "memory"); BAR; asm volatile("" ::: "memory");
  }
  const unsigned gow = __builtin_amdgcn_readfirstlane(*(LAS unsigned*)(lds + LDS_BAD + 8));
  const int oexp = 512 - (int)((gow >> 12) & 0xfffu) - (int)(gow & 0xfffu);
  const int NL = (gow >> 30) & 1u ? 0 : 1;
  int rolex = role, mtx = mt, crx = cr;
  const int E0 = 8 + rho0_of(cr);
  int Tend = isP ? 64 : 256;
  unsigned tA, tB, tN;
  auto steps = [&](int t0) INL {
    const int t = t0 + lane, u = t - E0, c = u < 192 ? cr : crx, w = u - 8 * NL - 8;
    const int z = w - 56, zn = 128 - 64 * NL;
    const bool inl = z >= 0 && z < zn;
    const bool local = t < 8 || (u >= 0 && w < 0) || (inl && (z & 7) == 0);
    const int rho = u < 0 ? t - 8 : u >= 192 ? rho0_of(crx) + u - 24 : E0 - 8 + w - (z < 0 ? 0 : inl ? (z >> 3) + 1 : zn >> 3), pos = rho % 7;
    const int sgm = !local ? rho / 7 : t < 8 ? (t < ph ? t : t + 24) : inl ? ph + 8 + (z >> 3) : u < 8 * NL ? ph + 16 + u : ph + (u & 7);
    const int f = first_src(c), sr = local ? rank : (f + pos + (pos >= ((rank - f) & 7) ? 1 : 0)) & 7;
    const unsigned ix = isP ? (unsigned)t : (unsigned)(sgm * 8 + sr);
    tA = t < Tend ? (unsigned)(u < 192 ? mt : mtx) * PIECE + (ix & 63u) * A_STEP + (odd && (ix & 63u) < (unsigned)NDUP ? dup_of(true) : 0u) : 0x80000000u;
    tB = t < Tend ? (unsigned)c * PIECE + ix * bstep + (odd && !isP && ix < (unsigned)NDUP ? dup_of(false) : 0u) : 0x80000000u;
    tN = (isP || t >= Tend || local) ? 0u : seq0 + (unsigned)((sgm < 8 ? 240 * sgm + 128 : U1 - 8 * 112 + 112 * sgm) + 8 * ((c + rot(sr)) % RCOLS) + 8 + sr);
  };
  steps(0);
  STAMP(1);
  __builtin_amdgcn_s_waitcnt(0x0F70); SCHED;
  unsigned par = 0, l1 = PIECE + HTB, a1, fl = 0;
  unsigned a2 = __builtin_amdgcn_readlane(tA, 2), b2 = __builtin_amdgcn_readlane(tB, 2), need = 0;
  unsigned va = (unsigned)(size_t)lds + wr * 4 * FRS + lane * 16, vb = (unsigned)(size_t)lds + 2 * PIECE + wc * 2 * FRS + lane * 16;
  LAS unsigned* fimg = (LAS unsigned*)(lds + LDS_FLAG + (lane & 7) * 256); asm volatile("" : "+v"(fimg));
  { const unsigned a0 = __builtin_amdgcn_readlane(tA, 0), b0 = __builtin_amdgcn_readlane(tB, 0), b1 = __builtin_amdgcn_readlane(tB, 1); a1 = __builtin_amdgcn_readlane(tA, 1);
    STAGE(2 * PIECE, rb, b0); STAGE(0, ra, a0); STAGE(2 * PIECE + HTB, rb, b0 + HTB); STAGE(HTB, ra, a0 + HTB);
    if (wr == 1) BAR;
    WAIT_V(2); BAR;
    FLAGDMA();
    STAGE(3 * PIECE, rb, b1); STAGE(PIECE, ra, a1); STAGE(3 * PIECE + HTB, rb, b1 + HTB);
    a1 += HTB;
    BAR; }
  const unsigned vpk = (unsigned)tid * 16u;
  const unsigned vring = (unsigned)(size_t)lds + LDS_RING + wid * 1024 + lane * 16;
#define SLOW_T0
#define SLOW_T1
#define STEP_STAMP
#define KLOOP(ADD, LIM) for (;;) { \
      f32x4 pt; \
      ST1(l1, ra, a1, 0); LDB(B0, 0); SCHED; ST1(l1, ra, a1, 1024); LDB(B1, 1); SCHED; \
      if (ADD) { __builtin_amdgcn_raw_ptr_buffer_load_lds(rpk, (LAS void*)(lds + LDS_RING + rs_ + wid * 1024), 16, vpk, (unsigned)(li3 - a0_) * 8192u, 0, AUX_SC1 | AUX_NT); SCHED; \
        rs_ = rs_ == 16384u ? 0u : rs_ + 8192u; pt = *(const LAS f32x4*)(size_t)(vring + rs_); } \
      else FLAGDMA(); \
      SCHED; LDA(At, 0); WAIT_V(10); \
      MMA2(0); \
      if ((unsigned)__builtin_amdgcn_readlane(fl, need & 63u) < need) { SLOW_T0; \
        const unsigned f2 = *(const volatile LAS unsigned*)fimg; fl = f2 > fl ? f2 : fl;     \
        if ((unsigned)__builtin_amdgcn_readlane(fl, need & 63u) < need) { \
          if (!wave_wait(flagw, (unsigned)(lane & 7) != (need & 7u), need - 1u, tmo, call, &fl)) *(LAS unsigned*)(lds + LDS_BAD) = 1u; \
          *(volatile LAS unsigned*)fimg = fl; } \
        SLOW_T1; } \
      ST1(2 * PIECE + par, rb, b2, 0); RA(1, 0); ST1(2 * PIECE + par, rb, b2, 1024); RA(1, 1); \
      ST1(par, ra, a2, 0); RA(1, 2); ST1(par, ra, a2, 1024); RA(1, 3); \
      const unsigned fn = ADD ? 0u : *fimg; \
      if (ADD && li3 - a0_ >= 2) { unsigned keep_; \
        asm volatile("s_mov_b32 %0, m0\n\ts_set_gpr_idx_on %1, gpr_idx(SRC0,DST)\n\tv_add_f32 v128, v128, %2\n\tv_add_f32 v129, v129, %3\n\tv_add_f32 v130, v130, %4\n\tv_add_f32 v131, v131, %5\n\ts_set_gpr_idx_off\n\ts_mov_b32 m0, %0" \
            : "=&s"(keep_) : "s"(((li3 - a0_ - 2) & 31) * 4), "v"(pt[0]), "v"(pt[1]), "v"(pt[2]), "v"(pt[3])); } \
      SCHED; \
      ST1(2 * PIECE + par + HTB, rb, b2 + HTB, 0); \
      l1 = par + HTB; a1 = a2 + HTB; va ^= PIECE; vb ^= PIECE; \
      a2 = __builtin_amdgcn_readlane(tA, li3); need = __builtin_amdgcn_readlane(tN, li3); \
      asm volatile("" : "+v"(va), "+v"(vb), "+s"(a2), "+s"(need), "+s"(l1), "+s"(a1)); SCHED; \
      ST1(2 * PIECE + par + HTB, rb, b2 + HTB, 1024); \
      par ^= PIECE; b2 = __builtin_amdgcn_readlane(tB, li3); ++li3; \
      STEP_STAMP \
      fl = fn > fl ? fn : fl; \
      asm volatile("" : "+s"(par), "+s"(li3), "+s"(b2), "+v"(fl)); \
      WAIT_V(9); \
      MMA2_(1); \
      if (li3 >= (LIM)) break; \
      BAR; SCHED; }
  int t = 0;
  steps(3);
  float* const myout = p.out[rank] + (size_t)(wr * 64) * GN + wc * 64;
  for (int sg = isP ? 4 : -NL; sg < 5; ++sg) {
    const int Ts = sg == 4 ? Tend : sg == -NL ? E0 : t + (sg <= 0 ? 8 : (NL && sg == 3) ? 56 : 64);
    const int slot = sg <= 0 ? sg + NL : (sg == 4 ? 0 : 1);
    char* pk = myws + WS_PARK + (size_t)(slot * 256 + (sg == 4 ? rolex : role) * 32 + li) * 262144;
    const __amdgpu_buffer_rsrc_t rpk = __builtin_amdgcn_make_buffer_rsrc(pk, 0, 262144, 0x00020000);
    const int left = Ts - t; int li3 = 0;
    unsigned rs_ = 0;
    const bool add = !isP && (sg == 4 || (NL && sg == 3));
    const bool ref = sg == 2 && lr == ((role & 1) ? 4 : 0);
    const int a1_ = 29 - E0 - (sg == 4 ? ADDOFF : ADDOFF2),
              a0_ = ref ? 32 : (add && a1_ > 0) ? (a1_ < left - 35 ? a1_ : left - 35) : 0;
    const unsigned r0_ = (unsigned)__builtin_amdgcn_s_memrealtime(); __builtin_amdgcn_s_waitcnt(0xC07F);
    for (int part = a0_ ? 0 : 1; part < 2; ++part) {
      if (part == 1 && add) { KLOOP(true, a0_ + 34); BAR; SCHED; }
      const int lim_ = part ? left : a0_;
      KLOOP(false, lim_);
      if (!part) {
        BAR; SCHED;
        if (ref && tid == 0) __hip_atomic_store((unsigned long long*)(myws + WS_SPD + xcd * 128),
            (unsigned long long)((1u << 28) / __builtin_elementwise_max((unsigned)__builtin_amdgcn_s_memrealtime() - r0_, 256u)) << 32 | call, __ATOMIC_RELAXED, __HIP_MEMORY_SCOPE_AGENT);
      }
    }
    t = Ts;
    if (wr == 0) BAR;
    asm volatile("s_nop 15\n\ts_nop 15" ::: "memory");
    SCHED;
    STAMP(2 + 2 * r);
    if (sg <= 0) {
      unsigned vq = vpk; asm volatile("" : "+v"(vq));
      if (sg == -NL) {
#pragma unroll
      for (int i = 0; i < 32; ++i)
        asm volatile("s_nop 4\n\tglobal_store_dwordx4 %0, v[%c1:%c2], %3" " sc1" :: "v"(vq + i * 8192u), "i"(128 + 4 * i), "i"(128 + 4 * i + 3), "s"(pk) : "memory");
      } else {
#pragma unroll
      for (int i = 0; i < 32; ++i)
        asm volatile("s_nop 4\n\tglobal_store_dwordx4 %0, v[%c1:%c2], %3" " nt" :: "v"(vq + i * 8192u), "i"(128 + 4 * i), "i"(128 + 4 * i + 3), "s"(pk) : "memory");
      }
      asm volatile("s_nop 1");
      ZERO_ACC();
      steps(t + 3);
      if (sg == -NL) {
        WAIT_V(0);
        if (lane == 0) __hip_atomic_fetch_add((unsigned*)(myws + WS_PKC + xcd * 128), 1u, __ATOMIC_RELAXED, __HIP_MEMORY_SCOPE_AGENT);
      }
    } else {
      if (sg == 2) {
        unsigned mp = 0;
        if (!wave_wait(myws + WS_MAP + (lane & 1) * 4, lane & 1, call - 1u, tmo, call, &mp)) *(LAS unsigned*)(lds + LDS_BAD) = 1u;
        else rolex = (int)(((unsigned)__builtin_amdgcn_readlane(mp, 1) >> (3 * xcd)) & 7u);
        mtx = (rolex >> 1) * 4 + (lr & 3); crx = (rolex & 1) + 2 * (lr >> 2); Tend = E0 + 192 + 56 - rho0_of(crx);
      }
      if (__builtin_amdgcn_readfirstlane(*(const LAS unsigned*)(lds + LDS_BAD)) != 0u) {
#pragma unroll
        for (int i = 128; i < 256; ++i) asm volatile("v_mov_b32 v[%c0], 0x7fc00000" :: "i"(i));
      }
      const int nt = isP ? NRC + rank : sg == 4 ? crx : sg * RCOLS + cr;
      const __amdgpu_buffer_rsrc_t rc = __builtin_amdgcn_make_buffer_rsrc(myout + (size_t)((sg == 4 ? mtx : mt) * 256) * GN + nt * 256, 0, 192u * GN * 4u, 0x00020000);
      unsigned vo = (unsigned)((lane >> 4) * 4 * GN + (lane & 15) * 4) * 4u; asm volatile("" : "+v"(vo));
#pragma unroll
      for (int ai = 0; ai < 2; ++ai)
#pragma unroll
        for (int m = 0; m < 4; ++m) {
#pragma unroll
          for (int j = 0; j < 4; ++j) {
            f32x4 v4; ACC_ROW(v4, ai, m, j);
            __builtin_amdgcn_raw_buffer_store_b128(__builtin_bit_cast(u32x4, v4), rc, vo + (unsigned)((ai * 128 + m * 16 + j) * GN * 4), 0, AUX_NT);
          }
#pragma unroll
          for (int nn = 0; nn < 4; ++nn)
            asm volatile("v_mov_b64 v[%c0:%c1], 0\n\tv_mov_b64 v[%c2:%c3], 0" :: "i"(ACC(ai, nn, m)), "i"(ACC(ai, nn, m) + 1), "i"(ACC(ai, nn, m) + 2), "i"(ACC(ai, nn, m) + 3));
          if (ai == 0 && m == 1) {
            SCHED; steps(t + 3); asm volatile("" : "+v"(tA), "+v"(tB), "+v"(tN)); SCHED;
          }
        }
    }
    if (wr == 1) BAR;
    STAMP(3 + 2 * r);
    STAMPC(20 + r);
  }
  WAIT_V(0);
  STAMP(15);
  if (wr == 0) BAR;
  if (isP) {
    BAR; asm volatile("" ::: "memory");
    if (tid == 0) {
      __builtin_amdgcn_fence(__ATOMIC_RELEASE, "agent");
      WAIT_V(0);
      __hip_atomic_fetch_add((unsigned*)(myws + WS_PCNT), 1u, __ATOMIC_RELAXED, __HIP_MEMORY_SCOPE_AGENT);
    }
    STAMP(10);
    if (wid == 0) {
      if (li == 2) {
        progress(seq0 + (unsigned)PLEAVE1);
        WAIT_V(0);
        join_ops(gow, (unsigned)NUNITS);
        progress(seq0 + (unsigned)(NUNITS + 8));
      }
      send_out();
    }
  }
}
}

extern "C" void kernel_launch(int rank, int nranks, ncclComm_t comm, hipStream_t stream,
                              void* const* d_in, const int* in_sizes, int n_in,
                              void* d_out, int out_size, const int* in_dtypes, int out_dtype,
                              void* const* d_out_all, void* const* d_ws_all, size_t ws_bytes) {
  (void)comm; (void)in_dtypes; (void)out_dtype; (void)d_out;
  if (nranks != NR || (unsigned)rank >= (unsigned)NR || n_in < 2 || ws_bytes < WS_NEED || out_size != GM * GN ||
      in_sizes[0] != GM * KSH || in_sizes[1] != KSH * GN) { fprintf(stderr, "unexpected problem shape\n"); return; }
  static bool attr_set = false;
  if (!attr_set) { HIP_CHECK(hipFuncSetAttribute((const void*)fused_kernel, hipFuncAttributeMaxDynamicSharedMemorySize, LDS_BYTES)); attr_set = true; }
  Params p;
  p.x = (const float*)d_in[0]; p.w = (const float*)d_in[1]; p.rank = rank;
  for (int r = 0; r < NR; ++r) { p.out[r] = (float*)d_out_all[r]; p.ws[r] = (char*)d_ws_all[r]; }
  fused_kernel<<<256, 512, LDS_BYTES, stream>>>(p);
  HIP_CHECK(hipGetLastError());
}
```

```cpp
#include <hip/hip_runtime.h>
#include <rccl/rccl.h>
#include <cstdio>

#define HIP_CHECK(expr) do { hipError_t _e = (expr); if (_e != hipSuccess) { \
    fprintf(stderr, "HIP error %s at %s:%d\n", hipGetErrorString(_e), __FILE__, __LINE__); \
    return; } } while (0)

namespace {
#define LAS __attribute__((address_space(3)))
#define INL __attribute__((always_inline))
typedef _Float16 f16x8 __attribute__((ext_vector_type(8)));
typedef float f32x4 __attribute__((ext_vector_type(4)));
typedef unsigned u32x4 __attribute__((ext_vector_type(4)));

constexpr int NR = 8, GM = 4096, GN = 16384, KSH = 512;
constexpr int PIECE = 32768, HTB = 16384, NKS = 64;
constexpr int FRS = 2048, KHS = 1024;
constexpr int RCOLS = 14, ROUNDS = 4, NRC = RCOLS * ROUNDS;
constexpr unsigned A_STEP = 16u * PIECE, B_STEP = (unsigned)RCOLS * PIECE;
constexpr int NDUP = 16;
constexpr unsigned A_BYTES = (NKS + NDUP) * A_STEP, B_BYTES = (unsigned)(ROUNDS * NKS + NDUP) * B_STEP, P_BYTES = (unsigned)NKS * PIECE;
constexpr unsigned STG_BYTES = A_BYTES + B_BYTES + P_BYTES;
constexpr size_t WS_CTR = 0;
constexpr size_t WS_FLAG = 16384, WS_START = 18432, WS_PIN = 24576, WS_AMX = 28672, WS_DONE = 29696;
constexpr size_t WS_TMO = 20480, WS_TICKET = 21504, WS_PROG = 22528, WS_LCNT = 26624, WS_SENT = 26752, WS_PCNT = 26880, WS_PINC = 27008,
                 WS_LCNT2 = 27136, WS_DCNT = 27264, WS_LMAX = 27392, WS_GO = 27520, WS_LATE = 27648, WS_MAP = 27776;
constexpr size_t WS_PKC = 30720, WS_SPD = 31744;
constexpr size_t WS_STG = 1u << 20;
constexpr size_t WS_PARK = WS_STG + (size_t)STG_BYTES;
constexpr size_t WS_NEED = WS_PARK + 2 * 256 * (size_t)262144;
constexpr int PINAT = 2200;
constexpr int PLEAVE1 = 2600;
constexpr int PLEAVE = 3150;
constexpr int JOIN2 = 2600;
constexpr int PSTART = 200;
constexpr int PREDELAY = 40;
constexpr int ADDOFF = 8;
constexpr int ADDOFF2 = 3;
constexpr unsigned SWAPT = 1012u;
constexpr int POLLS = 32, LATEN = 4, LATED = 300;
constexpr int U1 = 8 * 240, NUNITS = U1 + 24 * 112, NOUT = GM / 4, SEQ = 8192, NAGENT = 32, NPROG = 48, NRW = 224;
constexpr int SHIFT0 = 8;
constexpr int LDS_FLAG = 131072, LDS_BAD = LDS_FLAG + 2048, LDS_RING = LDS_BAD + 128, LDS_BYTES = LDS_RING + 3 * 8192;
constexpr unsigned long long DEADLINE = 80000000ull;
constexpr int AUX_WT = 17, AUX_NT = 2, AUX_SC1 = 16;

template <int V> struct IC { static constexpr int value = V; };
struct Params { const float* x; const float* w; float* out[NR]; char* ws[NR]; int rank; };

#define WAIT_V(n) asm volatile("s_waitcnt vmcnt(" #n ")" ::: "memory")
#define WAIT_L(n) asm volatile("s_waitcnt lgkmcnt(" #n ")" ::: "memory")
#define BAR __builtin_amdgcn_s_barrier()
#define SCHED __builtin_amdgcn_sched_barrier(0)

template <bool FAST = false>
__device__ __forceinline__ bool wave_wait(const char* words, bool skip, unsigned need, unsigned* tmo, unsigned call, unsigned* seen = nullptr) {
  const unsigned long long t0 = __builtin_amdgcn_s_memrealtime();
  bool ok = true;
  for (unsigned n = 1;; ++n) {
    const unsigned v = (n & 7u) ? __hip_atomic_load((const unsigned*)words, __ATOMIC_RELAXED, __HIP_MEMORY_SCOPE_AGENT)
                                : __hip_atomic_load((const unsigned*)words, __ATOMIC_RELAXED, __HIP_MEMORY_SCOPE_SYSTEM);
    if (seen) *seen = v;
    if (__builtin_amdgcn_ballot_w64(skip || v > need) == ~0ull) break;
    if ((n & 7u) == 0u &&
        ((unsigned)__builtin_amdgcn_readfirstlane(__hip_atomic_load(tmo, __ATOMIC_RELAXED, __HIP_MEMORY_SCOPE_AGENT)) == call ||
         __builtin_amdgcn_s_memrealtime() - t0 > DEADLINE)) {
      __hip_atomic_store(tmo, call, __ATOMIC_RELAXED, __HIP_MEMORY_SCOPE_AGENT);
      ok = false; break;
    }
    __builtin_amdgcn_s_sleep(FAST ? 1 : POLLS);
  }
  __builtin_amdgcn_s_waitcnt(0xC07F);
  asm volatile("" ::: "memory");
  return ok;
}

__device__ __forceinline__ float pow2f(int e) { return __builtin_bit_cast(float, (unsigned)(e + 127) << 23); }
__device__ __forceinline__ u32x4 pack8(float m, float a0, float a1, float a2, float a3, float a4, float a5, float a6, float a7) {
  f16x8 h = {(_Float16)(a0 * m), (_Float16)(a1 * m), (_Float16)(a2 * m), (_Float16)(a3 * m), (_Float16)(a4 * m), (_Float16)(a5 * m), (_Float16)(a6 * m), (_Float16)(a7 * m)};
  return __builtin_bit_cast(u32x4, h);
}

__device__ __forceinline__ void load_send_(f32x4 (&v)[8], bool valid, bool copy, bool isx, unsigned src, unsigned dst,
                                           const void* stg, const void* xp, const void* wp, int lane) {
  const __amdgpu_buffer_rsrc_t rs = __builtin_amdgcn_make_buffer_rsrc((void*)(copy ? stg : isx ? xp : wp), 0,
                                                                      !valid ? 0u : copy ? STG_BYTES : isx ? GM * KSH * 4 : KSH * GN * 4, 0x00020000);
  const unsigned vv = copy ? (unsigned)lane * 16u : isx ? (unsigned)((lane & 15) * KSH + 4 * (lane >> 4)) * 4u : (unsigned)(4 * (lane >> 4) * GN + 4 * (lane & 15)) * 4u;
#pragma unroll
  for (int j = 0; j < 8; ++j) {
    const unsigned so = copy ? (j < 4 ? dst + (unsigned)((j >> 1) * HTB + (j & 1) * FRS) : 0x80000000u)
                             : src + (isx ? (unsigned)((j >> 1) * 16 * KSH + (j & 1) * 16) * 4u : (unsigned)(((j >> 2) * 16 + (j & 3)) * GN) * 4u);
    v[j] = __builtin_bit_cast(f32x4, __builtin_amdgcn_raw_buffer_load_b128(rs, vv, so, 16));
  }
}

#define SEND_UNIT(NPRE, REPORT) do { \
    _Pragma("unroll") for (int i = 0; i < 4; ++i) { \
      if (i == 2) { asm volatile("s_waitcnt vmcnt(%0)" :: "i"(14 + (NPRE)) : "memory"); if (REPORT) progress(seq0 + cur); } \
      SCHED; \
      _Pragma("unroll") for (int d = 0; d < NR - 1; ++d) __builtin_amdgcn_raw_buffer_store_b128(o[i], rd[d], oo[i], 0, AUX_WT); \
      SCHED; } } while (0)

__global__ void __launch_bounds__(512, 2) fused_kernel(Params p) {
  extern __shared__ __attribute__((aligned(16))) char smem[];
  LAS char* lds = (LAS char*)smem;
  const int tid = threadIdx.x, wid = __builtin_amdgcn_readfirstlane(tid >> 6), lane = tid & 63;
  const int b = blockIdx.x, xcd = b & 7, li = b >> 3, rank = p.rank;
  char* myws = p.ws[rank];
  unsigned* tmo = (unsigned*)(myws + WS_TMO);
  const int g4 = lane >> 4, r16 = lane & 15;
  const unsigned vx = (unsigned)(r16 * KSH + 4 * g4) * 4u, vw = (unsigned)(4 * g4 * GN + 4 * r16) * 4u, vd = (unsigned)lane * 16u;
  auto rot = [](int r) INL -> int { return (r * 7) >> 2; };
  auto first_src = [](int cr) INL -> int { return cr > 1 ? (4 * (14 - cr) + 6) / 7 : 0; };
  auto half_of = [](int cr) INL -> int { return ((cr >> 1) < 5 ? cr >> 1 : 5) + 1 - (cr & 1); };
  auto early = [&](int cr) INL -> bool { return ((rank - first_src(cr)) & 7) < 4; };
  auto phase_of = [&](int cr) INL -> int { return (half_of(cr) >> 1) + ((half_of(cr) & 1) && early(cr) ? 1 : 0); };
  auto rho0_of = [&](int cr) INL -> int { return 7 * (half_of(cr) >> 1) + ((half_of(cr) & 1) ? (early(cr) ? 3 : 4) : 0); };
  unsigned odd = 0;
  auto dup_of = [](bool isx) INL -> unsigned { return isx ? NKS * A_STEP : (unsigned)(ROUNDS * NKS) * B_STEP; };
  auto place = [&](bool isx, int col, int q4, int hs, unsigned& src, unsigned& dst) INL {
    const unsigned ks = (unsigned)((hs >> 1) * 8 + rank), kk = (unsigned)((hs & 1) * KHS);
    if (isx) { src = (unsigned)((col * 256 + q4 * 64) * KSH + hs * 32) * 4u; dst = ks * A_STEP + (unsigned)col * PIECE + kk + (unsigned)((q4 >> 1) * HTB + (q4 & 1) * 4 * FRS); }
    else {
      const int rr = col / RCOLS, cc = col - rr * RCOLS;
      const int wcol = col < NRC ? ((rr + ((hs >> 1) >= phase_of(cc) ? 1 : 0)) & 3) * RCOLS + cc : col;
      src = (unsigned)(hs * 32 * GN + wcol * 256 + q4 * 64) * 4u;
      dst = A_BYTES + (col < NRC ? ((unsigned)rr * NKS + ks) * B_STEP + (unsigned)cc * PIECE : B_BYTES + ks * PIECE) + kk + (unsigned)(q4 * 2 * FRS);
    }
    if (odd && ks < (unsigned)NDUP && (isx || col < RCOLS)) dst += dup_of(isx);
  };
  auto load_unit = [&](f32x4 (&v)[8], bool valid, bool isx, unsigned src, auto aux) INL {
    const __amdgpu_buffer_rsrc_t rs = __builtin_amdgcn_make_buffer_rsrc((void*)(isx ? p.x : p.w), 0, valid ? (isx ? GM * KSH * 4 : KSH * GN * 4) : 0, 0x00020000);
    const unsigned vv = isx ? vx : vw;
#pragma unroll
    for (int j = 0; j < 8; ++j)
      v[j] = __builtin_bit_cast(f32x4, __builtin_amdgcn_raw_buffer_load_b128(rs, vv, src + (isx ? (unsigned)((j >> 1) * 16 * KSH + (j & 1) * 16) * 4u : (unsigned)(((j >> 2) * 16 + (j & 3)) * GN) * 4u), decltype(aux)::value));
  };
  auto pack_unit = [&](const f32x4 (&v)[8], float m, bool isx, unsigned dst, u32x4 (&o)[4], unsigned (&oo)[4]) INL {
#pragma unroll
    for (int i = 0; i < 4; ++i) {
      o[i] = isx ? pack8(m, v[2 * i][0], v[2 * i][1], v[2 * i][2], v[2 * i][3], v[2 * i + 1][0], v[2 * i + 1][1], v[2 * i + 1][2], v[2 * i + 1][3])
                 : pack8(m, v[0][i], v[1][i], v[2][i], v[3][i], v[4][i], v[5][i], v[6][i], v[7][i]);
      oo[i] = vd + dst + (unsigned)(isx ? i * FRS : (i >> 1) * HTB + (i & 1) * FRS);
    }
  };

  auto decode = [&](unsigned u, bool& isx, unsigned& src, unsigned& dst) INL {
    int sg, i; isx = false;
    if (u < (unsigned)U1) { sg = u / 240u; i = u % 240u; isx = i < 128; if (!isx) i -= 128; }
    else { const unsigned v = u - U1; sg = 8 + v / 112u; i = v % 112u; }
    place(isx, isx ? i >> 3 : (sg >> 3) * RCOLS + ((i >> 3) + RCOLS - rot(rank)) % RCOLS, i & 3, (sg & 7) * 2 + ((i >> 2) & 1), src, dst);
  };
  const int aid = wid == 0 ? b : NAGENT + b;
  f32x4 v[8]; bool isx = false; unsigned src = 0, dst = 0;
  if (li < 4 && wid == 0) { decode((unsigned)b, isx, src, dst); load_unit(v, true, isx, src, IC<0>{}); }
  if (tid == 0) {
    const unsigned c = __hip_atomic_fetch_add((unsigned*)(myws + WS_CTR + b * 64), 1u, __ATOMIC_RELAXED, __HIP_MEMORY_SCOPE_AGENT) + 1u;
    *(LAS unsigned*)(lds + LDS_BAD + 4) = c; *(LAS unsigned*)(lds + LDS_BAD) = 0u;
  }
  *(LAS unsigned*)(lds + LDS_FLAG + tid * 4) = 0u;
  WAIT_L(0); BAR; asm volatile("" ::: "memory");
  const unsigned call = __builtin_amdgcn_readfirstlane(*(LAS unsigned*)(lds + LDS_BAD + 4));
  const unsigned seq0 = call * SEQ;
  constexpr size_t stg = WS_STG;
  odd = call & 1u;
  if (odd) dst += dup_of(isx);
  const char* flagw = myws + WS_FLAG + (lane & 7) * 128;

  unsigned* prog = (unsigned*)(myws + WS_PROG);
  unsigned* ticket = (unsigned*)(myws + WS_TICKET + (call & 3u) * 128u);
  auto draw = [&](unsigned* ctr, bool real) INL -> unsigned {
    return (unsigned)__builtin_amdgcn_raw_ptr_buffer_atomic_add_i32(1, __builtin_amdgcn_make_buffer_rsrc(ctr, 0, real ? 4 : 0, 0x00020000), (int)vd, 0, 0); };
  auto progress = [&](unsigned val) INL { __builtin_amdgcn_raw_buffer_store_b32(val, __builtin_amdgcn_make_buffer_rsrc(prog + aid, 0, 4, 0x00020000), vd, 0, 16); };
  auto read_go = [&]() INL -> unsigned { return __builtin_amdgcn_readfirstlane(__hip_atomic_load((unsigned*)(myws + WS_GO + 4), __ATOMIC_RELAXED, __HIP_MEMORY_SCOPE_AGENT)); };
  bool copy = false, staged = false;
#define load_send(valid) load_send_(v, valid, copy, isx, src, dst, myws + WS_STG, p.x, p.w, lane)
  auto send_ops = [&](unsigned go, unsigned cur, unsigned tnext, unsigned limit) INL {
    unsigned* ctr = ticket + (go >> 31) * 8; const unsigned base = (go >> 31) ? 0u : 2u * NAGENT;
    const float mx = pow2f((int)((go >> 12) & 0xfffu) - 256), mw = pow2f((int)(go & 0xfffu) - 256);
    __amdgpu_buffer_rsrc_t rd[NR - 1];
#pragma unroll
    for (int d = 0; d < NR - 1; ++d) rd[d] = __builtin_amdgcn_make_buffer_rsrc(p.ws[(rank + 1 + (d + b + wid) % (NR - 1)) & 7] + stg, 0, STG_BYTES, 0x00020000);
    bool real = true, open = go >> 31;
#pragma unroll
    for (int j = 0; j < 8; ++j) asm volatile("" : "+v"(v[j]));
    asm volatile("" : "+v"(tnext));
    while (cur < (unsigned)NUNITS) {
      u32x4 o[4]; unsigned oo[4];
      pack_unit(v, isx ? mx : mw, isx, dst, o, oo);
      if (copy) {
#pragma unroll
        for (int i = 0; i < 4; ++i) o[i] = __builtin_bit_cast(u32x4, v[i]);
      }
      const unsigned raw = __builtin_amdgcn_readfirstlane(tnext) + base, nxt = real ? raw & 0xffffffu : 0xffffffffu;
      open = open || (real && ((raw >> 24) & 1u));
      if (!open && cur >= (unsigned)(NDUP / 8 * 240)) { wave_wait<true>(myws + WS_START + (lane & 7) * 128, false, call - 1u, tmo, call); open = true; }
      SCHED;
      decode(nxt < (unsigned)NUNITS ? nxt : 0u, isx, src, dst);
      staged = staged || (real && (raw >> 25));
      copy = staged && nxt >= (unsigned)U1;
      load_send(nxt < (unsigned)NUNITS);
      real = nxt < limit; tnext = draw(ctr, real);
      SCHED;
      SEND_UNIT(9, true);
      cur = nxt;
    }
    WAIT_V(0);
  };
  auto join_ops = [&](unsigned go, unsigned limit) INL {
    unsigned* ctr = ticket + (go >> 31) * 8;
    const unsigned cur = (__builtin_amdgcn_readfirstlane(draw(ctr, true)) & 0xffffffu) + ((go >> 31) ? 0u : 2u * NAGENT);
    progress(seq0 + (cur < (unsigned)NUNITS ? cur : (unsigned)NUNITS));
    decode(cur < (unsigned)NUNITS ? cur : 0u, isx, src, dst); copy = false; load_send(cur < (unsigned)NUNITS);
    send_ops(go, cur, draw(ctr, true), limit);
  };
  auto send_out = [&]() INL {
    unsigned cur = draw(ticket + 16, true), tnext = draw(ticket + 16, true);
    const bool ok = wave_wait(myws + WS_PCNT, false, call * 16u - 1u, tmo, call);
    cur = __builtin_amdgcn_readfirstlane(cur);
    __amdgpu_buffer_rsrc_t rd[NR - 1];
#pragma unroll
    for (int d = 0; d < NR - 1; ++d) rd[d] = __builtin_amdgcn_make_buffer_rsrc(p.out[(rank + 1 + (d + b + wid) % (NR - 1)) & 7], 0, GM * GN * 4, 0x00020000);
    const unsigned vb = vd + (unsigned)((NRC + rank) * 256 * 4);
    auto orow = [](unsigned u, unsigned i) INL -> unsigned { return ((u * 37u) & 1023u) * 4u + i; };
    auto load_out = [&](unsigned u) INL {
      const __amdgpu_buffer_rsrc_t rs = __builtin_amdgcn_make_buffer_rsrc(p.out[rank], 0, u < (unsigned)NOUT ? GM * GN * 4 : 0, 0x00020000);
#pragma unroll
      for (int i = 0; i < 4; ++i) v[i] = __builtin_bit_cast(f32x4, __builtin_amdgcn_raw_buffer_load_b128(rs, vb, orow(u, i) * (unsigned)(GN * 4), 16));
    };
    load_out(cur);
    if (li >= 2) wave_wait<true>(myws + WS_SENT, false, seq0 + (unsigned)(NUNITS - PSTART), tmo, call);
#pragma unroll
    for (int j = 0; j < 4; ++j) asm volatile("" : "+v"(v[j]));
    asm volatile("" : "+v"(tnext));
    bool real = true;
    while (cur < (unsigned)NOUT) {
      u32x4 o[4]; unsigned oo[4];
#pragma unroll
      for (int i = 0; i < 4; ++i) { o[i] = __builtin_bit_cast(u32x4, v[i]); if (!ok) o[i] = (u32x4){0x7fc00000u, 0x7fc00000u, 0x7fc00000u, 0x7fc00000u}; oo[i] = vb + orow(cur, i) * (unsigned)(GN * 4); asm volatile("" : "+v"(o[i])); }
      const unsigned nxt = real ? __builtin_amdgcn_readfirstlane(tnext) : 0xffffffffu;
      SCHED;
      load_out(nxt);
      real = nxt < (unsigned)NOUT; tnext = draw(ticket + 16, real);
      SCHED;
      SEND_UNIT(5, false);
      cur = nxt;
    }
    WAIT_V(0);
    if (lane < NR) __hip_atomic_fetch_add((unsigned*)(p.ws[lane] + WS_DONE + rank * 128), 1u, __ATOMIC_RELAXED, __HIP_MEMORY_SCOPE_SYSTEM);
  };

  bool ready = false;
  if (li < 4) {
    if (wid == 0) {
      const unsigned limit = li < 2 ? (unsigned)NUNITS : li == 2 ? (unsigned)PLEAVE1 : (unsigned)PLEAVE;
      send_ops((unsigned)(SHIFT0 + 256) << 12 | (unsigned)(SHIFT0 + 256), (unsigned)b, (unsigned)(b - NAGENT), limit);
      unsigned go = 0;
      if (!staged) { wave_wait<true>(myws + WS_GO, false, call - 1u, tmo, call); go = read_go(); }
      if (go >> 31) { progress(seq0); WAIT_V(0); }
      if (lane == 0) __hip_atomic_fetch_add((unsigned*)(myws + WS_DCNT), 1u, __ATOMIC_RELAXED, __HIP_MEMORY_SCOPE_AGENT);
      if (go >> 31) { wave_wait<true>(myws + WS_DCNT, false, call * (unsigned)NAGENT - 1u, tmo, call); join_ops(go, limit); }
      progress(seq0 + (unsigned)(NUNITS + 8));
      if (li < 2) send_out();
    } else if (wid == 2 && li < 2) {
      progress(seq0 + (unsigned)(SEQ - 1));
      wave_wait(myws + WS_SENT, false, seq0 + (unsigned)JOIN2, tmo, call);
      progress(seq0 + (unsigned)JOIN2);
      WAIT_V(0);
      join_ops(read_go(), (unsigned)NUNITS);
      progress(seq0 + (unsigned)(SEQ - 1));
    } else if (wid == 6) {
      wave_wait(myws + WS_SENT, false, seq0 + (unsigned)(PINAT), tmo, call);
      const float mw = pow2f((int)(read_go() & 0xfffu) - 256);
#pragma unroll 1
      for (int d = 1; d < NR; ++d) {
        const int pr = (rank + 1 + (d + b) % (NR - 1)) & 7;
        const __amdgpu_buffer_rsrc_t rp = __builtin_amdgcn_make_buffer_rsrc(p.ws[pr] + stg, 0, STG_BYTES, 0x00020000);
        f32x4 pv[2][8]; unsigned psrc, pdst[2];
#pragma unroll
        for (int e = 0; e < 2; ++e) { place(false, NRC + pr, (b >> 4) * 2 + e, b & 15, psrc, pdst[e]); load_unit(pv[e], true, false, psrc, IC<0>{}); }
#pragma unroll
        for (int e = 0; e < 2; ++e) { u32x4 o[4]; unsigned oo[4]; pack_unit(pv[e], mw, false, pdst[e], o, oo);
#pragma unroll
          for (int i = 0; i < 4; ++i) __builtin_amdgcn_raw_buffer_store_b128(o[i], rp, oo[i], 0, AUX_WT); }
      }
      WAIT_V(0);
      if (lane == 0) __hip_atomic_fetch_add((unsigned*)(myws + WS_PINC), 1u, __ATOMIC_RELAXED, __HIP_MEMORY_SCOPE_AGENT);
    } else if (b == 1 && wid == 7) {
      wave_wait(myws + WS_PKC + (lane & 7) * 128, false, call * (unsigned)NRW - 1u, tmo, call);
      wave_wait(myws + WS_SPD + (lane & 7) * 128, false, call - 1u, tmo, call);
      const unsigned f = __hip_atomic_load((unsigned*)(myws + WS_SPD + (lane & 7) * 128 + 4), __ATOMIC_RELAXED, __HIP_MEMORY_SCOPE_AGENT);
      const unsigned rl = lane & 7;
      unsigned kth = 0, task = rl, word = 0;
#pragma unroll
      for (int j = 0; j < 8; ++j) { const unsigned fj = __builtin_amdgcn_readlane(f, j), rj = __builtin_amdgcn_readlane(rl, j);
        if (((rj ^ rl) & 1u) == 0u && (fj == f ? j < (lane & 7) : (rl & 1u) ? fj < f : fj > f)) ++kth; }
#pragma unroll
      for (int j = 0; j < 8; ++j) { const unsigned fj = __builtin_amdgcn_readlane(f, j), rj = __builtin_amdgcn_readlane(rl, j), kj = __builtin_amdgcn_readlane(kth, j);
        if (((rj ^ rl) & 1u) && kj == kth && ((rl & 1u) ? fj * 1000u > f * SWAPT : f * 1000u > fj * SWAPT)) task = rj; }
#pragma unroll
      for (int j = 0; j < 8; ++j) word |= (unsigned)__builtin_amdgcn_readlane(task, j) << (3 * j);
      if (lane == 0) __hip_atomic_store((unsigned long long*)(myws + WS_MAP), (unsigned long long)word << 32 | call, __ATOMIC_RELAXED, __HIP_MEMORY_SCOPE_AGENT);
    } else if (b == 0 && wid == 7) {
      if (lane < 3) __hip_atomic_store((unsigned*)(myws + WS_TICKET + ((call + 2u) & 3u) * 128u) + lane * 8, 0u, __ATOMIC_RELAXED, __HIP_MEMORY_SCOPE_AGENT);
      unsigned* myflag = (lane & 7) == rank ? (unsigned*)(myws + WS_SENT) : (unsigned*)(p.ws[lane & 7] + WS_FLAG + rank * 128);
      const unsigned long long t0 = __builtin_amdgcn_s_memrealtime();
      { const unsigned st = __hip_atomic_load((unsigned*)(myws + WS_START + (lane & 7) * 128), __ATOMIC_RELAXED, __HIP_MEMORY_SCOPE_SYSTEM);
        const int ahead = __builtin_popcountll(__builtin_amdgcn_ballot_w64(lane < NR && st >= call));
        if (lane == 0) __hip_atomic_store((unsigned*)(myws + WS_LATE), ahead >= LATEN ? call : 0u, __ATOMIC_RELAXED, __HIP_MEMORY_SCOPE_AGENT);
        while (__builtin_amdgcn_s_memrealtime() - t0 < (unsigned long long)LATED) __builtin_amdgcn_s_sleep(2);
        if (lane < NR) __hip_atomic_store((unsigned*)(p.ws[lane] + WS_START + rank * 128), call, __ATOMIC_RELAXED, __HIP_MEMORY_SCOPE_SYSTEM); }
      bool ok = wave_wait<true>(myws + WS_START + (lane & 7) * 128, false, call - 1u, tmo, call);
      if (ok && lane == 0) __hip_atomic_fetch_add(ticket, 1u << 24, __ATOMIC_RELAXED, __HIP_MEMORY_SCOPE_AGENT);
      ok = wave_wait<true>(myws + WS_GO, false, call - 1u, tmo, call) && ok;
      const unsigned redo = read_go() >> 31;
      if (redo) {
        ok = wave_wait<true>(myws + WS_LCNT2, false, call * (unsigned)NRW - 1u, tmo, call) && ok;
        ok = wave_wait<true>(myws + WS_DCNT, false, call * (unsigned)NAGENT - 1u, tmo, call) && ok;
      }
      if (lane == 0) __hip_atomic_store((unsigned*)(myws + WS_FLAG + rank * 128), seq0 + (unsigned)(SEQ - 1), __ATOMIC_RELAXED, __HIP_MEMORY_SCOPE_SYSTEM);
      if (ok && lane == 0) __hip_atomic_fetch_add(ticket + redo * 8, 1u << 25, __ATOMIC_RELAXED, __HIP_MEMORY_SCOPE_AGENT);
      unsigned last = seq0; bool pin = false;
      const unsigned long long t1 = ok ? __builtin_amdgcn_s_memrealtime() : t0;
      while (last < seq0 + (unsigned)(NUNITS + 8) || !pin) {
        unsigned m = 0xffffffffu;
#pragma unroll
        for (int j = 0; j < (NPROG + 63) / 64; ++j) { const int a = j * 64 + lane; if (a < NPROG) { const unsigned x = __hip_atomic_load(prog + a, __ATOMIC_RELAXED, __HIP_MEMORY_SCOPE_AGENT); m = x < m ? x : m; } }
        if (!pin && last > seq0 + (unsigned)(PINAT) && (unsigned)__builtin_amdgcn_readfirstlane(__hip_atomic_load((unsigned*)(myws + WS_PINC), __ATOMIC_RELAXED, __HIP_MEMORY_SCOPE_AGENT)) >= call * 32u) {
          pin = true;
          if (lane < NR) __hip_atomic_store((unsigned*)(p.ws[lane] + WS_PIN + rank * 128), call, __ATOMIC_RELAXED, __HIP_MEMORY_SCOPE_SYSTEM);
        }
#pragma unroll
        for (int sft = 32; sft >= 1; sft >>= 1) { const unsigned x = __shfl_xor(m, sft); m = x < m ? x : m; }
        m = __builtin_amdgcn_readfirstlane(m);
        if (m > last) {
          auto crossed = [&](int x) INL -> bool { return last <= seq0 + (unsigned)x && m > seq0 + (unsigned)x; };
          const bool own = crossed(PINAT) || crossed(JOIN2) || crossed(NUNITS - PSTART);
          if (lane < NR && (own || lane != rank)) __hip_atomic_store(myflag, m, __ATOMIC_RELAXED, __HIP_MEMORY_SCOPE_SYSTEM);
          last = m;
        }
        if (__builtin_amdgcn_s_memrealtime() - t1 > DEADLINE) { __hip_atomic_store(tmo, call, __ATOMIC_RELAXED, __HIP_MEMORY_SCOPE_AGENT); ok = false; break; }
      }
      ok = wave_wait<true>(myws + WS_DONE + (lane & 7) * 128, false, call * (unsigned)NAGENT - 1u, tmo, call) && ok;
      ok = ok && __hip_atomic_load(tmo, __ATOMIC_RELAXED, __HIP_MEMORY_SCOPE_AGENT) != call;
      if (!ok && lane == 0) __hip_atomic_store((unsigned*)p.out[rank] + (NRC + rank) * 256, 0x7fc00000u, __ATOMIC_RELAXED, __HIP_MEMORY_SCOPE_SYSTEM);
    }
    if (li < 2) return;
  } else {
    const __amdgpu_buffer_rsrc_t rl = __builtin_amdgcn_make_buffer_rsrc(myws + stg, 0, STG_BYTES, 0x00020000);
    { const unsigned long long t0 = __builtin_amdgcn_s_memrealtime();
      const unsigned st = __hip_atomic_load((unsigned*)(myws + WS_START + (lane & 7) * 128), __ATOMIC_RELAXED, __HIP_MEMORY_SCOPE_AGENT);
      if (__builtin_popcountll(__builtin_amdgcn_ballot_w64(lane < NR && st >= call)) < LATEN)
        while (__builtin_amdgcn_s_memrealtime() - t0 < (unsigned long long)(PREDELAY * (li - 3))) __builtin_amdgcn_s_sleep(4); }
    float mx = pow2f(SHIFT0), mw = mx;
    ready = true;
    for (int pass = 0; pass < 2; ++pass) {
      unsigned ax = 0, aw = 0;
      for (int j = b - 32; j < 640; j += NRW) {
        isx = j < 128;
        if (j < 128) place(true, 2 * (j & 7) + (wid >> 2), wid & 3, j >> 3, src, dst);
        else if (j < 576) place(false, 2 * ((j - 128) % 28) + (wid >> 2), wid & 3, (j - 128) / 28, src, dst);
        else place(false, NRC + ((j - 576) >> 3), wid & 3, 2 * (j & 7) + (wid >> 2), src, dst);
        if (j >= 128 && j < 576 && (j - 128) % 28 >= 7) load_unit(v, true, isx, src, IC<AUX_NT>{});
        else load_unit(v, true, isx, src, IC<0>{});
        unsigned am = 0;
#pragma unroll
        for (int q = 0; q < 32; ++q) { unsigned a = __builtin_bit_cast(unsigned, v[q >> 2][q & 3]) & 0x7fffffffu; a = a < 0x7f800000u ? a : 0u; am = a > am ? a : am; }
        if (isx) ax = am > ax ? am : ax; else aw = am > aw ? am : aw;
        if (j < 576 || ((j - 576) >> 3) == rank) {
          u32x4 o[4]; unsigned oo[4]; pack_unit(v, isx ? mx : mw, isx, dst, o, oo);
#pragma unroll
          for (int i = 0; i < 4; ++i) __builtin_amdgcn_raw_buffer_store_b128(o[i], rl, oo[i], 0, AUX_WT);
        }
      }
      if (pass == 0) {
#pragma unroll
        for (int sft = 32; sft >= 1; sft >>= 1) { const unsigned a = __shfl_xor(ax, sft), c = __shfl_xor(aw, sft); ax = a > ax ? a : ax; aw = c > aw ? c : aw; }
        if (lane == 0) { *(LAS unsigned*)(lds + LDS_BAD + 32 + wid * 8) = ax; *(LAS unsigned*)(lds + LDS_BAD + 36 + wid * 8) = aw; }
      }
      WAIT_V(0); WAIT_L(0); BAR; asm volatile("" ::: "memory");
      if (pass == 1) break;
      if (wid == 0) {
        if (lane < 2) {
          unsigned a = 0;
#pragma unroll
          for (int q = 0; q < 8; ++q) { const unsigned c = *(LAS unsigned*)(lds + LDS_BAD + 32 + q * 8 + lane * 4); a = c > a ? c : a; }
          __hip_atomic_fetch_max((unsigned long long*)(myws + WS_LMAX) + lane, (unsigned long long)call << 32 | a, __ATOMIC_RELAXED, __HIP_MEMORY_SCOPE_AGENT);
        }
        WAIT_V(0);
        if ((unsigned)__builtin_amdgcn_readfirstlane(draw((unsigned*)(myws + WS_LCNT), true)) == call * (unsigned)NRW - 1u) {
          const unsigned long long mm = __hip_atomic_load((unsigned long long*)(myws + WS_LMAX) + (lane & 1), __ATOMIC_RELAXED, __HIP_MEMORY_SCOPE_AGENT);
          const unsigned e1 = (unsigned)mm ? 1u + ((unsigned)mm >> 23) : 0u;
          const unsigned e12 = (unsigned)__builtin_amdgcn_readlane(e1, 0) << 16 | (unsigned)__builtin_amdgcn_readlane(e1, 1);
          if (lane < NR) __hip_atomic_store((unsigned long long*)(p.ws[lane] + WS_AMX + rank * 128), (unsigned long long)e12 << 32 | call, __ATOMIC_RELAXED, __HIP_MEMORY_SCOPE_SYSTEM);
          wave_wait<true>(myws + WS_AMX + (lane & 7) * 128, false, call - 1u, tmo, call);
          const unsigned e2 = __hip_atomic_load((unsigned*)(myws + WS_AMX + (lane & 7) * 128 + 4), __ATOMIC_RELAXED, __HIP_MEMORY_SCOPE_SYSTEM);
          unsigned gx = e2 >> 16, gw = e2 & 0xffffu;
#pragma unroll
          for (int sft = 4; sft >= 1; sft >>= 1) { const unsigned a = __shfl_xor(gx, sft), c = __shfl_xor(gw, sft); gx = a > gx ? a : gx; gw = c > gw ? c : gw; }
          auto shift_of = [](unsigned g) INL -> int { const int e = (int)g - 128; return (g == 0u || (e >= -SHIFT0 && e < 15 - SHIFT0)) ? SHIFT0 : (14 - e > 126 ? 126 : 14 - e); };
          const int sx = shift_of(__builtin_amdgcn_readfirstlane(gx)), sw = shift_of(__builtin_amdgcn_readfirstlane(gw));
          const bool redo = sx != SHIFT0 || sw != SHIFT0;
          const bool late = __hip_atomic_load((unsigned*)(myws + WS_LATE), __ATOMIC_RELAXED, __HIP_MEMORY_SCOPE_AGENT) == call;
          if (lane == 0) {
            __hip_atomic_store((unsigned long long*)(myws + WS_GO), (unsigned long long)((redo ? 0x80000000u : 0u) | (late ? 0x40000000u : 0u) | (unsigned)(sx + 256) << 12 | (unsigned)(sw + 256)) << 32 | call, __ATOMIC_RELAXED, __HIP_MEMORY_SCOPE_AGENT);
            if (redo) __hip_atomic_fetch_add(ticket, 1u << 20, __ATOMIC_RELAXED, __HIP_MEMORY_SCOPE_AGENT);
          }
        }
        if (!wave_wait(myws + WS_GO, false, call - 1u, tmo, call)) *(LAS unsigned*)(lds + LDS_BAD) = 1u;
        *(LAS unsigned*)(lds + LDS_BAD + 8) = read_go();
        WAIT_L(0);
      }
      asm volatile("" ::: "memory"); BAR; asm volatile("" ::: "memory");
      const unsigned go = __builtin_amdgcn_readfirstlane(*(LAS unsigned*)(lds + LDS_BAD + 8));
      if (!(go >> 31)) break;
      mx = pow2f((int)((go >> 12) & 0xfffu) - 256); mw = pow2f((int)(go & 0xfffu) - 256); ready = false;
    }
    if (tid == 0) __hip_atomic_fetch_add((unsigned*)(myws + WS_LCNT2), 1u, __ATOMIC_RELAXED, __HIP_MEMORY_SCOPE_AGENT);
  }

  const bool isP = li < 4;
  const int wr = wid >> 2, wc = wid & 3;
  const int lr = li - 4;
  const int role = xcd;
  const int mt = isP ? xcd * 2 + (li - 2) : (role >> 1) * 4 + (lr & 3);
  const int cr = isP ? 0 : (role & 1) + 2 * (lr >> 2);
  const unsigned voff = (unsigned)(wid * 2048 + lane * 16), ldsw = (unsigned)wid * 2048u;
  const char* fsrc = myws + WS_FLAG + wid * 128;
  const __amdgpu_buffer_rsrc_t ra = __builtin_amdgcn_make_buffer_rsrc(myws + stg, 0, A_BYTES, 0x00020000);
  const __amdgpu_buffer_rsrc_t rb = __builtin_amdgcn_make_buffer_rsrc(myws + stg + A_BYTES + (isP ? B_BYTES : 0u), 0, isP ? P_BYTES : B_BYTES, 0x00020000);
  const unsigned bstep = isP ? (unsigned)PIECE : B_STEP;
#define STAGE(ldsoff, rs, so) do { \
    __builtin_amdgcn_raw_ptr_buffer_load_lds(rs, (LAS void*)(lds + (ldsoff) + ldsw), 16, voff, (so), 0, AUX_SC1); \
    __builtin_amdgcn_raw_ptr_buffer_load_lds(rs, (LAS void*)(lds + (ldsoff) + ldsw), 16, voff, (so), 1024, AUX_SC1); } while (0)
#define ST1(ldsoff, rs, so, imm) do { __builtin_amdgcn_raw_ptr_buffer_load_lds(rs, (LAS void*)(lds + (ldsoff) + ldsw), 16, voff, (so), imm, AUX_SC1); SCHED; } while (0)
#define RA(h, m) do { _Pragma("unroll") for (int k = 0; k < 2; ++k) At[m][k] = *(const LAS f16x8*)((LAS char*)(size_t)va + (h) * HTB + (m) * FRS + k * KHS); SCHED; } while (0)
#define FLAGDMA() __builtin_amdgcn_global_load_lds((const unsigned*)fsrc, (LAS unsigned*)(lds + LDS_FLAG + wid * 256), 4, 0, AUX_SC1)
#define LDA(dst, h) do { _Pragma("unroll") for (int m = 0; m < 4; ++m) _Pragma("unroll") for (int k = 0; k < 2; ++k) dst[m][k] = *(const LAS f16x8*)((LAS char*)(size_t)va + (h) * HTB + m * FRS + k * KHS); } while (0)
#define LDB(dst, h) do { _Pragma("unroll") for (int n = 0; n < 2; ++n) _Pragma("unroll") for (int k = 0; k < 2; ++k) dst[n][k] = *(const LAS f16x8*)((LAS char*)(size_t)vb + (h) * HTB + n * FRS + k * KHS); } while (0)
#define ACC(ai, nn, m) (128 + (((ai) * 4 + (nn)) * 4 + (m)) * 4)
#define MMA2(ai) do { __builtin_amdgcn_s_waitcnt(0xC07F); SCHED; BAR; SCHED; \
    _Pragma("unroll") for (int m = 0; m < 4; ++m) _Pragma("unroll") for (int q = 0; q < 4; ++q) { const int nn = (m & 1) ? 3 - q : q; \
      asm volatile("v_mfma_f32_16x16x32_f16 v[%c0:%c1], %2, %3, v[%c0:%c1]\n\tv_mfma_f32_16x16x32_f16 v[%c0:%c1], %4, %5, v[%c0:%c1]" \
          :: "i"(ACC(ai, nn, m)), "i"(ACC(ai, nn, m) + 3), "v"(At[m][0]), "v"((nn >> 1) ? B1[nn & 1][0] : B0[nn & 1][0]), "v"(At[m][1]), "v"((nn >> 1) ? B1[nn & 1][1] : B0[nn & 1][1])); } \
    SCHED; BAR; SCHED; } while (0)
#define ZERO_ACC() do { _Pragma("unroll") for (int i_ = 128; i_ < 256; i_ += 2) asm volatile("v_mov_b64 v[%c0:%c1], 0" :: "i"(i_), "i"(i_ + 1)); } while (0)
#define ACC_ROW(dst, ai, m, j) do { float t0_, t1_, t2_, t3_; \
    asm volatile("v_ldexp_f32 %0, v[%c4], %8\n\tv_ldexp_f32 %1, v[%c5], %8\n\tv_ldexp_f32 %2, v[%c6], %8\n\tv_ldexp_f32 %3, v[%c7], %8" \
        : "=&v"(t0_), "=&v"(t1_), "=&v"(t2_), "=&v"(t3_) : "i"(ACC(ai, 0, m) + (j)), "i"(ACC(ai, 1, m) + (j)), "i"(ACC(ai, 2, m) + (j)), "i"(ACC(ai, 3, m) + (j)), "v"(oexp)); \
    dst = (f32x4){t0_, t1_, t2_, t3_}; } while (0)
  asm volatile("" ::: "v128","v129","v130","v131","v132","v133","v134","v135","v136","v137","v138","v139","v140","v141","v142","v143","v144","v145","v146","v147","v148","v149","v150","v151","v152","v153","v154","v155","v156","v157","v158","v159","v160","v161","v162","v163","v164","v165","v166","v167","v168","v169","v170","v171","v172","v173","v174","v175","v176","v177","v178","v179","v180","v181","v182","v183","v184","v185","v186","v187","v188","v189","v190","v191","v192","v193","v194","v195","v196","v197","v198","v199","v200","v201","v202","v203","v204","v205","v206","v207","v208","v209","v210","v211","v212","v213","v214","v215","v216","v217","v218","v219","v220","v221","v222","v223","v224","v225","v226","v227","v228","v229","v230","v231","v232","v233","v234","v235","v236","v237","v238","v239","v240","v241","v242","v243","v244","v245","v246","v247","v248","v249","v250","v251","v252","v253","v254","v255");
  ZERO_ACC();
  f16x8 At[4][2], B0[2][2], B1[2][2];
  const int ph = isP ? 0 : phase_of(cr);
  if (!ready) {
    if (wid == 0) {
      if (isP && !wave_wait(myws + WS_PIN + (lane & 7) * 128, false, call - 1u, tmo, call)) *(LAS unsigned*)(lds + LDS_BAD) = 1u;
      if (!wave_wait(isP ? flagw : myws + WS_FLAG + rank * 128, false, seq0 + (unsigned)(U1 - 1), tmo, call)) *(LAS unsigned*)(lds + LDS_BAD) = 1u;
      *(LAS unsigned*)(lds + LDS_BAD + 8) = read_go(); WAIT_L(0);
    }
    asm volatile("" ::: "memory"); BAR; asm volatile("" ::: "memory");
  }
  const unsigned gow = __builtin_amdgcn_readfirstlane(*(LAS unsigned*)(lds + LDS_BAD + 8));
  const int oexp = 512 - (int)((gow >> 12) & 0xfffu) - (int)(gow & 0xfffu);
  const int NL = (gow >> 30) & 1u ? 0 : 1;
  int rolex = role, mtx = mt, crx = cr;
  const int E0 = 8 + rho0_of(cr);
  int Tend = isP ? 64 : 256;
  unsigned tA, tB, tN;
  auto steps = [&](int t0) INL {
    const int t = t0 + lane, u = t - E0, c = u < 192 ? cr : crx, w = u - 8 * NL - 8;
    const int z = w - 56, zn = 128 - 64 * NL;
    const bool inl = z >= 0 && z < zn;
    const bool local = t < 8 || (u >= 0 && w < 0) || (inl && (z & 7) == 0);
    const int rho = u < 0 ? t - 8 : u >= 192 ? rho0_of(crx) + u - 24 : E0 - 8 + w - (z < 0 ? 0 : inl ? (z >> 3) + 1 : zn >> 3), pos = rho % 7;
    const int sgm = !local ? rho / 7 : t < 8 ? (t < ph ? t : t + 24) : inl ? ph + 8 + (z >> 3) : u < 8 * NL ? ph + 16 + u : ph + (u & 7);
    const int f = first_src(c), sr = local ? rank : (f + pos + (pos >= ((rank - f) & 7) ? 1 : 0)) & 7;
    const unsigned ix = isP ? (unsigned)t : (unsigned)(sgm * 8 + sr);
    tA = t < Tend ? (unsigned)(u < 192 ? mt : mtx) * PIECE + (ix & 63u) * A_STEP + (odd && (ix & 63u) < (unsigned)NDUP ? dup_of(true) : 0u) : 0x80000000u;
    tB = t < Tend ? (unsigned)c * PIECE + ix * bstep + (odd && !isP && ix < (unsigned)NDUP ? dup_of(false) : 0u) : 0x80000000u;
    tN = (isP || t >= Tend || local) ? 0u : seq0 + (unsigned)((sgm < 8 ? 240 * sgm + 128 : U1 - 8 * 112 + 112 * sgm) + 8 * ((c + rot(sr)) % RCOLS) + 8 + sr);
  };
  steps(0);
  __builtin_amdgcn_s_waitcnt(0x0F70); SCHED;
  unsigned par = 0, l1 = PIECE + HTB, a1, fl = 0;
  unsigned a2 = __builtin_amdgcn_readlane(tA, 2), b2 = __builtin_amdgcn_readlane(tB, 2), need = 0;
  unsigned va = (unsigned)(size_t)lds + wr * 4 * FRS + lane * 16, vb = (unsigned)(size_t)lds + 2 * PIECE + wc * 2 * FRS + lane * 16;
  LAS unsigned* fimg = (LAS unsigned*)(lds + LDS_FLAG + (lane & 7) * 256); asm volatile("" : "+v"(fimg));
  { const unsigned a0 = __builtin_amdgcn_readlane(tA, 0), b0 = __builtin_amdgcn_readlane(tB, 0), b1 = __builtin_amdgcn_readlane(tB, 1); a1 = __builtin_amdgcn_readlane(tA, 1);
    STAGE(2 * PIECE, rb, b0); STAGE(0, ra, a0); STAGE(2 * PIECE + HTB, rb, b0 + HTB); STAGE(HTB, ra, a0 + HTB);
    if (wr == 1) BAR;
    WAIT_V(2); BAR;
    FLAGDMA();
    STAGE(3 * PIECE, rb, b1); STAGE(PIECE, ra, a1); STAGE(3 * PIECE + HTB, rb, b1 + HTB);
    a1 += HTB;
    BAR; }
  const unsigned vpk = (unsigned)tid * 16u;
  const unsigned vring = (unsigned)(size_t)lds + LDS_RING + wid * 1024 + lane * 16;
#define KLOOP(ADD, LIM) for (;;) { \
      f32x4 pt; \
      ST1(l1, ra, a1, 0); LDB(B0, 0); SCHED; ST1(l1, ra, a1, 1024); LDB(B1, 1); SCHED; \
      if (ADD) { __builtin_amdgcn_raw_ptr_buffer_load_lds(rpk, (LAS void*)(lds + LDS_RING + rs_ + wid * 1024), 16, vpk, (unsigned)(li3 - a0_) * 8192u, 0, AUX_SC1 | AUX_NT); SCHED; \
        rs_ = rs_ == 16384u ? 0u : rs_ + 8192u; pt = *(const LAS f32x4*)(size_t)(vring + rs_); } \
      else FLAGDMA(); \
      SCHED; LDA(At, 0); WAIT_V(10); \
      MMA2(0); \
      if ((unsigned)__builtin_amdgcn_readlane(fl, need & 63u) < need) { \
        const unsigned f2 = *(const volatile LAS unsigned*)fimg; fl = f2 > fl ? f2 : fl;     \
        if ((unsigned)__builtin_amdgcn_readlane(fl, need & 63u) < need) { \
          if (!wave_wait(flagw, (unsigned)(lane & 7) != (need & 7u), need - 1u, tmo, call, &fl)) *(LAS unsigned*)(lds + LDS_BAD) = 1u; \
          *(volatile LAS unsigned*)fimg = fl; } \
      } \
      ST1(2 * PIECE + par, rb, b2, 0); RA(1, 0); ST1(2 * PIECE + par, rb, b2, 1024); RA(1, 1); \
      ST1(par, ra, a2, 0); RA(1, 2); ST1(par, ra, a2, 1024); RA(1, 3); \
      const unsigned fn = ADD ? 0u : *fimg; \
      if (ADD && li3 - a0_ >= 2) { unsigned keep_; \
        asm volatile("s_mov_b32 %0, m0\n\ts_set_gpr_idx_on %1, gpr_idx(SRC0,DST)\n\tv_add_f32 v128, v128, %2\n\tv_add_f32 v129, v129, %3\n\tv_add_f32 v130, v130, %4\n\tv_add_f32 v131, v131, %5\n\ts_set_gpr_idx_off\n\ts_mov_b32 m0, %0" \
            : "=&s"(keep_) : "s"(((li3 - a0_ - 2) & 31) * 4), "v"(pt[0]), "v"(pt[1]), "v"(pt[2]), "v"(pt[3])); } \
      SCHED; \
      ST1(2 * PIECE + par + HTB, rb, b2 + HTB, 0); \
      l1 = par + HTB; a1 = a2 + HTB; va ^= PIECE; vb ^= PIECE; \
      a2 = __builtin_amdgcn_readlane(tA, li3); need = __builtin_amdgcn_readlane(tN, li3); \
      asm volatile("" : "+v"(va), "+v"(vb), "+s"(a2), "+s"(need), "+s"(l1), "+s"(a1)); SCHED; \
      ST1(2 * PIECE + par + HTB, rb, b2 + HTB, 1024); \
      par ^= PIECE; b2 = __builtin_amdgcn_readlane(tB, li3); ++li3; \
      fl = fn > fl ? fn : fl; \
      asm volatile("" : "+s"(par), "+s"(li3), "+s"(b2), "+v"(fl)); \
      WAIT_V(9); \
      MMA2(1); \
      if (li3 >= (LIM)) break; }
  int t = 0;
  steps(3);
  float* const myout = p.out[rank] + (size_t)(wr * 64) * GN + wc * 64;
  for (int sg = isP ? 4 : -NL; sg < 5; ++sg) {
    const int Ts = sg == 4 ? Tend : sg == -NL ? E0 : t + (sg <= 0 ? 8 : (NL && sg == 3) ? 56 : 64);
    const int slot = sg <= 0 ? sg + NL : (sg == 4 ? 0 : 1);
    char* pk = myws + WS_PARK + (size_t)(slot * 256 + (sg == 4 ? rolex : role) * 32 + li) * 262144;
    const __amdgpu_buffer_rsrc_t rpk = __builtin_amdgcn_make_buffer_rsrc(pk, 0, 262144, 0x00020000);
    const int left = Ts - t; int li3 = 0;
    unsigned rs_ = 0;
    const bool add = !isP && (sg == 4 || (NL && sg == 3));
    const bool ref = sg == 2 && lr == ((role & 1) ? 4 : 0);
    const int a1_ = 29 - E0 - (sg == 4 ? ADDOFF : ADDOFF2),
              a0_ = ref ? 32 : (add && a1_ > 0) ? (a1_ < left - 35 ? a1_ : left - 35) : 0;
    const unsigned r0_ = (unsigned)__builtin_amdgcn_s_memrealtime(); __builtin_amdgcn_s_waitcnt(0xC07F);
    for (int part = a0_ ? 0 : 1; part < 2; ++part) {
      if (part == 1 && add) { KLOOP(true, a0_ + 34); }
      const int lim_ = part ? left : a0_;
      KLOOP(false, lim_);
      if (!part) {
        if (ref && tid == 0) __hip_atomic_store((unsigned long long*)(myws + WS_SPD + xcd * 128),
            (unsigned long long)((1u << 28) / __builtin_elementwise_max((unsigned)__builtin_amdgcn_s_memrealtime() - r0_, 256u)) << 32 | call, __ATOMIC_RELAXED, __HIP_MEMORY_SCOPE_AGENT);
      }
    }
    t = Ts;
    if (wr == 0) BAR;
    asm volatile("s_nop 15\n\ts_nop 15" ::: "memory");
    SCHED;
    if (sg <= 0) {
      unsigned vq = vpk; asm volatile("" : "+v"(vq));
      if (sg == -NL) {
#pragma unroll
      for (int i = 0; i < 32; ++i)
        asm volatile("s_nop 4\n\tglobal_store_dwordx4 %0, v[%c1:%c2], %3" " sc1" :: "v"(vq + i * 8192u), "i"(128 + 4 * i), "i"(128 + 4 * i + 3), "s"(pk) : "memory");
      } else {
#pragma unroll
      for (int i = 0; i < 32; ++i)
        asm volatile("s_nop 4\n\tglobal_store_dwordx4 %0, v[%c1:%c2], %3" " nt" :: "v"(vq + i * 8192u), "i"(128 + 4 * i), "i"(128 + 4 * i + 3), "s"(pk) : "memory");
      }
      asm volatile("s_nop 1");
      ZERO_ACC();
      steps(t + 3);
      if (sg == -NL) {
        WAIT_V(0);
        if (lane == 0) __hip_atomic_fetch_add((unsigned*)(myws + WS_PKC + xcd * 128), 1u, __ATOMIC_RELAXED, __HIP_MEMORY_SCOPE_AGENT);
      }
    } else {
      if (sg == 2) {
        unsigned mp = 0;
        if (!wave_wait(myws + WS_MAP + (lane & 1) * 4, lane & 1, call - 1u, tmo, call, &mp)) *(LAS unsigned*)(lds + LDS_BAD) = 1u;
        else rolex = (int)(((unsigned)__builtin_amdgcn_readlane(mp, 1) >> (3 * xcd)) & 7u);
        mtx = (rolex >> 1) * 4 + (lr & 3); crx = (rolex & 1) + 2 * (lr >> 2); Tend = E0 + 192 + 56 - rho0_of(crx);
      }
      if (__builtin_amdgcn_readfirstlane(*(const LAS unsigned*)(lds + LDS_BAD)) != 0u) {
#pragma unroll
        for (int i = 128; i < 256; ++i) asm volatile("v_mov_b32 v[%c0], 0x7fc00000" :: "i"(i));
      }
      const int nt = isP ? NRC + rank : sg == 4 ? crx : sg * RCOLS + cr;
      const __amdgpu_buffer_rsrc_t rc = __builtin_amdgcn_make_buffer_rsrc(myout + (size_t)((sg == 4 ? mtx : mt) * 256) * GN + nt * 256, 0, 192u * GN * 4u, 0x00020000);
      unsigned vo = (unsigned)((lane >> 4) * 4 * GN + (lane & 15) * 4) * 4u; asm volatile("" : "+v"(vo));
#pragma unroll
      for (int ai = 0; ai < 2; ++ai)
#pragma unroll
        for (int m = 0; m < 4; ++m) {
#pragma unroll
          for (int j = 0; j < 4; ++j) {
            f32x4 v4; ACC_ROW(v4, ai, m, j);
            __builtin_amdgcn_raw_buffer_store_b128(__builtin_bit_cast(u32x4, v4), rc, vo + (unsigned)((ai * 128 + m * 16 + j) * GN * 4), 0, AUX_NT);
          }
#pragma unroll
          for (int nn = 0; nn < 4; ++nn)
            asm volatile("v_mov_b64 v[%c0:%c1], 0\n\tv_mov_b64 v[%c2:%c3], 0" :: "i"(ACC(ai, nn, m)), "i"(ACC(ai, nn, m) + 1), "i"(ACC(ai, nn, m) + 2), "i"(ACC(ai, nn, m) + 3));
          if (ai == 0 && m == 1) {
            SCHED; steps(t + 3); asm volatile("" : "+v"(tA), "+v"(tB), "+v"(tN)); SCHED;
          }
        }
    }
    if (wr == 1) BAR;
  }
  WAIT_V(0);
  if (wr == 0) BAR;
  if (isP) {
    BAR; asm volatile("" ::: "memory");
    if (tid == 0) {
      __builtin_amdgcn_fence(__ATOMIC_RELEASE, "agent");
      WAIT_V(0);
      __hip_atomic_fetch_add((unsigned*)(myws + WS_PCNT), 1u, __ATOMIC_RELAXED, __HIP_MEMORY_SCOPE_AGENT);
    }
    if (wid == 0) {
      if (li == 2) {
        progress(seq0 + (unsigned)PLEAVE1);
        WAIT_V(0);
        join_ops(gow, (unsigned)NUNITS);
        progress(seq0 + (unsigned)(NUNITS + 8));
      }
      send_out();
    }
  }
}
}

extern "C" void kernel_launch(int rank, int nranks, ncclComm_t comm, hipStream_t stream,
                              void* const* d_in, const int* in_sizes, int n_in,
                              void* d_out, int out_size, const int* in_dtypes, int out_dtype,
                              void* const* d_out_all, void* const* d_ws_all, size_t ws_bytes) {
  (void)comm; (void)in_dtypes; (void)out_dtype; (void)d_out;
  if (nranks != NR || (unsigned)rank >= (unsigned)NR || n_in < 2 || ws_bytes < WS_NEED || out_size != GM * GN ||
      in_sizes[0] != GM * KSH || in_sizes[1] != KSH * GN) { fprintf(stderr, "unexpected problem shape\n"); return; }
  static bool attr_set = false;
  if (!attr_set) { HIP_CHECK(hipFuncSetAttribute((const void*)fused_kernel, hipFuncAttributeMaxDynamicSharedMemorySize, LDS_BYTES)); attr_set = true; }
  Params p;
  p.x = (const float*)d_in[0]; p.w = (const float*)d_in[1]; p.rank = rank;
  for (int r = 0; r < NR; ++r) { p.out[r] = (float*)d_out_all[r]; p.ws[r] = (char*)d_ws_all[r]; }
  fused_kernel<<<256, 512, LDS_BYTES, stream>>>(p);
  HIP_CHECK(hipGetLastError());
}
```
